```python
import numpy as np

kernel_name = "gpu_gemm_a2a_m4096_k4096_n16384_e4m3e4m3_dqsilu"


ABSMAX_RTOL = 0.03

COMPUTE_DTYPE = "float32"
EPILOGUE = "dequant_silu"
IN_QUANT = None
OUT_QUANT = None
IN_FP8 = "e4m3e4m3"
IN_DTYPES = {"x": "e4m3", "w_mat": "e4m3",
             "scale_x": "float32", "scale_w": "float32"}
WORLD_SIZE = 8

def _snap_fp8(x, mant_bits, max_abs, min_normal):
    a = np.abs(x.astype(np.float64))
    _m, _e = np.frexp(a)
    q = np.where(a >= min_normal, np.ldexp(1.0, _e - 1 - mant_bits),
                 min_normal * 2.0 ** -mant_bits)
    snapped = np.minimum(np.round(a / q) * q, max_abs)
    return (np.sign(x) * snapped).astype(np.float32)

def setup_inputs():

    x = _snap_fp8(np.random.randn(4096, 4096).astype(np.float32), 3, 448.0, 2.0 ** -6)
    w_mat = _snap_fp8(np.random.randn(4096, 16384).astype(np.float32), 3, 448.0, 2.0 ** -6)
    scale_x = (0.02 * (1.0 + np.random.rand(1))).astype(np.float32)
    scale_w = (0.02 * (1.0 + np.random.rand(1))).astype(np.float32)
    return {"x": x, "w_mat": w_mat, "scale_x": scale_x, "scale_w": scale_w}

def reference(x, w_mat, scale_x, scale_w):
    acc = (x @ w_mat).astype(np.float32)
    y = (acc * (scale_x * scale_w)).astype(np.float32)
    return (y / (1.0 + np.exp(-np.clip(y, -60.0, 60.0)))).astype(np.float32)
```

```cpp
#include <hip/hip_runtime.h>
#include <rccl/rccl.h>
#include <cstdio>

#define HIP_CHECK(expr) do { hipError_t _e = (expr); if (_e != hipSuccess) { \
    fprintf(stderr, "HIP error %s at %s:%d\n", hipGetErrorString(_e), __FILE__, __LINE__); \
    return; } } while (0)

namespace {

#define LAS __attribute__((address_space(3)))
typedef int v8i __attribute__((ext_vector_type(8)));
typedef int v4i __attribute__((ext_vector_type(4)));
typedef int v2i __attribute__((ext_vector_type(2)));
typedef float f32x4 __attribute__((ext_vector_type(4)));
typedef unsigned u32x4 __attribute__((ext_vector_type(4)));

constexpr int NR = 8;
constexpr int KD = 4096, ND = 16384;
constexpr int MS = 512, NS = 2048;
constexpr int BM = 128, BN = 256, BK = 128;
constexpr int NSTEP = KD / BK;
constexpr int MT = MS / BM;
constexpr int NWG = 256, NTHREADS = 768;
constexpr int NSLOT = 3;
constexpr int A_BYTES = BM * BK, E_BYTES = 16 * BK, B_BYTES = BK * BN;
constexpr int E_OFF = A_BYTES, B_OFF = A_BYTES + E_BYTES, SLOT_BYTES = B_OFF + B_BYTES;
constexpr int SYNC_OFF = NSLOT * SLOT_BYTES;
constexpr int LDS_BYTES = SYNC_OFF + 128;

constexpr int NPUSH = 16;
constexpr int PGROUPS = NPUSH / 4, NUNIT = MT * NSTEP, PUNITS = NUNIT / PGROUPS;
static_assert(PUNITS * 4096 <= SYNC_OFF && PUNITS == 32, "staged shard share fits below the sync words");

constexpr size_t UNIT = A_BYTES;
constexpr size_t SHARD = (size_t)MS * KD;
constexpr size_t XBUF = (size_t)NR * SHARD;
constexpr size_t WS_FLAG = 2 * XBUF;
constexpr size_t WS_CNT = WS_FLAG + 65536;
constexpr size_t WS_TMO = WS_CNT + (size_t)NWG * 8 * 128;
constexpr size_t WS_DBG = WS_TMO + 4096;
constexpr size_t WS_END = WS_DBG + (size_t)NWG * 64 * 8;
constexpr unsigned long long DEADLINE = 50000000ull;

struct Params {
    const unsigned char* x;
    const unsigned char* w;
    const float* sx; const float* sw;
    float* out;
    unsigned char* ws[NR];
    int rank;
    int mode;
};

template <int N> __device__ __forceinline__ void wait_vm() { asm volatile("s_waitcnt vmcnt(%0)" :: "i"(N) : "memory"); }
template <int I> struct IC { static constexpr int v = I; };
template <int I, int E, class F> __device__ __forceinline__ void static_for(F&& f) { if constexpr (I < E) { f(IC<I>{}); static_for<I + 1, E>(f); } }
__device__ __forceinline__ unsigned lds_ld(unsigned addr) {
    unsigned v; asm volatile("ds_read_b32 %0, %1\n\ts_waitcnt lgkmcnt(0)" : "=v"(v) : "v"(addr) : "memory"); return v;
}
__device__ __forceinline__ void lds_st(unsigned addr, unsigned v) { asm volatile("ds_write_b32 %0, %1" :: "v"(addr), "v"(v) : "memory"); }
template <bool SYS, int OFF> __device__ __forceinline__ void glds(const unsigned char* base, unsigned voff, unsigned dst) {
    if (SYS) asm volatile("s_add_u32 m0, %2, %3\n\ts_nop 0\n\tglobal_load_lds_dwordx4 %0, %1 sc0 sc1"
                          :: "v"(voff), "s"(base), "s"(dst), "i"(OFF) : "memory", "scc");
    else     asm volatile("s_add_u32 m0, %2, %3\n\ts_nop 0\n\tglobal_load_lds_dwordx4 %0, %1"
                          :: "v"(voff), "s"(base), "s"(dst), "i"(OFF) : "memory", "scc");
}
__device__ __forceinline__ unsigned ticket_issue(unsigned char* me, int role) {
    unsigned t = 0;
    unsigned char* a = me + WS_CNT + ((size_t)blockIdx.x * 8 + role) * 128;
    if ((threadIdx.x & 63) == 0) asm volatile("global_atomic_add %0, %1, %2, off sc0" : "+v"(t) : "v"(a), "v"(1u) : "memory");
    return t;
}
#define STAMP(i) do {} while (0)
#define STAMPC(i) do {} while (0)
#define DIAG_MODE(bit) false

#define POLL_SLEEP 2
#define TILEMAP 2
#define TILE_INDEX(b) ((((b) - NPUSH) & 7) * 30 + (((b) - NPUSH) >> 3))
#define DIAG_LOCAL(b) ((b) < NPUSH || (TILE_INDEX(b) >> 3) >= 28)

__device__ __forceinline__ void unit_of(int u, int& t, int& k) { t = u & (MT - 1); k = u >> 2; }

__global__ __launch_bounds__(NTHREADS) void fused_ag_gemm(Params p) {
    extern __shared__ __attribute__((aligned(16))) unsigned char smem[];
    const int tid = threadIdx.x, lane = tid & 63, wid = __builtin_amdgcn_readfirstlane(tid >> 6);
    const unsigned lds0 = (unsigned)(size_t)(LAS unsigned char*)smem;
    unsigned char* me = p.ws[p.rank];
    const int b = blockIdx.x;
    const unsigned aoff0 = (unsigned)(lane >> 3) * KD + (unsigned)(((lane & 7) ^ (lane >> 4)) << 4);
    const unsigned aoff1 = (unsigned)(lane >> 3) * KD + (unsigned)(((lane & 7) ^ (4 + (lane >> 4))) << 4);

    if (tid < 32) *(LAS unsigned*)(smem + SYNC_OFF + tid * 4) = 0u;
    if (wid == 0) { STAMP(0); STAMPC(4); }

    if (b < NPUSH) {
        const int g = b >> 2, j = b & 3;
        if (wid >= 8) {
            const int L = wid - 8;
#pragma unroll 1
            for (int i0 = 0; i0 < PUNITS / 4; ++i0) {
                const int i = i0 * 4 + L;
                int ut, uk; unit_of(g + i * PGROUPS, ut, uk);
                const unsigned char* sp = p.x + (size_t)(ut * BM + j * 32) * KD + uk * BK;
                static_for<0, 4>([&](auto qc) {
                    constexpr int q = decltype(qc)::v;
                    glds<false, q * 1024>(sp, ((q & 1) ? aoff1 : aoff0) + q * 8 * KD, lds0 + i * 4096);
                });
            }
            __builtin_amdgcn_s_barrier();
            static_for<0, PUNITS / 4>([&](auto ic) {
                constexpr int i0 = decltype(ic)::v;
                wait_vm<(PUNITS / 4 - 1 - i0) * 4>();
                lds_st(lds0 + SYNC_OFF + L * 4, (unsigned)(i0 + 1));
            });
            return;
        }
        if (wid == 7 || DIAG_MODE(2)) { __syncthreads(); return; }
        unsigned tk = ticket_issue(me, wid);
        const int dest = (p.rank + 1 + (wid + b) % 7) & (NR - 1);
        __syncthreads();
        asm volatile("s_waitcnt vmcnt(0)" : "+v"(tk) :: "memory");
        const unsigned call = __builtin_amdgcn_readfirstlane(tk) + 1u;
        unsigned char* dst = p.ws[dest] + (size_t)(call & 1u) * XBUF + (size_t)p.rank * SHARD + (size_t)j * 4096;
        unsigned char* fl = p.ws[dest] + WS_FLAG + ((size_t)p.rank * NUNIT * 4 + j) * 4;
        const unsigned voff = (unsigned)lane * 16u;
        unsigned char* fprev = fl;
        for (int i = 0; i < PUNITS; ++i) {
            while (__builtin_amdgcn_readfirstlane(lds_ld(lds0 + SYNC_OFF + (i & 3) * 4)) <= (unsigned)(i >> 2)) __builtin_amdgcn_s_sleep(1);
            u32x4 r0, r1, r2, r3;
            asm volatile("ds_read_b128 %0, %4\n\tds_read_b128 %1, %4 offset:1024\n\tds_read_b128 %2, %4 offset:2048\n\t"
                         "ds_read_b128 %3, %4 offset:3072\n\ts_waitcnt lgkmcnt(0)"
                         : "=&v"(r0), "=&v"(r1), "=&v"(r2), "=&v"(r3) : "v"(lds0 + (unsigned)i * 4096u + voff) : "memory");
            int ut, uk; unit_of(g + i * PGROUPS, ut, uk);
            unsigned char* o = dst + (size_t)(ut * NSTEP + uk) * UNIT;
            wait_vm<3>();
            asm volatile("global_store_dwordx4 %0, %1, %3 sc0 sc1\n\tglobal_store_dwordx4 %0, %2, %3 offset:1024 sc0 sc1\n\ts_nop 1"
                         :: "v"(voff), "v"(r0), "v"(r1), "s"(o) : "memory");
            wait_vm<2>();
            if (i > 0 && lane == 0) asm volatile("global_store_dword %0, %1, %2 sc0 sc1" :: "v"(0u), "v"(call), "s"(fprev) : "memory");
            asm volatile("global_store_dwordx4 %0, %1, %3 offset:2048 sc0 sc1\n\tglobal_store_dwordx4 %0, %2, %3 offset:3072 sc0 sc1\n\ts_nop 1"
                         :: "v"(voff), "v"(r2), "v"(r3), "s"(o) : "memory");
            fprev = fl + (size_t)(ut * NSTEP + uk) * 16;
        }
        wait_vm<0>();
        if (lane == 0) asm volatile("global_store_dword %0, %1, %2 sc0 sc1" :: "v"(0u), "v"(call), "s"(fprev) : "memory");
        wait_vm<0>();
        STAMP(8 + wid);
        return;
    }

    const int tile = TILE_INDEX(b), ms = tile >> 3, nt = tile & 7;
    const bool local = ms >= 28;
    const int t = local ? ms - 28 : (ms & 3);
    const int src = local ? p.rank : ((p.rank + 1 + (ms >> 2)) & (NR - 1));
    const bool has_extra = t < 2;
    const int xrow = local ? 256 + 16 * t : 288 + 16 * ((ms >> 2) * 2 + t);

    if (wid == 11) { __builtin_amdgcn_s_barrier(); return; }
    if (wid >= 8) {
        const int li = wid - 8;
        const unsigned slot = lds0 + li * SLOT_BYTES;
        unsigned boff[4];
#pragma unroll
        for (int q = 0; q < 4; ++q) {
            const int kk = (q & 1) * 4 + (lane >> 4), f = kk | ((q >> 1) << 3);
            boff[q] = (unsigned)(lane >> 4) * ND + (unsigned)(((lane & 15) ^ f) << 4);
        }
        const unsigned char* wb = p.w + nt * BN;
        unsigned call = 0, tk = 0;
        if (!local) tk = ticket_issue(me, li);
        const unsigned char* xr = me + (size_t)src * SHARD + (size_t)t * NSTEP * UNIT;
        const unsigned char* xl = p.x + (size_t)t * BM * KD;
        const unsigned char* xe = p.x + (size_t)xrow * KD;
        const unsigned char* fl = me + WS_FLAG + (size_t)(src * MT + t) * NSTEP * 16 + (lane & 3) * 4;
        unsigned* tmo = (unsigned*)(me + WS_TMO);
        bool dead = false;
        for (int s = li; s < NSTEP; s += NSLOT) {
            if (s >= NSLOT) {
                const unsigned need = (unsigned)(s - NSLOT + 1);
                while (!__all(lds_ld(lds0 + SYNC_OFF + 32 + (lane & 7) * 4) >= need)) __builtin_amdgcn_s_sleep(1);
            }
            if (DIAG_MODE(4)) { if (s == li) __builtin_amdgcn_s_barrier(); lds_st(lds0 + SYNC_OFF + li * 4, (unsigned)(s + 1)); continue; }
            if (has_extra) {
                glds<false, E_OFF>(xe + s * BK, aoff0, slot);
                glds<false, E_OFF + 1024>(xe + s * BK, aoff1 + 8 * KD, slot);
            }
            unsigned fv = 0;
            if (!local) asm volatile("global_load_dword %0, %1, off sc0 sc1" : "=v"(fv) : "v"(fl + s * 16) : "memory");
            const unsigned char* ws_ = wb + (size_t)s * BK * ND;
            static_for<0, 32>([&](auto qc) {
                constexpr int q = decltype(qc)::v;
                glds<false, B_OFF + q * 1024>(ws_, boff[(q & 1) | (((q >> 2) & 1) << 1)] + q * 4 * ND, slot);
            });
            if (s == li) __builtin_amdgcn_s_barrier();
            if (local) {
                static_for<0, 16>([&](auto qc) {
                    constexpr int q = decltype(qc)::v;
                    glds<false, q * 1024>(xl + s * BK, ((q & 1) ? aoff1 : aoff0) + q * 8 * KD, slot);
                });
            } else {
                asm volatile("s_waitcnt vmcnt(32)" : "+v"(fv), "+v"(tk) :: "memory");
                if (s == li) { call = __builtin_amdgcn_readfirstlane(tk) + 1u; xr += (size_t)(call & 1u) * XBUF; }
                if (DIAG_MODE(1)) fv = call;
                if (!dead && !__all(fv >= call)) {
                    const unsigned long long t0 = __builtin_amdgcn_s_memrealtime();
                    for (unsigned polls = 1;; ++polls) {
                        fv = __hip_atomic_load((const unsigned*)(fl + s * 16), __ATOMIC_RELAXED, __HIP_MEMORY_SCOPE_SYSTEM);
                        if (__all(fv >= call)) break;
                        if ((polls & 63u) == 0u
                            && (__builtin_amdgcn_readfirstlane(__hip_atomic_load(tmo, __ATOMIC_RELAXED, __HIP_MEMORY_SCOPE_AGENT)) == call
                                || __builtin_amdgcn_s_memrealtime() - t0 > DEADLINE)) {
                            if (lane == 0) __hip_atomic_store(tmo, call, __ATOMIC_RELAXED, __HIP_MEMORY_SCOPE_AGENT);
                            dead = true;
                            break;
                        }
                        __builtin_amdgcn_s_sleep(POLL_SLEEP);
                    }
                }
                STAMP(16 + s);
                if (!dead) {
                    static_for<0, 16>([&](auto qc) {
                        constexpr int q = decltype(qc)::v;
                        glds<true, q * 1024>(xr + (size_t)s * UNIT, (unsigned)lane * 16u + q * 1024, slot);
                    });
                }
            }
            wait_vm<0>();
            if (dead) lds_st(lds0 + SYNC_OFF + 64, 1u);
            lds_st(lds0 + SYNC_OFF + li * 4, (unsigned)(s + 1));
        }
        return;
    }

    const int wr = wid >> 2, wc = wid & 3, fr = lane & 15, fq = lane >> 4;
    float scale = p.sx[0] * p.sw[0];
    const unsigned e0 = (unsigned)(fr * 128 + ((fq ^ ((fr >> 1) & 7)) << 4)), a0 = e0 + wr * 64 * 128;
    unsigned bo[4];
    {
        const int q = fr >> 1, f = q | ((fq & 1) << 3);
#pragma unroll
        for (int n = 0; n < 4; ++n) bo[n] = (unsigned)(B_OFF + (16 * fq + q) * 256 + (((wc * 4 + (n ^ (2 * wr))) ^ f) << 4) + (fr & 1) * 8);
    }
    f32x4 acc[4][4], accx[2];
#pragma unroll
    for (int m = 0; m < 4; ++m)
#pragma unroll
        for (int n = 0; n < 4; ++n) acc[m][n] = (f32x4){0.f, 0.f, 0.f, 0.f};
    accx[0] = accx[1] = (f32x4){0.f, 0.f, 0.f, 0.f};
    __syncthreads();
    int slot = 0;
    for (int s = 0; s < NSTEP; ++s) {
        while (__builtin_amdgcn_readfirstlane(lds_ld(lds0 + SYNC_OFF + slot * 4)) < (unsigned)(s + 1)) __builtin_amdgcn_s_sleep(1);
        if (DIAG_MODE(8)) { lds_st(lds0 + SYNC_OFF + 32 + wid * 4, (unsigned)(s + 1)); slot = (slot == NSLOT - 1) ? 0 : slot + 1; continue; }
        LAS unsigned char* sb = (LAS unsigned char*)smem + slot * SLOT_BYTES;
        v8i af[4], bf[4], ef;
#pragma unroll
        for (int n = 0; n < 4; ++n) {
            const v2i t0 = __builtin_amdgcn_ds_read_tr8_b64_v2i32((LAS v2i*)(sb + bo[n]));
            const v2i t1 = __builtin_amdgcn_ds_read_tr8_b64_v2i32((LAS v2i*)(sb + bo[n] + 8 * 256));
            const v2i t2 = __builtin_amdgcn_ds_read_tr8_b64_v2i32((LAS v2i*)(sb + bo[n] + 64 * 256));
            const v2i t3 = __builtin_amdgcn_ds_read_tr8_b64_v2i32((LAS v2i*)(sb + bo[n] + 72 * 256));
            bf[n] = (v8i){t0.x, t0.y, t1.x, t1.y, t2.x, t2.y, t3.x, t3.y};
        }
#pragma unroll
        for (int m = 0; m < 4; ++m) {
            const v4i lo = *(LAS v4i*)(sb + a0 + m * 2048), hi = *(LAS v4i*)(sb + (a0 ^ 64u) + m * 2048);
            af[m] = (v8i){lo.x, lo.y, lo.z, lo.w, hi.x, hi.y, hi.z, hi.w};
        }
        if (has_extra) {
            const v4i lo = *(LAS v4i*)(sb + E_OFF + e0), hi = *(LAS v4i*)(sb + E_OFF + (e0 ^ 64u));
            ef = (v8i){lo.x, lo.y, lo.z, lo.w, hi.x, hi.y, hi.z, hi.w};
        }
        asm volatile("s_waitcnt lgkmcnt(0)" ::: "memory");
        lds_st(lds0 + SYNC_OFF + 32 + wid * 4, (unsigned)(s + 1));
        __builtin_amdgcn_sched_barrier(0);
        __builtin_amdgcn_s_setprio(1);
#pragma unroll
        for (int m = 0; m < 4; ++m)
#pragma unroll
            for (int n = 0; n < 4; ++n)
                acc[m][n] = __builtin_amdgcn_mfma_scale_f32_16x16x128_f8f6f4(af[m], bf[n], acc[m][n], 0, 0, 0, 0, 0, 0);
        if (has_extra) {
            accx[0] = __builtin_amdgcn_mfma_scale_f32_16x16x128_f8f6f4(ef, bf[0], accx[0], 0, 0, 0, 0, 0, 0);
            accx[1] = __builtin_amdgcn_mfma_scale_f32_16x16x128_f8f6f4(ef, bf[1], accx[1], 0, 0, 0, 0, 0, 0);
        }
        __builtin_amdgcn_s_setprio(0);
        __builtin_amdgcn_sched_barrier(0);
        slot = (slot == NSLOT - 1) ? 0 : slot + 1;
    }
    if (wid == 0) { STAMP(1); STAMPC(5); }
    if (__builtin_amdgcn_readfirstlane(lds_ld(lds0 + SYNC_OFF + 64)) != 0u) scale = __builtin_nanf("");
    const float zs = scale * -1.44269504088896340736f, zmax = 60.f * 1.44269504088896340736f;
    auto act = [&](float a) {
        const float e = __builtin_amdgcn_exp2f(__builtin_amdgcn_fmed3f(a * zs, -zmax, zmax));
        return (a * scale) * __builtin_amdgcn_rcpf(1.0f + e);
    };
    const unsigned ovoff = (unsigned)((lane >> 5) * 8 * NS + (lane & 31)) * 4u;
    auto store_pair = [&](const f32x4& c0, const f32x4& c1, const float* rowp) {
#pragma unroll
        for (int j = 0; j < 4; ++j) {
            const auto r = __builtin_amdgcn_permlane16_swap(__float_as_uint(act(c0[j])), __float_as_uint(act(c1[j])), false, false);
            asm volatile("global_store_dword %0, %1, %2 sc1" :: "v"(ovoff), "v"(r[0]), "s"(rowp + (size_t)j * NS) : "memory");
            asm volatile("global_store_dword %0, %1, %2 sc1" :: "v"(ovoff), "v"(r[1]), "s"(rowp + (size_t)(j + 4) * NS) : "memory");
        }
    };
    const float* ob = p.out + (size_t)(src * MS + t * BM + wr * 64) * NS + nt * BN + wc * 64;
#pragma unroll
    for (int m = 0; m < 4; ++m) {
        store_pair(acc[m][0], acc[m][1], ob + (size_t)m * 16 * NS + wr * 32);
        store_pair(acc[m][2], acc[m][3], ob + (size_t)m * 16 * NS + (wr ^ 1) * 32);
    }
    if (has_extra) store_pair(accx[0], accx[1], p.out + (size_t)(p.rank * MS + xrow) * NS + nt * BN + wc * 64 + wr * 32);
}

}

extern "C" void kernel_launch(int rank, int nranks, ncclComm_t comm,
                              hipStream_t stream,
                              void* const* d_in, const int* in_sizes, int n_in,
                              void* d_out, int out_size,
                              const int* in_dtypes, int out_dtype,
                              void* const* d_out_all,
                              void* const* d_ws_all, size_t ws_bytes) {
    (void)comm; (void)in_dtypes; (void)out_dtype; (void)d_out_all;
    if (nranks != NR || (unsigned)rank >= (unsigned)NR || n_in != 4 || ws_bytes < WS_END || in_sizes[0] != MS * KD
        || in_sizes[1] != KD * ND || out_size != NR * MS * NS) {
        fprintf(stderr, "kernel_launch: unexpected problem shape\n");
        return;
    }
    HIP_CHECK(hipFuncSetAttribute((const void*)fused_ag_gemm, hipFuncAttributeMaxDynamicSharedMemorySize, LDS_BYTES));
    Params p;
    p.x = (const unsigned char*)d_in[0];
    p.w = (const unsigned char*)d_in[1] + (size_t)rank * NS;
    p.sx = (const float*)d_in[2];
    p.sw = (const float*)d_in[3];
    p.out = (float*)d_out;
    for (int i = 0; i < NR; ++i) p.ws[i] = (unsigned char*)d_ws_all[i];
    p.rank = rank;
    p.mode = 0;
    hipLaunchKernelGGL(fused_ag_gemm, dim3(NWG), dim3(NTHREADS), LDS_BYTES, stream, p);
    HIP_CHECK(hipGetLastError());
}
```

```cpp
#include <hip/hip_runtime.h>
#include <rccl/rccl.h>
#include <cstdio>

#define HIP_CHECK(expr) do { hipError_t _e = (expr); if (_e != hipSuccess) { \
    fprintf(stderr, "HIP error %s at %s:%d\n", hipGetErrorString(_e), __FILE__, __LINE__); \
    return; } } while (0)

namespace {

#define LAS __attribute__((address_space(3)))
typedef int v8i __attribute__((ext_vector_type(8)));
typedef int v4i __attribute__((ext_vector_type(4)));
typedef int v2i __attribute__((ext_vector_type(2)));
typedef float f32x4 __attribute__((ext_vector_type(4)));
typedef unsigned u32x4 __attribute__((ext_vector_type(4)));

constexpr int NR = 8;
constexpr int KD = 4096, ND = 16384;
constexpr int MS = 512, NS = 2048;
constexpr int BM = 128, BN = 256, BK = 128;
constexpr int NSTEP = KD / BK;
constexpr int MT = MS / BM;
constexpr int NWG = 256, NTHREADS = 768;
constexpr int NSLOT = 3;
constexpr int A_BYTES = BM * BK, E_BYTES = 16 * BK, B_BYTES = BK * BN;
constexpr int E_OFF = A_BYTES, B_OFF = A_BYTES + E_BYTES, SLOT_BYTES = B_OFF + B_BYTES;
constexpr int SYNC_OFF = NSLOT * SLOT_BYTES;
constexpr int SY_BAD = SYNC_OFF;
constexpr int SY_PROG = SYNC_OFF, SY_META = SYNC_OFF + 64;
constexpr int LDS_BYTES = SYNC_OFF + 256;

constexpr int NPUSH = 16;
constexpr int NITEM = MT * NSTEP * 4;
constexpr int NSTATIC = 4 * NPUSH;
constexpr int NDRAW = 512;
static_assert(NITEM - NSTATIC + 4 * NPUSH == NDRAW && 32 * 4096 <= SYNC_OFF, "draw count; staging ring");

constexpr size_t UNIT = A_BYTES;
constexpr size_t SHARD = (size_t)MS * KD;
constexpr size_t XBUF = (size_t)NR * SHARD;
constexpr size_t WS_FLAG = 2 * XBUF;
constexpr size_t WS_CNT = WS_FLAG + 65536;
constexpr size_t WS_TMO = WS_CNT + (size_t)NWG * 8 * 128;
constexpr size_t WS_DRAW = WS_TMO + 1024;
constexpr size_t WS_DBG = WS_TMO + 4096;
constexpr size_t WS_TRACE = WS_DBG + (size_t)NWG * 64 * 8;
constexpr size_t WS_END = WS_TRACE + 8192;
constexpr unsigned long long DEADLINE = 50000000ull;

struct Params {
    const unsigned char* x;
    unsigned char* me;
    const unsigned char* w;
    float* out;
    unsigned char* peer[NR - 1];
    int rank;
    int mode;
    const float* sx; const float* sw;
};

template <int N> __device__ __forceinline__ void wait_vm() { asm volatile("s_waitcnt vmcnt(%0)" :: "i"(N) : "memory"); }
template <int I> struct IC { static constexpr int v = I; };
template <int I, int E, class F> __device__ __forceinline__ void static_for(F&& f) { if constexpr (I < E) { f(IC<I>{}); static_for<I + 1, E>(f); } }
__device__ __forceinline__ unsigned lds_ld(unsigned addr) {
    unsigned v; asm volatile("ds_read_b32 %0, %1\n\ts_waitcnt lgkmcnt(0)" : "=v"(v) : "v"(addr) : "memory"); return v;
}
__device__ __forceinline__ void lds_st(unsigned addr, unsigned v) { asm volatile("ds_write_b32 %0, %1" :: "v"(addr), "v"(v) : "memory"); }
#define GLDS_BITS ""
template <bool SYS, int OFF> __device__ __forceinline__ void glds(const unsigned char* base, unsigned voff, unsigned dst) {
    if (SYS) asm volatile("s_add_u32 m0, %2, %3\n\ts_nop 0\n\tglobal_load_lds_dwordx4 %0, %1 sc0 sc1"
                          :: "v"(voff), "s"(base), "s"(dst), "i"(OFF) : "memory", "scc");
    else     asm volatile("s_add_u32 m0, %2, %3\n\ts_nop 0\n\tglobal_load_lds_dwordx4 %0, %1" GLDS_BITS
                          :: "v"(voff), "s"(base), "s"(dst), "i"(OFF) : "memory", "scc");
}
__device__ __forceinline__ unsigned ticket_issue(unsigned char* me, int role) {
    unsigned t = 0;
    unsigned char* a = me + WS_CNT + ((size_t)blockIdx.x * 8 + role) * 128;
    if ((threadIdx.x & 63) == 0) asm volatile("global_atomic_add %0, %1, %2, off sc0" : "+v"(t) : "v"(a), "v"(1u) : "memory");
    return t;
}
#define STAMP(i) do {} while (0)
#define STAMPC(i) do {} while (0)
#define DIAG_MODE(bit) false

#define TRACE_WG 255
#define TRACE_LA 17
#define BAR_AT 1
#define MPRIO1 1
#define MPRIO2 2
#define WARM 0
#define FREE_START 1
#define POLL_EARLY 0
#define STAGE_AHEAD 5
#define POLL_SLEEP 2
#define TILEMAP 2
#define TILE_INDEX(b) ((((b) - NPUSH) & 7) * 30 + (((b) - NPUSH) >> 3))
#define DIAG_GROUP(b) ((b) < NPUSH ? -1 : (TILE_INDEX(b) >> 3) >= 28 ? 4 : ((TILE_INDEX(b) >> 3) & 3))
#define DIAG_LOCAL(b) ((b) < NPUSH || (TILE_INDEX(b) >> 3) >= 28)

#define ORD_R 0
#define ORD_FIN 870, 870, 1000, 1000
struct Order { unsigned q[NITEM]; };
constexpr Order make_order() {
    constexpr int fin[4] = {ORD_FIN};
    Order o{}; int key[NITEM] = {}; int n = 0;
    for (int k = 0; k < NSTEP; ++k)
        for (int t = 0; t < MT; ++t)
            for (int j = 0; j < 4; ++j) {
                int due = fin[t] * 2 * (k + 1);
                if (fin[t] == 1000 && k >= NSTEP - ORD_R) due = 1000 * (2 * (NSTEP - ORD_R) + (j >= 2 ? ORD_R : 0) + (k - (NSTEP - ORD_R)) + 1);
                if (k < 4) due = 0;
                int i = n++;
                while (i > 0 && key[i - 1] > due) { key[i] = key[i - 1]; o.q[i] = o.q[i - 1]; --i; }
                key[i] = due; o.q[i] = (unsigned)((t * NSTEP + k) * 4 + j);
            }
    return o;
}
__constant__ const Order k_order = make_order();
__device__ __forceinline__ int static_item(int id) { return (((id >> 2) & 3) * NSTEP + (id >> 4)) * 4 + (id & 3); }
__device__ __forceinline__ size_t item_src(int q) { return (size_t)((q >> 7) * BM + (q & 3) * 32) * KD + ((q >> 2) & 31) * BK; }

__global__ __launch_bounds__(NTHREADS) void fused_ag_gemm(Params p) {
    extern __shared__ __attribute__((aligned(16))) unsigned char smem[];
    const int tid = threadIdx.x, lane = tid & 63, wid = __builtin_amdgcn_readfirstlane(tid >> 6);
    const unsigned lds0 = (unsigned)(size_t)(LAS unsigned char*)smem;
    unsigned char* me = p.me;
    const int b = blockIdx.x;
    const unsigned aoff0 = (unsigned)(lane >> 3) * KD + (unsigned)(((lane & 7) ^ (lane >> 4)) << 4);
    const unsigned aoff1 = (unsigned)(lane >> 3) * KD + (unsigned)(((lane & 7) ^ (4 + (lane >> 4))) << 4);

    if (DIAG_MODE(32)) return;
    if (wid == 0) { STAMP(0); STAMPC(4); }

    if (__builtin_expect(b < NPUSH, 1)) {
        if (tid < 64) *(LAS unsigned*)(smem + SYNC_OFF + tid * 4) = 0u;
        if (wid >= 8) {
            const int L = wid - 8;
            __builtin_amdgcn_s_barrier();
            if (DIAG_MODE(2)) return;
#pragma unroll 1
            for (int i = L ? L : 4;; i += 4) {
                if (i > STAGE_AHEAD) {
                    for (;;) {
                        const unsigned pr = lds_ld(lds0 + SY_PROG + (lane % 7) * 4);
                        if (__any(pr >= (unsigned)(i - STAGE_AHEAD)) && __all(pr + 32u > (unsigned)i)) break;
                        __builtin_amdgcn_s_sleep(2);
                    }
                }
                int q;
                if (i < 4) q = static_item(i * NPUSH + b);
                else {
                    unsigned d = 0;
                    if (lane == 0) d = __hip_atomic_fetch_add((unsigned*)(me + WS_DRAW), 1u, __ATOMIC_RELAXED, __HIP_MEMORY_SCOPE_AGENT);
                    const unsigned id = (__builtin_amdgcn_readfirstlane(d) & (NDRAW - 1)) + NSTATIC;
                    if (id >= (unsigned)NITEM) { lds_st(lds0 + SY_META + (i & 31) * 4, (unsigned)(i + 1) << 16 | 0xffffu); return; }
                    q = (int)k_order.q[id];
                }
                const unsigned char* sp = p.x + item_src(q);
                static_for<0, 4>([&](auto qc) {
                    constexpr int c = decltype(qc)::v;
                    glds<false, c * 1024>(sp, ((c & 1) ? aoff1 : aoff0) + c * 8 * KD, lds0 + (i & 31) * 4096);
                });
                wait_vm<0>();
                lds_st(lds0 + SY_META + (i & 31) * 4, (unsigned)(i + 1) << 16 | (unsigned)q);
            }
        }
        if (wid == 7 || DIAG_MODE(2)) { __syncthreads(); return; }
        unsigned char* peer = p.peer[(wid + b) % 7];
        const int rank = p.rank;
        int q = static_item(b);
        u32x4 r0, r1, r2, r3;
        asm volatile("global_load_dwordx4 %0, %4, %8\n\tglobal_load_dwordx4 %1, %5, %8\n\t"
                     "global_load_dwordx4 %2, %6, %8\n\tglobal_load_dwordx4 %3, %7, %8"
                     : "=&v"(r0), "=&v"(r1), "=&v"(r2), "=&v"(r3)
                     : "v"(aoff0), "v"(aoff1 + 8 * KD), "v"(aoff0 + 16 * KD), "v"(aoff1 + 24 * KD), "s"(p.x + item_src(q)) : "memory");
        unsigned tk = ticket_issue(me, wid);
        unsigned char* fl = peer - (size_t)rank * SHARD + WS_FLAG + (size_t)rank * NITEM * 4;
        asm volatile("s_waitcnt lgkmcnt(0)\n\ts_barrier\n\ts_waitcnt vmcnt(0)" : "+v"(tk), "+v"(r0), "+v"(r1), "+v"(r2), "+v"(r3) :: "memory");
        const unsigned call = __builtin_amdgcn_readfirstlane(tk) + 1u;
        unsigned char* dst = peer + (size_t)(call & 1u) * XBUF;
        const unsigned voff = (unsigned)lane * 16u;
        unsigned char* fprev = nullptr;
        unsigned ended = 0;
        for (int i = 0; ended != 15u; ++i) {
            if (i > 0) {
                if (ended >> (i & 3) & 1u) continue;
                unsigned m;
                while (((m = __builtin_amdgcn_readfirstlane(lds_ld(lds0 + SY_META + (i & 31) * 4))) >> 16) != (unsigned)(i + 1)) __builtin_amdgcn_s_sleep(1);
                q = (int)(m & 0xffffu);
                if (q == 0xffff) { ended |= 1u << (i & 3); lds_st(lds0 + SY_PROG + wid * 4, (unsigned)i); continue; }
                asm volatile("ds_read_b128 %0, %4\n\tds_read_b128 %1, %4 offset:1024\n\tds_read_b128 %2, %4 offset:2048\n\t"
                             "ds_read_b128 %3, %4 offset:3072\n\ts_waitcnt lgkmcnt(0)"
                             : "=&v"(r0), "=&v"(r1), "=&v"(r2), "=&v"(r3) : "v"(lds0 + (unsigned)(i & 31) * 4096u + voff) : "memory");
                lds_st(lds0 + SY_PROG + wid * 4, (unsigned)i);
            }
            unsigned char* o = dst + (size_t)q * 4096;
            if (i >= FREE_START) wait_vm<3>();
            if (i == 0 && wid == 0) STAMP(6);
            asm volatile("global_store_dwordx4 %0, %1, %3 sc0 sc1\n\tglobal_store_dwordx4 %0, %2, %3 offset:1024 sc0 sc1\n\ts_nop 1"
                         :: "v"(voff), "v"(r0), "v"(r1), "s"(o) : "memory");
            if (i > 0) wait_vm<2>();
            if (i == 1 && wid == 0) STAMP(7);
            if (i > 0 && lane == 0) asm volatile("global_store_dword %0, %1, %2 sc0 sc1" :: "v"(0u), "v"(call), "s"(fprev) : "memory");
            asm volatile("global_store_dwordx4 %0, %1, %3 offset:2048 sc0 sc1\n\tglobal_store_dwordx4 %0, %2, %3 offset:3072 sc0 sc1\n\ts_nop 1"
                         :: "v"(voff), "v"(r2), "v"(r3), "s"(o) : "memory");
            fprev = fl + (size_t)q * 4;
        }
        wait_vm<0>();
        if (lane == 0) asm volatile("global_store_dword %0, %1, %2 sc0 sc1" :: "v"(0u), "v"(call), "s"(fprev) : "memory");
        wait_vm<0>();
        STAMP(8 + wid);
        return;
    }

    const int tile = TILE_INDEX(b), ms = tile >> 3, nt = tile & 7;
    const bool local = ms >= 28;
    const int t = local ? ms - 28 : (ms & 3);
    const int src = local ? p.rank : ((p.rank + 1 + (ms >> 2)) & (NR - 1));
    const bool has_extra = t < 2;
    const int xrow = local ? 256 + 16 * t : 288 + 16 * ((ms >> 2) * 2 + t);

    if (wid >= 9) {
        unsigned boff[4];
#pragma unroll
        for (int q = 0; q < 4; ++q) {
            const int kk = (q & 1) * 4 + (lane >> 4), f = kk | ((q >> 1) << 3);
            boff[q] = (unsigned)(lane >> 4) * ND + (unsigned)(((lane & 15) ^ f) << 4);
        }
        const unsigned char* wb = p.w + nt * BN;
        const unsigned char* xe = p.x + (size_t)xrow * KD;
        auto run = [&](auto q0c, auto q1c, auto exc) {
            constexpr int Q0 = decltype(q0c)::v, Q1 = decltype(q1c)::v, NI = Q1 - Q0 + 2 * decltype(exc)::v;
            unsigned nslot = lds0;
            auto issue = [&](int k) {
                const unsigned slot = nslot;
                nslot = slot == lds0 + (NSLOT - 1) * SLOT_BYTES ? lds0 : slot + SLOT_BYTES;
                if (DIAG_MODE(4)) return;
                const unsigned char* wk = wb + (size_t)k * BK * ND;
                static_for<Q0, Q1>([&](auto qc) {
                    constexpr int q = decltype(qc)::v;
                    glds<false, B_OFF + q * 1024>(wk, boff[(q & 1) | (((q >> 2) & 1) << 1)] + q * 4 * ND, slot);
                });
                if (decltype(exc)::v) {
                    glds<false, E_OFF>(xe + k * BK, aoff0, slot);
                    glds<false, E_OFF + 1024>(xe + k * BK, aoff1 + 8 * KD, slot);
                }
            };
            issue(0); issue(1); issue(2);
            wait_vm<2 * NI>(); __builtin_amdgcn_s_barrier();
            wait_vm<NI>(); __builtin_amdgcn_s_barrier();
#pragma unroll 1
            for (int s = 1; s < NSTEP - 2; ++s) { issue(s + 2); wait_vm<NI>(); __builtin_amdgcn_s_barrier(); }
            wait_vm<0>(); __builtin_amdgcn_s_barrier();
        };
        if (wid == 9) run(IC<0>{}, IC<11>{}, IC<0>{});
        else if (wid == 10) run(IC<11>{}, IC<22>{}, IC<0>{});
        else if (has_extra) run(IC<22>{}, IC<32>{}, IC<1>{});
        else run(IC<22>{}, IC<32>{}, IC<0>{});
        return;
    }
    if (wid == 8) {
        if (lane == 0) lds_st(lds0 + SY_BAD, 0u);
        unsigned call = 0, tk = 0, fv = 0;
        if (!local) tk = ticket_issue(me, 0);
        const unsigned char* xr = me + (size_t)src * SHARD + (size_t)t * NSTEP * UNIT;
        const unsigned char* xl = p.x + (size_t)t * BM * KD;
        const unsigned char* fl = me + WS_FLAG + (size_t)(src * MT + t) * NSTEP * 16 + lane * 4;
        unsigned* tmo = (unsigned*)(me + WS_TMO);
        bool dead = false, pend = false;
        const bool noflags = local || DIAG_MODE(1);
        unsigned long long mask = 0;
        int base = 0, pbase = 0, next = 0;
        auto known = [&](int k) { return noflags || dead || (k - base < 16 && ((mask >> (4 * (k - base))) & 15u) == 15u); };
        auto poll = [&](int k) {
            asm volatile("global_load_dword %0, %1, off sc0 sc1" : "=v"(fv) : "v"(fl + k * 16) : "memory");
            pbase = k;
        };
        auto take = [&]() {
            asm volatile("s_waitcnt vmcnt(0)" : "+v"(fv), "+v"(tk) :: "memory");
            if (!call) { call = __builtin_amdgcn_readfirstlane(tk) + 1u; xr += (size_t)(call & 1u) * XBUF; }
            mask = __ballot(fv >= call); base = pbase;
        };
        unsigned nslot = lds0;
#define LTR(v, i) do {} while (0)
        auto issue = [&](int k) {
            LTR(lt0, k);
            const unsigned slot = nslot;
            nslot = slot == lds0 + (NSLOT - 1) * SLOT_BYTES ? lds0 : slot + SLOT_BYTES;
            if (DIAG_MODE(4) || dead) return;
            if (local)
                static_for<0, 16>([&](auto qc) {
                    constexpr int q = decltype(qc)::v;
                    glds<false, q * 1024>(xl + k * BK, ((q & 1) ? aoff1 : aoff0) + q * 8 * KD, slot);
                });
            else
                static_for<0, 16>([&](auto qc) {
                    constexpr int q = decltype(qc)::v;
                    glds<true, q * 1024>(xr + (size_t)k * UNIT, (unsigned)lane * 16u + q * 1024, slot);
                });
        };
        if (!local && DIAG_MODE(1)) { poll(0); take(); }
#pragma unroll 1
        for (int s = -1; s < NSTEP - 1; ++s) {
            const int lim = s < 0 ? 2 : (s + 2 < NSTEP ? s + 2 : NSTEP - 1);
            if (pend) { take(); pend = false; }
            while (next <= lim && known(next)) issue(next++);
            if (next <= s + 1) {
                const unsigned long long t0 = __builtin_amdgcn_s_memrealtime();
                for (unsigned polls = 1;; ++polls) {
                    poll(next); take();
                    if (known(next)) break;
                    if ((polls & 63u) == 0u
                        && (__builtin_amdgcn_readfirstlane(__hip_atomic_load(tmo, __ATOMIC_RELAXED, __HIP_MEMORY_SCOPE_AGENT)) == call
                            || __builtin_amdgcn_s_memrealtime() - t0 > DEADLINE)) {
                        if (lane == 0) { __hip_atomic_store(tmo, call, __ATOMIC_RELAXED, __HIP_MEMORY_SCOPE_AGENT); lds_st(lds0 + SY_BAD, 1u); }
                        dead = true;
                        break;
                    }
                    __builtin_amdgcn_s_sleep(POLL_SLEEP);
                }
                STAMP(16 + next);
                while (next <= lim && known(next)) issue(next++);
            }
            if (next > s + 2 && !dead) wait_vm<16>(); else wait_vm<0>();
            LTR(lt1, s + 1);
            if (next < NSTEP && !known(next)) { poll(next); pend = true; }
            asm volatile("s_waitcnt lgkmcnt(0)\n\ts_barrier" ::: "memory");
            LTR(lt2, s + 1);
        }
        wait_vm<0>();
        return;
    }

    const int wr = wid >> 2, wc = wid & 3;
    unsigned sx = __builtin_amdgcn_readfirstlane(__float_as_uint(p.sx[0])), sw = __builtin_amdgcn_readfirstlane(__float_as_uint(p.sw[0]));
    int ex = 2 - t, top = 1 - wr;
    f32x4 acc[4][4], accx[2];
#pragma unroll
    for (int m = 0; m < 4; ++m)
#pragma unroll
        for (int n = 0; n < 4; ++n) acc[m][n] = (f32x4){0.f, 0.f, 0.f, 0.f};
    accx[0] = accx[1] = (f32x4){0.f, 0.f, 0.f, 0.f};
    const float* ob = p.out + (size_t)(src * MS + t * BM + wr * 64) * NS + nt * BN + wc * 64;
    const float* ox = p.out + (size_t)(p.rank * MS + xrow) * NS + nt * BN + wc * 64 + wr * 32;
    asm volatile("" : "+s"(ob), "+s"(ox), "+s"(sx), "+s"(sw), "+s"(ex), "+s"(top));
    unsigned aa[2], ea[2], bo[4];
    {
        const int fr = lane & 15, fq = lane >> 4, q = fr >> 1, f = q | ((fq & 1) << 3);
        aa[0] = lds0 + (unsigned)((wr * 64 + fr) * 128 + ((fq ^ ((fr >> 1) & 7)) << 4));
        aa[1] = aa[0] ^ 64u;
        ea[0] = aa[0] + (unsigned)(E_OFF - wr * 8192);
        ea[1] = ea[0] ^ 64u;
#pragma unroll
        for (int n = 0; n < 4; ++n) bo[n] = lds0 + (unsigned)((16 * fq + q) * 256 + (((wc * 4 + (n ^ (2 * wr))) ^ f) << 4) + (fr & 1) * 8);
    }
    asm volatile("s_barrier" ::: "memory");
    {
        int slot = 0;
#pragma unroll 1
        for (int s = 0; s < NSTEP; ++s) {
            v8i af[4], bf[4], ef;
#pragma unroll
            for (int n = 0; n < 4; ++n) {
                LAS unsigned char* q = (LAS unsigned char*)(size_t)bo[n] + B_OFF;
                const v2i t0 = __builtin_amdgcn_ds_read_tr8_b64_v2i32((LAS v2i*)q);
                const v2i t1 = __builtin_amdgcn_ds_read_tr8_b64_v2i32((LAS v2i*)(q + 8 * 256));
                const v2i t2 = __builtin_amdgcn_ds_read_tr8_b64_v2i32((LAS v2i*)(q + 64 * 256));
                const v2i t3 = __builtin_amdgcn_ds_read_tr8_b64_v2i32((LAS v2i*)(q + 72 * 256));
                bf[n] = (v8i){t0.x, t0.y, t1.x, t1.y, t2.x, t2.y, t3.x, t3.y};
            }
#pragma unroll
            for (int m = 0; m < 4; ++m) {
                const v4i lo = *(LAS v4i*)((LAS unsigned char*)(size_t)aa[0] + m * 2048), hi = *(LAS v4i*)((LAS unsigned char*)(size_t)aa[1] + m * 2048);
                af[m] = (v8i){lo.x, lo.y, lo.z, lo.w, hi.x, hi.y, hi.z, hi.w};
            }
            asm volatile("" : "+s"(ex));
            if (ex > 0) {
                const v4i lo = *(LAS v4i*)(size_t)ea[0], hi = *(LAS v4i*)(size_t)ea[1];
                ef = (v8i){lo.x, lo.y, lo.z, lo.w, hi.x, hi.y, hi.z, hi.w};
            }
            asm volatile("" ::: "memory");
            __builtin_amdgcn_s_waitcnt(0xC07F);
            slot = (slot == NSLOT - 1) ? 0 : slot + 1;
            const int delta = slot ? SLOT_BYTES : -(NSLOT - 1) * SLOT_BYTES;
            int bar = s < NSTEP - 1 ? 1 + top : 0;
            asm volatile("" : "+s"(bar));
            if (bar == 1) asm volatile("s_barrier" ::: "memory");
            __builtin_amdgcn_sched_barrier(0);
            __builtin_amdgcn_s_setprio(MPRIO1);
#pragma unroll
            for (int m = 0; m < 4; ++m) {
#pragma unroll
                for (int n = 0; n < 4; ++n) {
                    acc[m][n] = __builtin_amdgcn_mfma_scale_f32_16x16x128_f8f6f4(bf[n], af[m], acc[m][n], 0, 0, 0, 0, 0, 0);
                    if (m == 0 && n == 0) { __builtin_amdgcn_sched_barrier(0); __builtin_amdgcn_s_setprio(MPRIO2); __builtin_amdgcn_sched_barrier(0); }
                    if (m == 3 && n == BAR_AT) {
                        __builtin_amdgcn_sched_barrier(0);
                        asm volatile("" : "+s"(bar));
                        if (bar == 2) asm volatile("s_barrier" ::: "memory");
                        __builtin_amdgcn_sched_barrier(0);
                    }
                }
                __builtin_amdgcn_sched_barrier(0);
                if (m == 0) asm volatile("v_add_u32 %0, %4, %0\n\tv_add_u32 %1, %4, %1\n\tv_add_u32 %2, %4, %2\n\tv_add_u32 %3, %4, %3"
                                         : "+v"(bo[0]), "+v"(bo[1]), "+v"(bo[2]), "+v"(bo[3]) : "s"(delta));
                if (m == 1) asm volatile("v_add_u32 %0, %4, %0\n\tv_add_u32 %1, %4, %1\n\tv_add_u32 %2, %4, %2\n\tv_add_u32 %3, %4, %3"
                                         : "+v"(aa[0]), "+v"(aa[1]), "+v"(ea[0]), "+v"(ea[1]) : "s"(delta));
                __builtin_amdgcn_sched_barrier(0);
            }
            asm volatile("" : "+s"(ex));
            if (ex > 0) {
                accx[0] = __builtin_amdgcn_mfma_scale_f32_16x16x128_f8f6f4(bf[0], ef, accx[0], 0, 0, 0, 0, 0, 0);
                accx[1] = __builtin_amdgcn_mfma_scale_f32_16x16x128_f8f6f4(bf[1], ef, accx[1], 0, 0, 0, 0, 0, 0);
            }
            __builtin_amdgcn_s_setprio(0);
            __builtin_amdgcn_sched_barrier(0);
        }
    }
    if (wid == 0) { STAMP(1); STAMPC(5); }
    STAMP(48 + wid);
    float scale = __uint_as_float(sx) * __uint_as_float(sw);
    if (__builtin_amdgcn_readfirstlane(lds_ld(lds0 + SY_BAD)) != 0u) scale = __builtin_nanf("");
    const int fr = lane & 15, fq = lane >> 4;
    const float zs = scale * -1.44269504088896340736f, zmax = 60.f * 1.44269504088896340736f;
    LAS unsigned char* tb = (LAS unsigned char*)smem + ((NSTEP - 3 + (wc >> 1)) % NSLOT) * SLOT_BYTES + wr * 8192 + (wc & 1) * 4096;
    typedef float f32x2 __attribute__((ext_vector_type(2)));
    f32x2 e[8];
    f32x4 tv[4];
    auto expo = [&](auto nqc, const f32x4* c) {
        constexpr int NQ = decltype(nqc)::v;
#pragma unroll
        for (int h = 0; h < 2 * NQ; ++h) {
            f32x2 z = (f32x2){c[h >> 1][(h & 1) * 2], c[h >> 1][(h & 1) * 2 + 1]} * zs;
            asm("" : "+v"(z));
            e[h].x = __builtin_amdgcn_exp2f(__builtin_amdgcn_fmed3f(z.x, -zmax, zmax));
            e[h].y = __builtin_amdgcn_exp2f(__builtin_amdgcn_fmed3f(z.y, -zmax, zmax));
        }
        __builtin_amdgcn_sched_barrier(0);
    };
    auto stage = [&](auto nqc, const f32x4* c) {
        constexpr int NQ = decltype(nqc)::v;
#pragma unroll
        for (int q = 0; q < NQ; ++q) {
            f32x2 d0 = e[2 * q] + 1.0f, d1 = e[2 * q + 1] + 1.0f;
            asm("" : "+v"(d0)); asm("" : "+v"(d1));
            const f32x4 r = {__builtin_amdgcn_rcpf(d0.x), __builtin_amdgcn_rcpf(d0.y), __builtin_amdgcn_rcpf(d1.x), __builtin_amdgcn_rcpf(d1.y)};
            const int cb = NQ == 4 ? (q ^ (2 * wr)) : q;
            *(LAS f32x4*)(tb + fr * 256 + (((cb * 4 + fq) ^ (fr & 7)) << 4)) = (c[q] * scale) * r;
        }
        constexpr int LPR = NQ * 4, RPI = 64 / LPR;
#pragma unroll
        for (int i = 0; i < NQ; ++i) {
            const int r = i * RPI + lane / LPR;
            tv[i] = *(LAS f32x4*)(tb + r * 256 + (((lane % LPR) ^ (r & 7)) << 4));
        }
        __builtin_amdgcn_sched_barrier(0);
    };
    auto put = [&](auto nqc, const float* rowp) {
        constexpr int NQ = decltype(nqc)::v, LPR = NQ * 4, RPI = 64 / LPR;
        const unsigned ovoff = (unsigned)((lane / LPR) * NS * 4 + (lane % LPR) * 16);
#pragma unroll
        for (int i = 0; i < NQ; ++i)
            asm volatile("global_store_dwordx4 %0, %1, %2 sc1\n\ts_nop 1" :: "v"(ovoff), "v"(tv[i]), "s"(rowp + (size_t)i * RPI * NS) : "memory");
        __builtin_amdgcn_sched_barrier(0);
    };
    expo(IC<4>{}, acc[0]);
#pragma unroll
    for (int m = 0; m < 4; ++m) {
        stage(IC<4>{}, acc[m]);
        if (m < 3) expo(IC<4>{}, acc[m + 1]);
        else if (has_extra) expo(IC<2>{}, accx);
        put(IC<4>{}, ob + (size_t)m * 16 * NS);
    }
    if (has_extra) { stage(IC<2>{}, accx); put(IC<2>{}, ox); }
}

}

extern "C" void kernel_launch(int rank, int nranks, ncclComm_t comm,
                              hipStream_t stream,
                              void* const* d_in, const int* in_sizes, int n_in,
                              void* d_out, int out_size,
                              const int* in_dtypes, int out_dtype,
                              void* const* d_out_all,
                              void* const* d_ws_all, size_t ws_bytes) {
    (void)comm; (void)in_dtypes; (void)out_dtype; (void)d_out_all;
    if (nranks != NR || (unsigned)rank >= (unsigned)NR || n_in != 4 || ws_bytes < WS_END || in_sizes[0] != MS * KD
        || in_sizes[1] != KD * ND || out_size != NR * MS * NS) {
        fprintf(stderr, "kernel_launch: unexpected problem shape\n");
        return;
    }
    HIP_CHECK(hipFuncSetAttribute((const void*)fused_ag_gemm, hipFuncAttributeMaxDynamicSharedMemorySize, LDS_BYTES));
    Params p;
    p.x = (const unsigned char*)d_in[0];
    p.w = (const unsigned char*)d_in[1] + (size_t)rank * NS;
    p.sx = (const float*)d_in[2];
    p.sw = (const float*)d_in[3];
    p.out = (float*)d_out;
    p.me = (unsigned char*)d_ws_all[rank];
    for (int i = 0; i < NR - 1; ++i) p.peer[i] = (unsigned char*)d_ws_all[(rank + 1 + i) % NR] + (size_t)rank * SHARD;
    p.rank = rank;
    p.mode = 0;
    hipLaunchKernelGGL(fused_ag_gemm, dim3(NWG), dim3(NTHREADS), LDS_BYTES, stream, p);
    HIP_CHECK(hipGetLastError());
}
```

```cpp
#include <hip/hip_runtime.h>
#include <rccl/rccl.h>
#include <cstdio>

#define HIP_CHECK(expr) do { hipError_t _e = (expr); if (_e != hipSuccess) { \
    fprintf(stderr, "HIP error %s at %s:%d\n", hipGetErrorString(_e), __FILE__, __LINE__); \
    return; } } while (0)

namespace {

#define LAS __attribute__((address_space(3)))
typedef int v8i __attribute__((ext_vector_type(8)));
typedef int v4i __attribute__((ext_vector_type(4)));
typedef int v2i __attribute__((ext_vector_type(2)));
typedef float f32x4 __attribute__((ext_vector_type(4)));
typedef unsigned u32x4 __attribute__((ext_vector_type(4)));

constexpr int NR = 8;
constexpr int KD = 4096, ND = 16384;
constexpr int MS = 512, NS = 2048;
constexpr int BM = 128, BN = 256, BK = 128;
constexpr int NSTEP = KD / BK;
constexpr int MT = MS / BM;
constexpr int NWG = 256, NTHREADS = 768;
constexpr int NSLOT = 3;
constexpr int A_BYTES = BM * BK, E_BYTES = 16 * BK, B_BYTES = BK * BN;
constexpr int E_OFF = A_BYTES, B_OFF = A_BYTES + E_BYTES, SLOT_BYTES = B_OFF + B_BYTES;
constexpr int SYNC_OFF = NSLOT * SLOT_BYTES;
constexpr int SY_BAD = SYNC_OFF;
constexpr int SY_PROG = SYNC_OFF, SY_META = SYNC_OFF + 64;
constexpr int JUNK_OFF = SYNC_OFF + 256;
constexpr int LDS_BYTES = JUNK_OFF + 2048;

constexpr int NPUSH = 16;
constexpr int NITEM = MT * NSTEP * 4;
constexpr int NSTATIC = 4 * NPUSH;
constexpr int NDRAW = 512;
static_assert(NITEM - NSTATIC + 4 * NPUSH == NDRAW && 32 * 4096 <= SYNC_OFF, "draw count; staging ring");

constexpr size_t UNIT = A_BYTES;
constexpr size_t SHARD = (size_t)MS * KD;
constexpr size_t XBUF = (size_t)NR * SHARD;
constexpr size_t WS_FLAG = 2 * XBUF;
constexpr size_t WS_CNT = WS_FLAG + 65536;
constexpr size_t WS_TMO = WS_CNT + (size_t)NWG * 8 * 128;
constexpr size_t WS_DRAW = WS_TMO + 1024;
constexpr size_t WS_DBG = WS_TMO + 4096;
constexpr size_t WS_TRACE = WS_DBG + (size_t)NWG * 64 * 8;
constexpr size_t WS_END = WS_TRACE + 8192;
constexpr unsigned long long DEADLINE = 50000000ull;

struct Params {
    const unsigned char* x;
    unsigned char* me;
    const unsigned char* w;
    float* out;
    unsigned char* peer[NR - 1];
    int rank;
    int mode;
    const float* sx; const float* sw;
};

template <int N> __device__ __forceinline__ void wait_vm() { asm volatile("s_waitcnt vmcnt(%0)" :: "i"(N) : "memory"); }
template <int I> struct IC { static constexpr int v = I; };
template <int I, int E, class F> __device__ __forceinline__ void static_for(F&& f) { if constexpr (I < E) { f(IC<I>{}); static_for<I + 1, E>(f); } }
__device__ __forceinline__ unsigned lds_ld(unsigned addr) {
    unsigned v; asm volatile("ds_read_b32 %0, %1\n\ts_waitcnt lgkmcnt(0)" : "=v"(v) : "v"(addr) : "memory"); return v;
}
__device__ __forceinline__ void lds_st(unsigned addr, unsigned v) { asm volatile("ds_write_b32 %0, %1" :: "v"(addr), "v"(v) : "memory"); }
#define GLDS_BITS ""
template <bool SYS, int OFF> __device__ __forceinline__ void glds(const unsigned char* base, unsigned voff, unsigned dst) {
    if (SYS) asm volatile("s_add_u32 m0, %2, %3\n\ts_nop 0\n\tglobal_load_lds_dwordx4 %0, %1 sc0 sc1"
                          :: "v"(voff), "s"(base), "s"(dst), "i"(OFF) : "memory", "scc");
    else     asm volatile("s_add_u32 m0, %2, %3\n\ts_nop 0\n\tglobal_load_lds_dwordx4 %0, %1" GLDS_BITS
                          :: "v"(voff), "s"(base), "s"(dst), "i"(OFF) : "memory", "scc");
}
__device__ __forceinline__ unsigned ticket_issue(unsigned char* me, int role) {
    unsigned t = 0;
    unsigned char* a = me + WS_CNT + ((size_t)blockIdx.x * 8 + role) * 128;
    if ((threadIdx.x & 63) == 0) asm volatile("global_atomic_add %0, %1, %2, off sc0" : "+v"(t) : "v"(a), "v"(1u) : "memory");
    return t;
}
#define STAMP(i) do {} while (0)
#define STAMPC(i) do {} while (0)
#define DIAG_MODE(bit) false

#define TRACE_WG 255
#define TRACE_LA 17
#define BAR_AT 1
#define MPRIO1 1
#define MPRIO2 2
#define WARM 0
#define FREE_START 1
#define POLL_EARLY 0
#define PREFETCH 1
#define PF_DELAY 100
#define PF_AHEAD 7
#define STAGE_AHEAD 5
#define POLL_SLEEP 2
#define TILEMAP 2
#define TILE_INDEX(b) ((((b) - NPUSH) & 7) * 30 + (((b) - NPUSH) >> 3))
#define DIAG_GROUP(b) ((b) < NPUSH ? -1 : (TILE_INDEX(b) >> 3) >= 28 ? 4 : ((TILE_INDEX(b) >> 3) & 3))
#define DIAG_LOCAL(b) ((b) < NPUSH || (TILE_INDEX(b) >> 3) >= 28)

#define ORD_R 0
#define ORD_FIN 870, 870, 1000, 1000
struct Order { unsigned q[NITEM]; };
constexpr Order make_order() {
    constexpr int fin[4] = {ORD_FIN};
    Order o{}; int key[NITEM] = {}; int n = 0;
    for (int k = 0; k < NSTEP; ++k)
        for (int t = 0; t < MT; ++t)
            for (int j = 0; j < 4; ++j) {
                int due = fin[t] * 2 * (k + 1);
                if (fin[t] == 1000 && k >= NSTEP - ORD_R) due = 1000 * (2 * (NSTEP - ORD_R) + (j >= 2 ? ORD_R : 0) + (k - (NSTEP - ORD_R)) + 1);
                if (k < 4) due = 0;
                int i = n++;
                while (i > 0 && key[i - 1] > due) { key[i] = key[i - 1]; o.q[i] = o.q[i - 1]; --i; }
                key[i] = due; o.q[i] = (unsigned)((t * NSTEP + k) * 4 + j);
            }
    return o;
}
__constant__ const Order k_order = make_order();
__device__ __forceinline__ int static_item(int id) { return (((id >> 2) & 3) * NSTEP + (id >> 4)) * 4 + (id & 3); }
__device__ __forceinline__ size_t item_src(int q) { return (size_t)((q >> 7) * BM + (q & 3) * 32) * KD + ((q >> 2) & 31) * BK; }

__global__ __launch_bounds__(NTHREADS) void fused_ag_gemm(Params p) {
    extern __shared__ __attribute__((aligned(16))) unsigned char smem[];
    const int tid = threadIdx.x, lane = tid & 63, wid = __builtin_amdgcn_readfirstlane(tid >> 6);
    const unsigned lds0 = (unsigned)(size_t)(LAS unsigned char*)smem;
    unsigned char* me = p.me;
    const int b = blockIdx.x;
    const unsigned aoff0 = (unsigned)(lane >> 3) * KD + (unsigned)(((lane & 7) ^ (lane >> 4)) << 4);
    const unsigned aoff1 = (unsigned)(lane >> 3) * KD + (unsigned)(((lane & 7) ^ (4 + (lane >> 4))) << 4);

    if (DIAG_MODE(32)) return;
    if (wid == 0) { STAMP(0); STAMPC(4); }

    if (__builtin_expect(b < NPUSH, 1)) {
        if (tid < 64) *(LAS unsigned*)(smem + SYNC_OFF + tid * 4) = 0u;
        if (wid >= 8) {
            const int L = wid - 8;
            __builtin_amdgcn_s_barrier();
            if (DIAG_MODE(2)) return;
#pragma unroll 1
            for (int i = L ? L : 4;; i += 4) {
                if (i > STAGE_AHEAD) {
                    for (;;) {
                        const unsigned pr = lds_ld(lds0 + SY_PROG + (lane % 7) * 4);
                        if (__any(pr >= (unsigned)(i - STAGE_AHEAD)) && __all(pr + 32u > (unsigned)i)) break;
                        __builtin_amdgcn_s_sleep(2);
                    }
                }
                int q;
                if (i < 4) q = static_item(i * NPUSH + b);
                else {
                    unsigned d = 0;
                    if (lane == 0) d = __hip_atomic_fetch_add((unsigned*)(me + WS_DRAW), 1u, __ATOMIC_RELAXED, __HIP_MEMORY_SCOPE_AGENT);
                    const unsigned id = (__builtin_amdgcn_readfirstlane(d) & (NDRAW - 1)) + NSTATIC;
                    if (id >= (unsigned)NITEM) { lds_st(lds0 + SY_META + (i & 31) * 4, (unsigned)(i + 1) << 16 | 0xffffu); return; }
                    q = (int)k_order.q[id];
                }
                const unsigned char* sp = p.x + item_src(q);
                static_for<0, 4>([&](auto qc) {
                    constexpr int c = decltype(qc)::v;
                    glds<false, c * 1024>(sp, ((c & 1) ? aoff1 : aoff0) + c * 8 * KD, lds0 + (i & 31) * 4096);
                });
                wait_vm<0>();
                lds_st(lds0 + SY_META + (i & 31) * 4, (unsigned)(i + 1) << 16 | (unsigned)q);
            }
        }
        if (wid == 7 || DIAG_MODE(2)) { __syncthreads(); return; }
        unsigned char* peer = p.peer[(wid + b) % 7];
        const int rank = p.rank;
        int q = static_item(b);
        u32x4 r0, r1, r2, r3;
        asm volatile("global_load_dwordx4 %0, %4, %8\n\tglobal_load_dwordx4 %1, %5, %8\n\t"
                     "global_load_dwordx4 %2, %6, %8\n\tglobal_load_dwordx4 %3, %7, %8"
                     : "=&v"(r0), "=&v"(r1), "=&v"(r2), "=&v"(r3)
                     : "v"(aoff0), "v"(aoff1 + 8 * KD), "v"(aoff0 + 16 * KD), "v"(aoff1 + 24 * KD), "s"(p.x + item_src(q)) : "memory");
        unsigned tk = ticket_issue(me, wid);
        unsigned char* fl = peer - (size_t)rank * SHARD + WS_FLAG + (size_t)rank * NITEM * 4;
        asm volatile("s_waitcnt lgkmcnt(0)\n\ts_barrier\n\ts_waitcnt vmcnt(0)" : "+v"(tk), "+v"(r0), "+v"(r1), "+v"(r2), "+v"(r3) :: "memory");
        const unsigned call = __builtin_amdgcn_readfirstlane(tk) + 1u;
        unsigned char* dst = peer + (size_t)(call & 1u) * XBUF;
        const unsigned voff = (unsigned)lane * 16u;
        unsigned char* fprev = nullptr;
        unsigned ended = 0;
        for (int i = 0; ended != 15u; ++i) {
            if (i > 0) {
                if (ended >> (i & 3) & 1u) continue;
                unsigned m;
                while (((m = __builtin_amdgcn_readfirstlane(lds_ld(lds0 + SY_META + (i & 31) * 4))) >> 16) != (unsigned)(i + 1)) __builtin_amdgcn_s_sleep(1);
                q = (int)(m & 0xffffu);
                if (q == 0xffff) { ended |= 1u << (i & 3); lds_st(lds0 + SY_PROG + wid * 4, (unsigned)i); continue; }
                asm volatile("ds_read_b128 %0, %4\n\tds_read_b128 %1, %4 offset:1024\n\tds_read_b128 %2, %4 offset:2048\n\t"
                             "ds_read_b128 %3, %4 offset:3072\n\ts_waitcnt lgkmcnt(0)"
                             : "=&v"(r0), "=&v"(r1), "=&v"(r2), "=&v"(r3) : "v"(lds0 + (unsigned)(i & 31) * 4096u + voff) : "memory");
                lds_st(lds0 + SY_PROG + wid * 4, (unsigned)i);
            }
            unsigned char* o = dst + (size_t)q * 4096;
            if (i >= FREE_START) wait_vm<3>();
            if (i == 0 && wid == 0) STAMP(6);
            asm volatile("global_store_dwordx4 %0, %1, %3 sc0 sc1\n\tglobal_store_dwordx4 %0, %2, %3 offset:1024 sc0 sc1\n\ts_nop 1"
                         :: "v"(voff), "v"(r0), "v"(r1), "s"(o) : "memory");
            if (i > 0) wait_vm<2>();
            if (i == 1 && wid == 0) STAMP(7);
            if (i > 0 && lane == 0) asm volatile("global_store_dword %0, %1, %2 sc0 sc1" :: "v"(0u), "v"(call), "s"(fprev) : "memory");
            asm volatile("global_store_dwordx4 %0, %1, %3 offset:2048 sc0 sc1\n\tglobal_store_dwordx4 %0, %2, %3 offset:3072 sc0 sc1\n\ts_nop 1"
                         :: "v"(voff), "v"(r2), "v"(r3), "s"(o) : "memory");
            fprev = fl + (size_t)q * 4;
        }
        wait_vm<0>();
        if (lane == 0) asm volatile("global_store_dword %0, %1, %2 sc0 sc1" :: "v"(0u), "v"(call), "s"(fprev) : "memory");
        wait_vm<0>();
        STAMP(8 + wid);
        return;
    }

    const int tile = TILE_INDEX(b), ms = tile >> 3, nt = tile & 7;
    const bool local = ms >= 28;
    const int t = local ? ms - 28 : (ms & 3);
    const int src = local ? p.rank : ((p.rank + 1 + (ms >> 2)) & (NR - 1));
    const bool has_extra = t < 2;
    const int xrow = local ? 256 + 16 * t : 288 + 16 * ((ms >> 2) * 2 + t);

    if (wid >= 9) {
        unsigned boff[4];
#pragma unroll
        for (int q = 0; q < 4; ++q) {
            const int kk = (q & 1) * 4 + (lane >> 4), f = kk | ((q >> 1) << 3);
            boff[q] = (unsigned)(lane >> 4) * ND + (unsigned)(((lane & 15) ^ f) << 4);
        }
        const unsigned char* wb = p.w + nt * BN;
        const unsigned char* xe = p.x + (size_t)xrow * KD;
        auto run = [&](auto q0c, auto q1c, auto exc) {
            constexpr int Q0 = decltype(q0c)::v, Q1 = decltype(q1c)::v, NI = Q1 - Q0 + 2 * decltype(exc)::v;
            unsigned nslot = lds0;
            auto issue = [&](int k) {
                const unsigned slot = nslot;
                nslot = slot == lds0 + (NSLOT - 1) * SLOT_BYTES ? lds0 : slot + SLOT_BYTES;
                if (DIAG_MODE(4)) return;
                const unsigned char* wk = wb + (size_t)k * BK * ND;
                static_for<Q0, Q1>([&](auto qc) {
                    constexpr int q = decltype(qc)::v;
                    glds<false, B_OFF + q * 1024>(wk, boff[(q & 1) | (((q >> 2) & 1) << 1)] + q * 4 * ND, slot);
                });
                if (decltype(exc)::v) {
                    glds<false, E_OFF>(xe + k * BK, aoff0, slot);
                    glds<false, E_OFF + 1024>(xe + k * BK, aoff1 + 8 * KD, slot);
                }
            };
            issue(0); issue(1); issue(2);
            wait_vm<2 * NI>(); __builtin_amdgcn_s_barrier();
            wait_vm<NI>(); __builtin_amdgcn_s_barrier();
#pragma unroll 1
            for (int s = 1; s < NSTEP - 2; ++s) { issue(s + 2); wait_vm<NI>(); __builtin_amdgcn_s_barrier(); }
            wait_vm<0>(); __builtin_amdgcn_s_barrier();
        };
        if (wid == 9) run(IC<0>{}, IC<11>{}, IC<0>{});
        else if (wid == 10) run(IC<11>{}, IC<22>{}, IC<0>{});
        else if (has_extra) run(IC<22>{}, IC<32>{}, IC<1>{});
        else run(IC<22>{}, IC<32>{}, IC<0>{});
        return;
    }
    if (wid == 8) {
        if (lane == 0) lds_st(lds0 + SY_BAD, 0u);
        unsigned call = 0, tk = 0, fv = 0;
        if (!local) tk = ticket_issue(me, 0);
        const unsigned char* xr = me + (size_t)src * SHARD + (size_t)t * NSTEP * UNIT;
        const unsigned char* xl = p.x + (size_t)t * BM * KD;
        const unsigned char* fl = me + WS_FLAG + (size_t)(src * MT + t) * NSTEP * 16 + lane * 4;
        unsigned* tmo = (unsigned*)(me + WS_TMO);
        bool dead = false, pend = false;
        const bool noflags = local || DIAG_MODE(1);
        unsigned long long mask = 0;
        int base = 0, pbase = 0, next = 0;
        auto known = [&](int k) { return noflags || dead || (k - base < 16 && ((mask >> (4 * (k - base))) & 15u) == 15u); };
        auto poll = [&](int k) {
            asm volatile("global_load_dword %0, %1, off sc0 sc1" : "=v"(fv) : "v"(fl + k * 16) : "memory");
            pbase = k;
        };
        auto take = [&]() {
            asm volatile("s_waitcnt vmcnt(0)" : "+v"(fv), "+v"(tk) :: "memory");
            if (!call) { call = __builtin_amdgcn_readfirstlane(tk) + 1u; xr += (size_t)(call & 1u) * XBUF; }
            mask = __ballot(fv >= call); base = pbase;
        };
        unsigned nslot = lds0;
#define LTR(v, i) do {} while (0)
        auto issue = [&](int k) {
            LTR(lt0, k);
            const unsigned slot = nslot;
            nslot = slot == lds0 + (NSLOT - 1) * SLOT_BYTES ? lds0 : slot + SLOT_BYTES;
            if (DIAG_MODE(4) || dead) return;
            if (local)
                static_for<0, 16>([&](auto qc) {
                    constexpr int q = decltype(qc)::v;
                    glds<false, q * 1024>(xl + k * BK, ((q & 1) ? aoff1 : aoff0) + q * 8 * KD, slot);
                });
            else
                static_for<0, 16>([&](auto qc) {
                    constexpr int q = decltype(qc)::v;
                    glds<true, q * 1024>(xr + (size_t)k * UNIT, (unsigned)lane * 16u + q * 1024, slot);
                });
        };
        if (!local && DIAG_MODE(1)) { poll(0); take(); }
#pragma unroll 1
        for (int s = -1; s < NSTEP - 1; ++s) {
            const int lim = s < 0 ? 2 : (s + 2 < NSTEP ? s + 2 : NSTEP - 1);
            if (pend) { take(); pend = false; }
            while (next <= lim && known(next)) issue(next++);
            if (next <= s + 1) {
                const unsigned long long t0 = __builtin_amdgcn_s_memrealtime();
                for (unsigned polls = 1;; ++polls) {
                    poll(next); take();
                    if (known(next)) break;
                    if ((polls & 63u) == 0u
                        && (__builtin_amdgcn_readfirstlane(__hip_atomic_load(tmo, __ATOMIC_RELAXED, __HIP_MEMORY_SCOPE_AGENT)) == call
                            || __builtin_amdgcn_s_memrealtime() - t0 > DEADLINE)) {
                        if (lane == 0) { __hip_atomic_store(tmo, call, __ATOMIC_RELAXED, __HIP_MEMORY_SCOPE_AGENT); lds_st(lds0 + SY_BAD, 1u); }
                        dead = true;
                        break;
                    }
                    __builtin_amdgcn_s_sleep(POLL_SLEEP);
                }
                STAMP(16 + next);
                while (next <= lim && known(next)) issue(next++);
            }
            if (next > s + 2 && !dead) wait_vm<16>(); else wait_vm<0>();
            LTR(lt1, s + 1);
            if (next < NSTEP && !known(next)) { poll(next); pend = true; }
            asm volatile("s_waitcnt lgkmcnt(0)\n\ts_barrier" ::: "memory");
            LTR(lt2, s + 1);
        }
        wait_vm<0>();
        return;
    }

    const int wr = wid >> 2, wc = wid & 3;
    unsigned v16 = (unsigned)lane * 16u;
    int pfs = (local || t >= 2 || !PREFETCH) ? NSTEP : ((ms >> 2) * 16 + t * 8 + nt) % 7;
    const unsigned char* pfp = p.w + (size_t)((pfs + PF_AHEAD) * BK + (((ms >> 2) * 16 + t * 8 + nt) / 7) * 8 + wid) * ND;
    if (PREFETCH) {
        const int n = (b - NPUSH) * 8 + wid;
        if (n < (PF_AHEAD - 3) * 256) {
            const unsigned long long t0 = __builtin_amdgcn_s_memrealtime();
            while (__builtin_amdgcn_s_memrealtime() - t0 < (unsigned long long)PF_DELAY) __builtin_amdgcn_s_sleep(4);
            glds<false, JUNK_OFF>(p.w + (size_t)(3 * BK + (n >> 1)) * ND + (n & 1) * 1024, v16, lds0);
        }
    }
    asm volatile("" : "+v"(v16));
    unsigned sx = __builtin_amdgcn_readfirstlane(__float_as_uint(p.sx[0])), sw = __builtin_amdgcn_readfirstlane(__float_as_uint(p.sw[0]));
    int ex = 2 - t, top = 1 - wr;
    f32x4 acc[4][4], accx[2];
#pragma unroll
    for (int m = 0; m < 4; ++m)
#pragma unroll
        for (int n = 0; n < 4; ++n) acc[m][n] = (f32x4){0.f, 0.f, 0.f, 0.f};
    accx[0] = accx[1] = (f32x4){0.f, 0.f, 0.f, 0.f};
    const float* ob = p.out + (size_t)(src * MS + t * BM + wr * 64) * NS + nt * BN + wc * 64;
    const float* ox = p.out + (size_t)(p.rank * MS + xrow) * NS + nt * BN + wc * 64 + wr * 32;
    asm volatile("" : "+s"(ob), "+s"(ox), "+s"(sx), "+s"(sw), "+s"(ex), "+s"(top));
    unsigned aa[2], ea[2], bo[4];
    {
        const int fr = lane & 15, fq = lane >> 4, q = fr >> 1, f = q | ((fq & 1) << 3);
        aa[0] = lds0 + (unsigned)((wr * 64 + fr) * 128 + ((fq ^ ((fr >> 1) & 7)) << 4));
        aa[1] = aa[0] ^ 64u;
        ea[0] = aa[0] + (unsigned)(E_OFF - wr * 8192);
        ea[1] = ea[0] ^ 64u;
#pragma unroll
        for (int n = 0; n < 4; ++n) bo[n] = lds0 + (unsigned)((16 * fq + q) * 256 + (((wc * 4 + (n ^ (2 * wr))) ^ f) << 4) + (fr & 1) * 8);
    }
    asm volatile("s_barrier" ::: "memory");
    {
        int slot = 0;
#pragma unroll 1
        for (int s = 0; s < NSTEP; ++s) {
            v8i af[4], bf[4], ef;
#pragma unroll
            for (int n = 0; n < 4; ++n) {
                LAS unsigned char* q = (LAS unsigned char*)(size_t)bo[n] + B_OFF;
                const v2i t0 = __builtin_amdgcn_ds_read_tr8_b64_v2i32((LAS v2i*)q);
                const v2i t1 = __builtin_amdgcn_ds_read_tr8_b64_v2i32((LAS v2i*)(q + 8 * 256));
                const v2i t2 = __builtin_amdgcn_ds_read_tr8_b64_v2i32((LAS v2i*)(q + 64 * 256));
                const v2i t3 = __builtin_amdgcn_ds_read_tr8_b64_v2i32((LAS v2i*)(q + 72 * 256));
                bf[n] = (v8i){t0.x, t0.y, t1.x, t1.y, t2.x, t2.y, t3.x, t3.y};
            }
#pragma unroll
            for (int m = 0; m < 4; ++m) {
                const v4i lo = *(LAS v4i*)((LAS unsigned char*)(size_t)aa[0] + m * 2048), hi = *(LAS v4i*)((LAS unsigned char*)(size_t)aa[1] + m * 2048);
                af[m] = (v8i){lo.x, lo.y, lo.z, lo.w, hi.x, hi.y, hi.z, hi.w};
            }
            asm volatile("" : "+s"(ex));
            if (ex > 0) {
                const v4i lo = *(LAS v4i*)(size_t)ea[0], hi = *(LAS v4i*)(size_t)ea[1];
                ef = (v8i){lo.x, lo.y, lo.z, lo.w, hi.x, hi.y, hi.z, hi.w};
            }
            if (s == pfs) {
                asm volatile("s_add_u32 m0, %2, %3\n\ts_nop 0\n\tglobal_load_lds_dwordx4 %0, %1\n\tglobal_load_lds_dwordx4 %0, %1 offset:1024"
                             :: "v"(v16), "s"(pfp), "s"(lds0), "i"(JUNK_OFF) : "memory", "scc");
                pfs = pfs + 7 + PF_AHEAD < NSTEP ? pfs + 7 : NSTEP;
                pfp += (size_t)7 * BK * ND;
            }
            asm volatile("" ::: "memory");
            __builtin_amdgcn_s_waitcnt(0xC07F);
            slot = (slot == NSLOT - 1) ? 0 : slot + 1;
            const int delta = slot ? SLOT_BYTES : -(NSLOT - 1) * SLOT_BYTES;
            int bar = s < NSTEP - 1 ? 1 + top : 0;
            asm volatile("" : "+s"(bar));
            if (bar == 1) asm volatile("s_barrier" ::: "memory");
            __builtin_amdgcn_sched_barrier(0);
            __builtin_amdgcn_s_setprio(MPRIO1);
#pragma unroll
            for (int m = 0; m < 4; ++m) {
#pragma unroll
                for (int n = 0; n < 4; ++n) {
                    acc[m][n] = __builtin_amdgcn_mfma_scale_f32_16x16x128_f8f6f4(bf[n], af[m], acc[m][n], 0, 0, 0, 0, 0, 0);
                    if (m == 0 && n == 0) { __builtin_amdgcn_sched_barrier(0); __builtin_amdgcn_s_setprio(MPRIO2); __builtin_amdgcn_sched_barrier(0); }
                    if (m == 3 && n == BAR_AT) {
                        __builtin_amdgcn_sched_barrier(0);
                        asm volatile("" : "+s"(bar));
                        if (bar == 2) asm volatile("s_barrier" ::: "memory");
                        __builtin_amdgcn_sched_barrier(0);
                    }
                }
                __builtin_amdgcn_sched_barrier(0);
                if (m == 0) asm volatile("v_add_u32 %0, %4, %0\n\tv_add_u32 %1, %4, %1\n\tv_add_u32 %2, %4, %2\n\tv_add_u32 %3, %4, %3"
                                         : "+v"(bo[0]), "+v"(bo[1]), "+v"(bo[2]), "+v"(bo[3]) : "s"(delta));
                if (m == 1) asm volatile("v_add_u32 %0, %4, %0\n\tv_add_u32 %1, %4, %1\n\tv_add_u32 %2, %4, %2\n\tv_add_u32 %3, %4, %3"
                                         : "+v"(aa[0]), "+v"(aa[1]), "+v"(ea[0]), "+v"(ea[1]) : "s"(delta));
                __builtin_amdgcn_sched_barrier(0);
            }
            asm volatile("" : "+s"(ex));
            if (ex > 0) {
                accx[0] = __builtin_amdgcn_mfma_scale_f32_16x16x128_f8f6f4(bf[0], ef, accx[0], 0, 0, 0, 0, 0, 0);
                accx[1] = __builtin_amdgcn_mfma_scale_f32_16x16x128_f8f6f4(bf[1], ef, accx[1], 0, 0, 0, 0, 0, 0);
            }
            __builtin_amdgcn_s_setprio(0);
            __builtin_amdgcn_sched_barrier(0);
        }
    }
    if (wid == 0) { STAMP(1); STAMPC(5); }
    STAMP(48 + wid);
    float scale = __uint_as_float(sx) * __uint_as_float(sw);
    if (__builtin_amdgcn_readfirstlane(lds_ld(lds0 + SY_BAD)) != 0u) scale = __builtin_nanf("");
    const int fr = lane & 15, fq = lane >> 4;
    const float zs = scale * -1.44269504088896340736f, zmax = 60.f * 1.44269504088896340736f;
    LAS unsigned char* tb = (LAS unsigned char*)smem + ((NSTEP - 3 + (wc >> 1)) % NSLOT) * SLOT_BYTES + wr * 8192 + (wc & 1) * 4096;
    typedef float f32x2 __attribute__((ext_vector_type(2)));
    f32x2 e[8];
    f32x4 tv[4];
    auto expo = [&](auto nqc, const f32x4* c) {
        constexpr int NQ = decltype(nqc)::v;
#pragma unroll
        for (int h = 0; h < 2 * NQ; ++h) {
            f32x2 z = (f32x2){c[h >> 1][(h & 1) * 2], c[h >> 1][(h & 1) * 2 + 1]} * zs;
            asm("" : "+v"(z));
            e[h].x = __builtin_amdgcn_exp2f(__builtin_amdgcn_fmed3f(z.x, -zmax, zmax));
            e[h].y = __builtin_amdgcn_exp2f(__builtin_amdgcn_fmed3f(z.y, -zmax, zmax));
        }
        __builtin_amdgcn_sched_barrier(0);
    };
    auto stage = [&](auto nqc, const f32x4* c) {
        constexpr int NQ = decltype(nqc)::v;
#pragma unroll
        for (int q = 0; q < NQ; ++q) {
            f32x2 d0 = e[2 * q] + 1.0f, d1 = e[2 * q + 1] + 1.0f;
            asm("" : "+v"(d0)); asm("" : "+v"(d1));
            const f32x4 r = {__builtin_amdgcn_rcpf(d0.x), __builtin_amdgcn_rcpf(d0.y), __builtin_amdgcn_rcpf(d1.x), __builtin_amdgcn_rcpf(d1.y)};
            const int cb = NQ == 4 ? (q ^ (2 * wr)) : q;
            *(LAS f32x4*)(tb + fr * 256 + (((cb * 4 + fq) ^ (fr & 7)) << 4)) = (c[q] * scale) * r;
        }
        constexpr int LPR = NQ * 4, RPI = 64 / LPR;
#pragma unroll
        for (int i = 0; i < NQ; ++i) {
            const int r = i * RPI + lane / LPR;
            tv[i] = *(LAS f32x4*)(tb + r * 256 + (((lane % LPR) ^ (r & 7)) << 4));
        }
        __builtin_amdgcn_sched_barrier(0);
    };
    auto put = [&](auto nqc, const float* rowp) {
        constexpr int NQ = decltype(nqc)::v, LPR = NQ * 4, RPI = 64 / LPR;
        const unsigned ovoff = (unsigned)((lane / LPR) * NS * 4 + (lane % LPR) * 16);
#pragma unroll
        for (int i = 0; i < NQ; ++i)
            asm volatile("global_store_dwordx4 %0, %1, %2 sc1\n\ts_nop 1" :: "v"(ovoff), "v"(tv[i]), "s"(rowp + (size_t)i * RPI * NS) : "memory");
        __builtin_amdgcn_sched_barrier(0);
    };
    expo(IC<4>{}, acc[0]);
#pragma unroll
    for (int m = 0; m < 4; ++m) {
        stage(IC<4>{}, acc[m]);
        if (m < 3) expo(IC<4>{}, acc[m + 1]);
        else if (has_extra) expo(IC<2>{}, accx);
        put(IC<4>{}, ob + (size_t)m * 16 * NS);
    }
    if (has_extra) { stage(IC<2>{}, accx); put(IC<2>{}, ox); }
}

}

extern "C" void kernel_launch(int rank, int nranks, ncclComm_t comm,
                              hipStream_t stream,
                              void* const* d_in, const int* in_sizes, int n_in,
                              void* d_out, int out_size,
                              const int* in_dtypes, int out_dtype,
                              void* const* d_out_all,
                              void* const* d_ws_all, size_t ws_bytes) {
    (void)comm; (void)in_dtypes; (void)out_dtype; (void)d_out_all;
    if (nranks != NR || (unsigned)rank >= (unsigned)NR || n_in != 4 || ws_bytes < WS_END || in_sizes[0] != MS * KD
        || in_sizes[1] != KD * ND || out_size != NR * MS * NS) {
        fprintf(stderr, "kernel_launch: unexpected problem shape\n");
        return;
    }
    HIP_CHECK(hipFuncSetAttribute((const void*)fused_ag_gemm, hipFuncAttributeMaxDynamicSharedMemorySize, LDS_BYTES));
    Params p;
    p.x = (const unsigned char*)d_in[0];
    p.w = (const unsigned char*)d_in[1] + (size_t)rank * NS;
    p.sx = (const float*)d_in[2];
    p.sw = (const float*)d_in[3];
    p.out = (float*)d_out;
    p.me = (unsigned char*)d_ws_all[rank];
    for (int i = 0; i < NR - 1; ++i) p.peer[i] = (unsigned char*)d_ws_all[(rank + 1 + i) % NR] + (size_t)rank * SHARD;
    p.rank = rank;
    p.mode = 0;
    hipLaunchKernelGGL(fused_ag_gemm, dim3(NWG), dim3(NTHREADS), LDS_BYTES, stream, p);
    HIP_CHECK(hipGetLastError());
}
```

```cpp
#include <hip/hip_runtime.h>
#include <rccl/rccl.h>
#include <cstdio>

#define HIP_CHECK(expr) do { hipError_t _e = (expr); if (_e != hipSuccess) { \
    fprintf(stderr, "HIP error %s at %s:%d\n", hipGetErrorString(_e), __FILE__, __LINE__); \
    return; } } while (0)

namespace {

#define LAS __attribute__((address_space(3)))
typedef int v8i __attribute__((ext_vector_type(8)));
typedef int v4i __attribute__((ext_vector_type(4)));
typedef int v2i __attribute__((ext_vector_type(2)));
typedef float f32x2 __attribute__((ext_vector_type(2)));
typedef float f32x4 __attribute__((ext_vector_type(4)));
typedef unsigned u32x2 __attribute__((ext_vector_type(2)));
typedef unsigned u32x4 __attribute__((ext_vector_type(4)));

constexpr int NR = 8;
constexpr int KD = 4096, ND = 16384;
constexpr int MS = 512, NS = 2048;
constexpr int BM = 128, BN = 256, BK = 128;
constexpr int NSTEP = KD / BK;
constexpr int MT = MS / BM;
constexpr int NWG = 256, NTHREADS = 768;
constexpr int NSLOT = 3;
constexpr int A_BYTES = BM * BK, E_BYTES = 16 * BK, B_BYTES = BK * BN;
constexpr int E_OFF = A_BYTES, B_OFF = A_BYTES + E_BYTES, SLOT_BYTES = B_OFF + B_BYTES;
constexpr int SYNC_OFF = NSLOT * SLOT_BYTES;
constexpr int SY_BAD = SYNC_OFF;
constexpr int SY_PROG = SYNC_OFF, SY_META = SYNC_OFF + 64;
constexpr int JUNK_OFF = SYNC_OFF + 512;
constexpr int LDS_BYTES = JUNK_OFF + 2048;
static_assert(LDS_BYTES <= 163840, "LDS");

constexpr int NPUSH = 16;
constexpr int ITEM = 4096;
constexpr int NITEM = MT * NSTEP * 4;
constexpr int NSTATIC = 4 * NPUSH;
constexpr int NDRAW = 512;
static_assert(NITEM - NSTATIC + 4 * NPUSH == NDRAW && 32 * ITEM <= SYNC_OFF, "draw count; staging ring");
constexpr int EXC_OFF = 3584, EXC_MAX = 256;
constexpr unsigned META_RAW = 0x80000000u;

constexpr size_t SHARD = (size_t)MS * KD;
constexpr size_t XBUF = (size_t)NR * SHARD;
constexpr size_t FLAG_SRC = (size_t)NITEM * 8, FLAG_PAR = NR * FLAG_SRC;
constexpr size_t WS_FLAG = 2 * XBUF;
constexpr size_t WS_CNT = WS_FLAG + 2 * FLAG_PAR;
constexpr size_t WS_TMO = WS_CNT + (size_t)NWG * 8 * 128;
constexpr size_t WS_DRAW = WS_TMO + 1024;
constexpr size_t WS_DBG = WS_TMO + 4096;
constexpr size_t WS_TRACE = WS_DBG + (size_t)NWG * 64 * 8;
constexpr size_t WS_END = WS_TRACE + 8192;
constexpr unsigned long long DEADLINE = 50000000ull;

#define LOOP_PAD 0
#define COMPRESS 1
#define BASE0 4
#define HPRIO 3
#define PREFETCH 1
#define PF_DELAY 100
#define PF_AHEAD 7
#define STAGE_AHEAD 5
#define POLL_SLEEP 2
#define ORD_FIN 880, 910, 1000, 1000

struct Params {
    const unsigned char* x;
    unsigned char* me;
    const unsigned char* w;
    float* out;
    unsigned char* peer[NR - 1];
    int rank;
    int mode;
    const float* sx; const float* sw;
};

template <int N> __device__ __forceinline__ void wait_vm() { asm volatile("s_waitcnt vmcnt(%0)" :: "i"(N) : "memory"); }
template <int I> struct IC { static constexpr int v = I; };
template <int I, int E, class F> __device__ __forceinline__ void static_for(F&& f) { if constexpr (I < E) { f(IC<I>{}); static_for<I + 1, E>(f); } }
__device__ __forceinline__ unsigned lds_ld(unsigned addr) {
    unsigned v; asm volatile("ds_read_b32 %0, %1\n\ts_waitcnt lgkmcnt(0)" : "=v"(v) : "v"(addr) : "memory"); return v;
}
__device__ __forceinline__ void lds_st(unsigned addr, unsigned v) { asm volatile("ds_write_b32 %0, %1" :: "v"(addr), "v"(v) : "memory"); }
template <int OFF> __device__ __forceinline__ void glds(const unsigned char* base, unsigned voff, unsigned dst) {
    asm volatile("s_add_u32 m0, %2, %3\n\ts_nop 0\n\tglobal_load_lds_dwordx4 %0, %1" :: "v"(voff), "s"(base), "s"(dst), "i"(OFF) : "memory", "scc");
}
__device__ __forceinline__ unsigned ticket_issue(unsigned char* me, int role) {
    unsigned t = 0;
    unsigned char* a = me + WS_CNT + ((size_t)blockIdx.x * 8 + role) * 128;
    if ((threadIdx.x & 63) == 0) asm volatile("global_atomic_add %0, %1, %2, off sc0" : "+v"(t) : "v"(a), "v"(1u) : "memory");
    return t;
}
__device__ __forceinline__ unsigned bfi(unsigned mask, unsigned a, unsigned b) { return (a & mask) | (b & ~mask); }

__device__ __forceinline__ unsigned rebase(unsigned d, unsigned b4) { return bfi(0x7f7f7f7fu, (d | 0x80808080u) - b4, d); }
__device__ __forceinline__ void pack8(const unsigned* d, unsigned* q) {
#pragma unroll
    for (int i = 0; i < 7; ++i) q[i] = bfi(0x40404040u, i < 6 ? d[7] << (1 + i) : d[7] >> 1, d[i]);
}
#define XREGS "v120", "v121", "v122", "v123", "v124", "v125", "v126", "v127", "v128", "v129", "v130", "v131", "v132", "v133", "v134", "v135", "v136", \
              "v140", "v141", "v142", "v143", "v144", "v145", "v146", "v147", "v148", "v149", "v150", "v151", "v152", "v153", "v154", "v155", "v156"
template <int R> __device__ __forceinline__ void item_loads(const unsigned char* a, unsigned v16, unsigned tail, unsigned ve) {
    asm volatile("global_load_dwordx4 v[%c4:%c5], %0, %3 sc0 sc1\n\tglobal_load_dwordx4 v[%c6:%c7], %0, %3 offset:1024 sc0 sc1\n\t"
                 "global_load_dwordx4 v[%c8:%c9], %0, %3 offset:2048 sc0 sc1\n\tglobal_load_dwordx4 v[%c10:%c11], %1, %3 offset:3072 sc0 sc1\n\t"
                 "global_load_ushort v%c12, %2, %3 offset:3584 sc0 sc1"
                 :: "v"(v16), "v"(tail), "v"(ve), "s"(a), "i"(R), "i"(R + 3), "i"(R + 4), "i"(R + 7), "i"(R + 8), "i"(R + 11),
                    "i"(R + 12), "i"(R + 15), "i"(R + 16) : "memory", XREGS);
}
template <int R> __device__ __forceinline__ void raw_stores(unsigned oa) {
    asm volatile("ds_write_b128 %0, v[%c1:%c2]\n\tds_write_b128 %0, v[%c3:%c4] offset:1024\n\t"
                 "ds_write_b128 %0, v[%c5:%c6] offset:2048\n\tds_write_b128 %0, v[%c7:%c8] offset:3072"
                 :: "v"(oa), "i"(R), "i"(R + 3), "i"(R + 4), "i"(R + 7), "i"(R + 8), "i"(R + 11), "i"(R + 12), "i"(R + 15) : "memory");
}
template <int R> __device__ __forceinline__ unsigned reg_get() { unsigned v; asm volatile("v_mov_b32 %0, v%c1" : "=v"(v) : "i"(R)); return v; }
template <int R> __device__ __forceinline__ void split(unsigned& c, unsigned& d, unsigned b4) {
    asm volatile("v_and_b32 %0, 0x40404040, v%c2\n\tv_xad_u32 %1, v%c2, %0, %3" : "=&v"(c), "=v"(d) : "i"(R), "s"(b4));
}
template <int R> __device__ __forceinline__ void unpack8(unsigned* d, unsigned b4) {
    unsigned c[7];
    split<R>(c[0], d[0], b4); split<R + 1>(c[1], d[1], b4); split<R + 2>(c[2], d[2], b4); split<R + 3>(c[3], d[3], b4);
    split<R + 4>(c[4], d[4], b4); split<R + 5>(c[5], d[5], b4); split<R + 6>(c[6], d[6], b4);
    const unsigned t0 = (c[0] << 1) | c[1], t1 = (c[2] << 1) | c[3], t2 = (c[4] << 1) | c[5];
    d[7] = ((t0 >> 2) + (t1 >> 4) + (t2 >> 6)) + ((c[6] << 1) + b4);
}
__device__ __forceinline__ unsigned tops8(const unsigned* d) {
    unsigned m = 0;
#pragma unroll
    for (int i = 0; i < 8; ++i) m |= (i < 7 ? d[i] >> (6 - i) : d[i] << 1) & (0x01010101u << i);
    return m;
}

#define HT(j) do {} while (0)
#define HSAVE() do {} while (0)
#define STAMP(i) do {} while (0)
#define STAMPC(i) do {} while (0)
#define DIAG_MODE(bit) false
#define LTR(v, i) do {} while (0)
#define TRACE_WG 255
#define TRACE_LA 17

#define TILE_INDEX(b) ((((b) - NPUSH) & 7) * 30 + (((b) - NPUSH) >> 3))
#define DIAG_GROUP(b) ((b) < NPUSH ? -1 : (TILE_INDEX(b) >> 3) >= 28 ? 4 : ((TILE_INDEX(b) >> 3) & 3))
#define DIAG_LOCAL(b) ((b) < NPUSH || (TILE_INDEX(b) >> 3) >= 28)

struct Order { unsigned q[NITEM]; };
constexpr Order make_order() {
    constexpr int fin[4] = {ORD_FIN};
    Order o{}; int key[NITEM] = {}; int n = 0;
    for (int k = 0; k < NSTEP; ++k)
        for (int t = 0; t < MT; ++t)
            for (int j = 0; j < 4; ++j) {
                const int due = k < 4 ? 0 : fin[t] * (k + 1);
                int i = n++;
                while (i > 0 && key[i - 1] > due) { key[i] = key[i - 1]; o.q[i] = o.q[i - 1]; --i; }
                key[i] = due; o.q[i] = (unsigned)((t * NSTEP + k) * 4 + j);
            }
    return o;
}
__constant__ const Order k_order = make_order();
__device__ __forceinline__ int static_item(int id) { return (((id >> 2) & 3) * NSTEP + (id >> 4)) * 4 + (id & 3); }
__device__ __forceinline__ size_t item_src(int q) { return (size_t)((q >> 7) * BM + (q & 3) * 32) * KD + ((q >> 2) & 31) * BK; }
__device__ __forceinline__ size_t item_flag(int q) { return (size_t)((((q >> 7) * 4 + (q & 3)) * NSTEP) + ((q >> 2) & 31)) * 8; }

__global__ __launch_bounds__(NTHREADS) void fused_ag_gemm(Params p) {
    extern __shared__ __attribute__((aligned(16))) unsigned char smem[];
    const int tid = threadIdx.x, lane = tid & 63, wid = __builtin_amdgcn_readfirstlane(tid >> 6);
    const unsigned lds0 = (unsigned)(size_t)(LAS unsigned char*)smem;
    unsigned char* me = p.me;
    const int b = blockIdx.x;
    const unsigned aoff0 = (unsigned)(lane >> 3) * KD + (unsigned)(((lane & 7) ^ (lane >> 4)) << 4);
    const unsigned aoff1 = (unsigned)(lane >> 3) * KD + (unsigned)(((lane & 7) ^ (4 + (lane >> 4))) << 4);
    unsigned v16 = (unsigned)lane * 16u;

    if (DIAG_MODE(32)) return;
    if (wid == 0) { STAMP(0); STAMPC(4); }

    if (__builtin_expect(b < NPUSH, 1)) {
        if (tid < 128) *(LAS unsigned*)(smem + SYNC_OFF + tid * 4) = 0u;
        auto load_item = [&](int q, u32x4& r0, u32x4& r1, u32x4& r2, u32x4& r3) {
            asm volatile("global_load_dwordx4 %0, %4, %8\n\tglobal_load_dwordx4 %1, %5, %8\n\t"
                         "global_load_dwordx4 %2, %6, %8\n\tglobal_load_dwordx4 %3, %7, %8"
                         : "=&v"(r0), "=&v"(r1), "=&v"(r2), "=&v"(r3)
                         : "v"(aoff0), "v"(aoff1 + 8 * KD), "v"(aoff0 + 16 * KD), "v"(aoff1 + 24 * KD), "s"(p.x + item_src(q)) : "memory");
        };
        if (wid >= 8) {
            const int L = wid - 8;
            unsigned base = BASE0;
            __syncthreads();
            if (DIAG_MODE(2)) return;
#pragma unroll 1
            for (int i = L ? L : 4;; i += 4) {
                if (i > STAGE_AHEAD) {
                    for (;;) {
                        const unsigned pr = lds_ld(lds0 + SY_PROG + (lane % 7) * 4);
                        if (__any(pr >= (unsigned)(i - STAGE_AHEAD)) && __all(pr + 32u > (unsigned)i)) break;
                        __builtin_amdgcn_s_sleep(2);
                    }
                }
                volatile LAS u32x2* mp = (volatile LAS u32x2*)(smem + SY_META + (i & 31) * 8);
                int q;
                if (i < 4) q = static_item(i * NPUSH + b);
                else {
                    unsigned d = 0;
                    if (lane == 0) d = __hip_atomic_fetch_add((unsigned*)(me + WS_DRAW), 1u, __ATOMIC_RELAXED, __HIP_MEMORY_SCOPE_AGENT);
                    const unsigned id = (__builtin_amdgcn_readfirstlane(d) & (NDRAW - 1)) + NSTATIC;
                    if (id >= (unsigned)NITEM) { *mp = (u32x2){(unsigned)(i + 1) << 16 | 0xffffu, 0u}; return; }
                    q = (int)k_order.q[id];
                }
                u32x4 r0, r1, r2, r3;
                load_item(q, r0, r1, r2, r3);
                asm volatile("s_waitcnt vmcnt(0)" : "+v"(r0), "+v"(r1), "+v"(r2), "+v"(r3) :: "memory");
                const unsigned d[16] = {r0.x, r0.y, r0.z, r0.w, r1.x, r1.y, r1.z, r1.w, r2.x, r2.y, r2.z, r2.w, r3.x, r3.y, r3.z, r3.w};
                LAS unsigned char* slot = (LAS unsigned char*)smem + (i & 31) * ITEM;
                unsigned meta = META_RAW, g[16];
                if (COMPRESS) {
                    for (int attempt = 0; attempt < 2; ++attempt) {
#pragma unroll
                        for (int e = 0; e < 16; ++e) g[e] = rebase(d[e], base * 0x01010101u);
                        unsigned long long m = tops8(g) | (unsigned long long)tops8(g + 8) << 32;
                        const unsigned limit = attempt ? EXC_MAX : 96;
                        unsigned cnt = 0;
                        for (;;) {
                            const unsigned long long act = __ballot(m != 0);
                            if (!act) { meta = base | cnt << 8; break; }
                            const unsigned n = (unsigned)__popcll(act);
                            if (cnt + n > limit) break;
                            if (m) {
                                const unsigned bit = (unsigned)__builtin_ctzll(m), dw = (bit >> 5) * 8 + (bit & 7);
                                const unsigned at = cnt + __builtin_amdgcn_mbcnt_hi((unsigned)(act >> 32), __builtin_amdgcn_mbcnt_lo((unsigned)act, 0u));
                                *(LAS unsigned short*)(slot + EXC_OFF + at * 2) = (unsigned short)((dw >> 2) * 1024 + v16 + (dw & 3) * 4 + ((bit >> 3) & 3));
                                m &= m - 1;
                            }
                            cnt += n;
                        }
                        if (meta != META_RAW || attempt) break;
                        unsigned mx = 0;
#pragma unroll
                        for (int e = 0; e < 16; ++e) {
                            unsigned v = d[e];
                            asm volatile("" : "+v"(v));
#pragma unroll
                            for (int s = 0; s < 32; s += 8) mx = max(mx, (v >> s) & 0x7fu);
                        }
#pragma unroll
                        for (int o = 32; o; o >>= 1) mx = max(mx, (unsigned)__shfl_xor((int)mx, o));
                        mx = __builtin_amdgcn_readfirstlane(mx);
                        base = mx > 67u ? mx - 67u : 0u;
                    }
                }
                if (meta == META_RAW) {
                    *(LAS u32x4*)(slot + v16) = r0; *(LAS u32x4*)(slot + 1024 + v16) = r1;
                    *(LAS u32x4*)(slot + 2048 + v16) = r2; *(LAS u32x4*)(slot + 3072 + v16) = r3;
                } else {
                    unsigned k[14];
                    pack8(g, k); pack8(g + 8, k + 7);
                    *(LAS u32x4*)(slot + v16) = (u32x4){k[0], k[1], k[2], k[3]};
                    *(LAS u32x4*)(slot + 1024 + v16) = (u32x4){k[4], k[5], k[6], k[7]};
                    *(LAS u32x4*)(slot + 2048 + v16) = (u32x4){k[8], k[9], k[10], k[11]};
                    *(LAS u32x2*)(slot + 3072 + lane * 8) = (u32x2){k[12], k[13]};
                }
                asm volatile("s_waitcnt lgkmcnt(0)" ::: "memory");
                *mp = (u32x2){(unsigned)(i + 1) << 16 | (unsigned)q, meta};
            }
        }
        if (wid == 7 || DIAG_MODE(2)) { __syncthreads(); return; }
        unsigned char* peer = p.peer[(wid + b) % 7];
        const int rank = p.rank;
        int q = static_item(b);
        u32x4 r0, r1, r2, r3;
        load_item(q, r0, r1, r2, r3);
        unsigned tk = ticket_issue(me, wid);
        asm volatile("" :: "s"(peer), "s"(rank));
        asm volatile("s_waitcnt lgkmcnt(0)\n\ts_barrier\n\ts_waitcnt vmcnt(0)" : "+v"(tk), "+v"(r0), "+v"(r1), "+v"(r2), "+v"(r3) :: "memory");
        const unsigned call = __builtin_amdgcn_readfirstlane(tk) + 1u;
        unsigned char* dst = peer + (call & 1u) * XBUF + (size_t)rank * SHARD;
        unsigned char* fl = peer + WS_FLAG + (call & 1u) * FLAG_PAR + (size_t)rank * FLAG_SRC;
        unsigned char* fprev = nullptr;
        u32x2 fval = {META_RAW, call};
        unsigned ended = 0, meta = META_RAW;
        for (int i = 0; ended != 15u; ++i) {
            if (i > 0) {
                if (ended >> (i & 3) & 1u) continue;
                u32x2 m;
                for (;;) {
                    asm volatile("ds_read_b64 %0, %1\n\ts_waitcnt lgkmcnt(0)" : "=v"(m) : "v"(lds0 + SY_META + (i & 31) * 8) : "memory");
                    if (__builtin_amdgcn_readfirstlane(m.x) >> 16 == (unsigned)(i + 1)) break;
                    __builtin_amdgcn_s_sleep(1);
                }
                q = (int)(__builtin_amdgcn_readfirstlane(m.x) & 0xffffu);
                meta = __builtin_amdgcn_readfirstlane(m.y);
                if (q == 0xffff) { ended |= 1u << (i & 3); lds_st(lds0 + SY_PROG + wid * 4, (unsigned)i); continue; }
                asm volatile("ds_read_b128 %0, %4\n\tds_read_b128 %1, %4 offset:1024\n\tds_read_b128 %2, %4 offset:2048\n\t"
                             "ds_read_b128 %3, %4 offset:3072\n\ts_waitcnt lgkmcnt(0)"
                             : "=&v"(r0), "=&v"(r1), "=&v"(r2), "=&v"(r3) : "v"(lds0 + (unsigned)(i & 31) * ITEM + v16) : "memory");
                lds_st(lds0 + SY_PROG + wid * 4, (unsigned)i);
            }
            unsigned char* o = dst + (size_t)q * ITEM;
            const unsigned nl = meta == META_RAW ? 64u : 32u + (((meta >> 8) + 31u) >> 5) * 4u;
            wait_vm<3>();
            if (i == 0 && wid == 0) STAMP(6);
            asm volatile("global_store_dwordx4 %0, %1, %3 sc0 sc1\n\tglobal_store_dwordx4 %0, %2, %3 offset:1024 sc0 sc1\n\ts_nop 1"
                         :: "v"(v16), "v"(r0), "v"(r1), "s"(o) : "memory");
            if (i > 0) {
                wait_vm<2>();
                if (i == 1 && wid == 0) STAMP(7);
                if (lane == 0) asm volatile("global_store_dwordx2 %0, %1, %2 sc0 sc1" :: "v"(0u), "v"(fval), "s"(fprev) : "memory");
            }
            asm volatile("global_store_dwordx4 %0, %1, %2 offset:2048 sc0 sc1\n\ts_nop 1" :: "v"(v16), "v"(r2), "s"(o) : "memory");
            if ((unsigned)lane < nl) asm volatile("global_store_dwordx4 %0, %1, %2 offset:3072 sc0 sc1\n\ts_nop 1" :: "v"(v16), "v"(r3), "s"(o) : "memory");
            fprev = fl + item_flag(q);
            fval.x = meta;
        }
        wait_vm<0>();
        if (lane == 0) asm volatile("global_store_dwordx2 %0, %1, %2 sc0 sc1" :: "v"(0u), "v"(fval), "s"(fprev) : "memory");
        wait_vm<0>();
        STAMP(8 + wid);
        return;
    }

    const int tile = TILE_INDEX(b), ms = tile >> 3, nt = tile & 7;
    const bool local = ms >= 28;
    const int t = local ? ms - 28 : (ms & 3);
    const int src = local ? p.rank : ((p.rank + 1 + (ms >> 2)) & (NR - 1));
    const bool has_extra = t < 2;
    const int xrow = local ? 256 + 16 * t : 288 + 16 * ((ms >> 2) * 2 + t);

    if (wid >= 8) {
        const int h = wid - 8;
        const bool ex = has_extra && h < 2;
        __builtin_amdgcn_s_setprio(HPRIO);
        if (h == 0 && lane == 0) lds_st(lds0 + SY_BAD, 0u);
        const unsigned eoff = h ? aoff1 + 8 * KD : aoff0;
        const unsigned char* xe = p.x + (size_t)xrow * KD;
        unsigned eslot = lds0 + h * 1024, aslot = lds0 + h * ITEM;
        auto advance = [&](unsigned& s, unsigned first) { s = s == first + (NSLOT - 1) * SLOT_BYTES ? first : s + SLOT_BYTES; };
        auto extra = [&](int k) {
            if (ex && k < NSTEP && !DIAG_MODE(4)) glds<E_OFF>(xe + k * BK, eoff, eslot);
            advance(eslot, lds0 + h * 1024);
        };
        auto barrier = [&]() { asm volatile("s_waitcnt lgkmcnt(0)\n\ts_barrier" ::: "memory"); };
        if (local || DIAG_MODE(4)) {
            const unsigned char* xl = p.x + (size_t)(t * BM + h * 32) * KD;
            auto issue = [&](int k) {
                extra(k);
                if (!DIAG_MODE(4))
                    static_for<0, 4>([&](auto cc) { constexpr int c = decltype(cc)::v; glds<c * 1024>(xl + k * BK, ((c & 1) ? aoff1 : aoff0) + c * 8 * KD, aslot); });
                advance(aslot, lds0 + h * ITEM);
            };
            issue(0); issue(1);
#pragma unroll 1
            for (int s = -1; s < NSTEP - 2; ++s) {
                if (s >= 0) issue(s + 2);
                if (ex) wait_vm<5>(); else wait_vm<4>();
                barrier();
            }
            wait_vm<0>();
            barrier();
            return;
        }
        unsigned tk = ticket_issue(me, h);
        extra(0);
        const int pi = (ms >> 2) * 16 + t * 8 + nt;
        int pfs = (t >= 2 || !PREFETCH) ? NSTEP : pi % 7;
        const unsigned char* pfp = p.w + (size_t)((pfs + PF_AHEAD) * BK + (pi / 7) * 8 + h * 2) * ND;
        auto prefetch_row = [&](const unsigned char* row) {
            asm volatile("s_add_u32 m0, %2, %3\n\ts_nop 0\n\tglobal_load_lds_dwordx4 %0, %1\n\tglobal_load_lds_dwordx4 %0, %1 offset:1024"
                         :: "v"(v16), "s"(row), "s"(lds0), "i"(JUNK_OFF) : "memory", "scc");
        };
        if (PREFETCH) {
            const int n = ms * 32 + nt * 4 + h;
            if (n < (PF_AHEAD - 2) * BK) {
                const unsigned long long t0 = __builtin_amdgcn_s_memrealtime();
                while (__builtin_amdgcn_s_memrealtime() - t0 < (unsigned long long)PF_DELAY) __builtin_amdgcn_s_sleep(4);
                prefetch_row(p.w + (size_t)(2 * BK + n) * ND);
            }
        }
        asm volatile("s_waitcnt vmcnt(0)" : "+v"(tk) :: "memory");
        const unsigned call = __builtin_amdgcn_readfirstlane(tk) + 1u;
        const unsigned char* xr = me + (call & 1u) * XBUF + (size_t)src * SHARD + (size_t)(t * NSTEP * 4 + h) * ITEM;
        const unsigned char* fl = me + WS_FLAG + (call & 1u) * FLAG_PAR + (size_t)src * FLAG_SRC + (size_t)(t * 4 + h) * NSTEP * 8;
        unsigned* tmo = (unsigned*)(me + WS_TMO);
        const unsigned v8 = (unsigned)(lane & 31) * 8u;
        u32x2 fv = {0u, 0u};
        unsigned meta = 0, seen = 0;
        bool dead = false, pend = false, polled = false, pflast = false;
        int next = 0;
        auto look = [&]() {
            seen = (unsigned)__ballot(fv.y >= call);
            if (DIAG_MODE(1)) { seen = ~0u; fv.x = META_RAW; }
        };
        const unsigned vt = v16 >> 1, ve = v16 >> 3;
        auto request = [&]() {
            LTR(lt0, next);
            meta = __builtin_amdgcn_readlane(fv.x, next);
            const unsigned char* a = xr + (size_t)next * 4 * ITEM;
            if (meta == META_RAW) { if (next & 1) item_loads<140>(a, v16, v16, ve); else item_loads<120>(a, v16, v16, ve); }
            else { if (next & 1) item_loads<140>(a, v16, vt, ve); else item_loads<120>(a, v16, vt, ve); }
            ++next;
        };
        auto poll_async = [&]() {
            asm volatile("global_load_dwordx2 v[158:159], %0, %1 sc0 sc1" :: "v"(v8), "s"(fl) : "memory", "v158", "v159");
            polled = true;
        };
        auto take_poll = [&]() {
            asm volatile("v_mov_b64 %0, v[158:159]" : "=v"(fv) :: "memory");
            look(); polled = false;
        };
        auto ahead = [&]() {
            if (next < NSTEP) {
                if (seen >> next & 1u) { request(); pend = true; }
                poll_async();
            }
        };
        auto flip = [&](unsigned pos, bool on) {
            if (on) asm volatile("ds_xor_b32 %0, %1" :: "v"(aslot + (pos & 0xffcu)), "v"(0x40u << ((pos & 3u) * 8u)) : "memory");
        };
        auto expand_from = [&](auto rc, unsigned m, int k) {
            constexpr int R = decltype(rc)::v;
            const unsigned oa = aslot + v16;
            if (m != META_RAW) {
                LAS unsigned char* o = (LAS unsigned char*)(size_t)oa;
                const unsigned b4 = (m & 0xffu) * 0x01010101u, cnt = m >> 8;
                unsigned d[16];
                unpack8<R>(d, b4);
                *(LAS u32x4*)o = (u32x4){d[0], d[1], d[2], d[3]}; *(LAS u32x4*)(o + 1024) = (u32x4){d[4], d[5], d[6], d[7]};
                __builtin_amdgcn_sched_barrier(0);
                unpack8<R + 7>(d + 8, b4);
                *(LAS u32x4*)(o + 2048) = (u32x4){d[8], d[9], d[10], d[11]}; *(LAS u32x4*)(o + 3072) = (u32x4){d[12], d[13], d[14], d[15]};
                flip(reg_get<R + 16>(), (unsigned)lane < cnt);
                for (unsigned c0 = 64; c0 < cnt && c0 < (unsigned)EXC_MAX; c0 += 64) {
                    unsigned e;
                    const unsigned eo = (c0 + lane) * 2;
                    const unsigned char* ea = xr + (size_t)k * 4 * ITEM;
                    asm volatile("global_load_ushort %0, %1, %2 offset:3584 sc0 sc1\n\ts_waitcnt vmcnt(0)" : "=&v"(e) : "v"(eo), "s"(ea) : "memory");
                    flip(e, c0 + lane < cnt);
                }
            } else raw_stores<R>(oa);
        };
        auto expand = [&](unsigned m, int k) {
            if (k & 1) expand_from(IC<140>{}, m, k); else expand_from(IC<120>{}, m, k);
            advance(aslot, lds0 + h * ITEM);
        };
#pragma unroll 1
        for (int s = -1; s < NSTEP - 1; ++s) {
            HT(0);
            if (pflast) wait_vm<4>(); else wait_vm<0>();
            if (polled) take_poll();
            if (pend) {
                HT(1);
                const unsigned m = meta;
                pend = false;
                ahead();
                expand(m, s + 1);
                HT(3);
                extra(s + 2);
            } else if (!dead) {
                extra(s + 2);
                if (!(seen >> next & 1u)) {
                    const unsigned long long t0 = __builtin_amdgcn_s_memrealtime();
                    for (unsigned polls = 1;; ++polls) {
                        asm volatile("global_load_dwordx2 %0, %1, %2 sc0 sc1\n\ts_waitcnt vmcnt(0)" : "=&v"(fv) : "v"(v8), "s"(fl) : "memory");
                        look();
                        if (seen >> next & 1u) break;
                        if ((polls & 63u) == 0u
                            && (__builtin_amdgcn_readfirstlane(__hip_atomic_load(tmo, __ATOMIC_RELAXED, __HIP_MEMORY_SCOPE_AGENT)) == call
                                || __builtin_amdgcn_s_memrealtime() - t0 > DEADLINE)) {
                            if (lane == 0) { __hip_atomic_store(tmo, call, __ATOMIC_RELAXED, __HIP_MEMORY_SCOPE_AGENT); lds_st(lds0 + SY_BAD, 1u); }
                            dead = true;
                            break;
                        }
                        __builtin_amdgcn_s_sleep(POLL_SLEEP);
                    }
                    if (h == 0 && next >= NSTEP - 3) STAMP(16 + next);
                }
                if (!dead) {
                    request(); poll_async();
                    wait_vm<0>();
                    const unsigned m = meta;
                    take_poll();
                    ahead();
                    expand(m, s + 1);
                }
            }
            pflast = s == pfs;
            if (pflast) {
                prefetch_row(pfp); prefetch_row(pfp + ND);
                pfs = pfs + 7 + PF_AHEAD < NSTEP ? pfs + 7 : NSTEP;
                pfp += (size_t)7 * BK * ND;
            }
            LTR(lt1, s + 1);
            barrier();
            LTR(lt2, s + 1);
        }
        wait_vm<0>();
        return;
    }

    const int wr = wid >> 2, wc = wid & 3;
    unsigned wo[4];
#pragma unroll
    for (int c = 0; c < 4; ++c)
        wo[c] = (unsigned)((c & 1) * 4 + (lane >> 4)) * ND + (unsigned)(((lane & 15) ^ (((c & 1) * 4 + (lane >> 4)) | ((c >> 1) << 3))) << 4);
    const unsigned char* wp = p.w + nt * BN + (size_t)wc * 32 * ND;
    unsigned wslot = lds0 + B_OFF + wc * 8192;
    auto issue_w = [&]() {
        if (!DIAG_MODE(4))
            static_for<0, 8>([&](auto cc) { constexpr int c = decltype(cc)::v; glds<c * 1024>(wp + (size_t)(c >> 1) * 8 * ND, wo[(c & 1) | ((c >> 2) << 1)], wslot); });
        wp += (size_t)BK * ND;
        wslot = wslot >= lds0 + B_OFF + (NSLOT - 1) * SLOT_BYTES ? wslot - (NSLOT - 1) * SLOT_BYTES : wslot + SLOT_BYTES;
    };
    if (wr) { issue_w(); issue_w(); }
    asm volatile("" : "+v"(wo[0]), "+v"(wo[1]), "+v"(wo[2]), "+v"(wo[3]));
    unsigned sx = __builtin_amdgcn_readfirstlane(__float_as_uint(p.sx[0])), sw = __builtin_amdgcn_readfirstlane(__float_as_uint(p.sw[0]));
    int ex = 2 - t, top = 1 - wr;
    f32x4 acc[4][4], accx[2];
#pragma unroll
    for (int m = 0; m < 4; ++m)
#pragma unroll
        for (int n = 0; n < 4; ++n) acc[m][n] = (f32x4){0.f, 0.f, 0.f, 0.f};
    accx[0] = accx[1] = (f32x4){0.f, 0.f, 0.f, 0.f};
    const float* ob = p.out + (size_t)(src * MS + t * BM + wr * 64) * NS + nt * BN + wc * 64;
    const float* ox = p.out + (size_t)(p.rank * MS + xrow) * NS + nt * BN + wc * 64 + wr * 32;
    asm volatile("" : "+s"(ob), "+s"(ox), "+s"(sx), "+s"(sw), "+s"(ex), "+s"(top));
    unsigned aa[2], ea[2], bo[4];
    {
        const int fr = lane & 15, fq = lane >> 4, q = fr >> 1, f = q | ((fq & 1) << 3);
        aa[0] = lds0 + (unsigned)((wr * 64 + fr) * 128 + ((fq ^ ((fr >> 1) & 7)) << 4));
        aa[1] = aa[0] ^ 64u;
        ea[0] = aa[0] + (unsigned)(E_OFF - wr * 8192);
        ea[1] = ea[0] ^ 64u;
#pragma unroll
        for (int n = 0; n < 4; ++n) bo[n] = lds0 + (unsigned)((16 * fq + q) * 256 + (((wc * 4 + (n ^ (2 * wr))) ^ f) << 4) + (fr & 1) * 8);
    }
    asm volatile(".p2align 8\n\t.rept %0\n\ts_nop 0\n\t.endr\n\ts_waitcnt vmcnt(8)\n\ts_barrier" :: "i"(LOOP_PAD) : "memory");
    {
        int slot = 0;
#pragma unroll 1
        for (int s = 0; s < NSTEP; ++s) {
            v8i af[4], bf[4], ef;
#pragma unroll
            for (int n = 0; n < 4; ++n) {
                LAS unsigned char* q = (LAS unsigned char*)(size_t)bo[n] + B_OFF;
                const v2i t0 = __builtin_amdgcn_ds_read_tr8_b64_v2i32((LAS v2i*)q);
                const v2i t1 = __builtin_amdgcn_ds_read_tr8_b64_v2i32((LAS v2i*)(q + 8 * 256));
                const v2i t2 = __builtin_amdgcn_ds_read_tr8_b64_v2i32((LAS v2i*)(q + 64 * 256));
                const v2i t3 = __builtin_amdgcn_ds_read_tr8_b64_v2i32((LAS v2i*)(q + 72 * 256));
                bf[n] = (v8i){t0.x, t0.y, t1.x, t1.y, t2.x, t2.y, t3.x, t3.y};
            }
#pragma unroll
            for (int m = 0; m < 4; ++m) {
                const v4i lo = *(LAS v4i*)((LAS unsigned char*)(size_t)aa[0] + m * 2048), hi = *(LAS v4i*)((LAS unsigned char*)(size_t)aa[1] + m * 2048);
                af[m] = (v8i){lo.x, lo.y, lo.z, lo.w, hi.x, hi.y, hi.z, hi.w};
            }
            asm volatile("" : "+s"(ex));
            if (ex > 0) {
                const v4i lo = *(LAS v4i*)(size_t)ea[0], hi = *(LAS v4i*)(size_t)ea[1];
                ef = (v8i){lo.x, lo.y, lo.z, lo.w, hi.x, hi.y, hi.z, hi.w};
            }
            int wdue = top ? 0 : NSTEP - 2 - s;
            asm volatile("" : "+s"(wdue));
            if (wdue > 0) issue_w();
            asm volatile("" ::: "memory");
            __builtin_amdgcn_s_waitcnt(0xC07F);
            slot = (slot == NSLOT - 1) ? 0 : slot + 1;
            const int delta = slot ? SLOT_BYTES : -(NSLOT - 1) * SLOT_BYTES;
            int bar = s < NSTEP - 1 ? 1 + top : 0;
            asm volatile("" : "+s"(bar));
            if (bar == 1) { if (wdue > 0) wait_vm<8>(); else wait_vm<0>(); asm volatile("s_barrier" ::: "memory"); }
            __builtin_amdgcn_sched_barrier(0);
            asm volatile("s_setprio 1\n\t.p2align 3");
            __builtin_amdgcn_sched_barrier(0);
#pragma unroll
            for (int m = 0; m < 4; ++m) {
#pragma unroll
                for (int n = 0; n < 4; ++n) {
                    acc[m][n] = __builtin_amdgcn_mfma_scale_f32_16x16x128_f8f6f4(bf[n], af[m], acc[m][n], 0, 0, 0, 0, 0, 0);
                    if (m == 0 && n == 0) { __builtin_amdgcn_sched_barrier(0); __builtin_amdgcn_s_setprio(2); __builtin_amdgcn_sched_barrier(0); }
                    if (m == 3 && n == 1) {
                        __builtin_amdgcn_sched_barrier(0);
                        asm volatile("" : "+s"(bar));
                        if (bar == 2) asm volatile("s_barrier" ::: "memory");
                        __builtin_amdgcn_sched_barrier(0);
                    }
                }
                __builtin_amdgcn_sched_barrier(0);
                if (m == 0) asm volatile("v_add_u32 %0, %4, %0\n\tv_add_u32 %1, %4, %1\n\tv_add_u32 %2, %4, %2\n\tv_add_u32 %3, %4, %3"
                                         : "+v"(bo[0]), "+v"(bo[1]), "+v"(bo[2]), "+v"(bo[3]) : "s"(delta));
                if (m == 1) asm volatile("v_add_u32 %0, %4, %0\n\tv_add_u32 %1, %4, %1\n\tv_add_u32 %2, %4, %2\n\tv_add_u32 %3, %4, %3"
                                         : "+v"(aa[0]), "+v"(aa[1]), "+v"(ea[0]), "+v"(ea[1]) : "s"(delta));
                __builtin_amdgcn_sched_barrier(0);
            }
            asm volatile("" : "+s"(ex));
            if (ex > 0) {
                accx[0] = __builtin_amdgcn_mfma_scale_f32_16x16x128_f8f6f4(bf[0], ef, accx[0], 0, 0, 0, 0, 0, 0);
                accx[1] = __builtin_amdgcn_mfma_scale_f32_16x16x128_f8f6f4(bf[1], ef, accx[1], 0, 0, 0, 0, 0, 0);
            }
            __builtin_amdgcn_s_setprio(0);
            __builtin_amdgcn_sched_barrier(0);
        }
    }
    if (wid == 0) { STAMP(1); STAMPC(5); }
    STAMP(48 + wid);
    float scale = __uint_as_float(sx) * __uint_as_float(sw);
    if (__builtin_amdgcn_readfirstlane(lds_ld(lds0 + SY_BAD)) != 0u) scale = __builtin_nanf("");
    int ln;
    asm volatile("v_mbcnt_lo_u32_b32 %0, -1, 0\n\tv_mbcnt_hi_u32_b32 %0, -1, %0" : "=v"(ln));
    const int fr = ln & 15, fq = ln >> 4;
    const float zs = scale * -1.44269504088896340736f, zmax = 60.f * 1.44269504088896340736f;
    LAS unsigned char* tb = (LAS unsigned char*)smem + ((NSTEP - 3 + (wc >> 1)) % NSLOT) * SLOT_BYTES + wr * 8192 + (wc & 1) * 4096;
    f32x2 e[8];
    f32x4 tv[4];
    auto expo = [&](auto nqc, const f32x4* c) {
        constexpr int NQ = decltype(nqc)::v;
#pragma unroll
        for (int h = 0; h < 2 * NQ; ++h) {
            f32x2 z = (f32x2){c[h >> 1][(h & 1) * 2], c[h >> 1][(h & 1) * 2 + 1]} * zs;
            asm("" : "+v"(z));
            e[h].x = __builtin_amdgcn_exp2f(__builtin_amdgcn_fmed3f(z.x, -zmax, zmax));
            e[h].y = __builtin_amdgcn_exp2f(__builtin_amdgcn_fmed3f(z.y, -zmax, zmax));
        }
        __builtin_amdgcn_sched_barrier(0);
    };
    auto stage = [&](auto nqc, const f32x4* c) {
        constexpr int NQ = decltype(nqc)::v;
#pragma unroll
        for (int q = 0; q < NQ; ++q) {
            f32x2 d0 = e[2 * q] + 1.0f, d1 = e[2 * q + 1] + 1.0f;
            asm("" : "+v"(d0)); asm("" : "+v"(d1));
            const f32x4 r = {__builtin_amdgcn_rcpf(d0.x), __builtin_amdgcn_rcpf(d0.y), __builtin_amdgcn_rcpf(d1.x), __builtin_amdgcn_rcpf(d1.y)};
            const int cb = NQ == 4 ? (q ^ (2 * wr)) : q;
            *(LAS f32x4*)(tb + fr * 256 + (((cb * 4 + fq) ^ (fr & 7)) << 4)) = (c[q] * scale) * r;
        }
        constexpr int LPR = NQ * 4, RPI = 64 / LPR;
#pragma unroll
        for (int i = 0; i < NQ; ++i) {
            const int r = i * RPI + ln / LPR;
            tv[i] = *(LAS f32x4*)(tb + r * 256 + (((ln % LPR) ^ (r & 7)) << 4));
        }
        __builtin_amdgcn_sched_barrier(0);
    };
    auto put = [&](auto nqc, const float* rowp) {
        constexpr int NQ = decltype(nqc)::v, LPR = NQ * 4, RPI = 64 / LPR;
        const unsigned ovoff = (unsigned)((ln / LPR) * NS * 4 + (ln % LPR) * 16);
#pragma unroll
        for (int i = 0; i < NQ; ++i)
            asm volatile("global_store_dwordx4 %0, %1, %2 sc1\n\ts_nop 1" :: "v"(ovoff), "v"(tv[i]), "s"(rowp + (size_t)i * RPI * NS) : "memory");
        __builtin_amdgcn_sched_barrier(0);
    };
    expo(IC<4>{}, acc[0]);
#pragma unroll
    for (int m = 0; m < 4; ++m) {
        stage(IC<4>{}, acc[m]);
        if (m < 3) expo(IC<4>{}, acc[m + 1]);
        else if (has_extra) expo(IC<2>{}, accx);
        put(IC<4>{}, ob + (size_t)m * 16 * NS);
    }
    if (has_extra) { stage(IC<2>{}, accx); put(IC<2>{}, ox); }
}

}

extern "C" void kernel_launch(int rank, int nranks, ncclComm_t comm,
                              hipStream_t stream,
                              void* const* d_in, const int* in_sizes, int n_in,
                              void* d_out, int out_size,
                              const int* in_dtypes, int out_dtype,
                              void* const* d_out_all,
                              void* const* d_ws_all, size_t ws_bytes) {
    (void)comm; (void)in_dtypes; (void)out_dtype; (void)d_out_all;
    if (nranks != NR || (unsigned)rank >= (unsigned)NR || n_in != 4 || ws_bytes < WS_END || in_sizes[0] != MS * KD
        || in_sizes[1] != KD * ND || out_size != NR * MS * NS) {
        fprintf(stderr, "kernel_launch: unexpected problem shape\n");
        return;
    }
    HIP_CHECK(hipFuncSetAttribute((const void*)fused_ag_gemm, hipFuncAttributeMaxDynamicSharedMemorySize, LDS_BYTES));
    Params p;
    p.x = (const unsigned char*)d_in[0];
    p.w = (const unsigned char*)d_in[1] + (size_t)rank * NS;
    p.sx = (const float*)d_in[2];
    p.sw = (const float*)d_in[3];
    p.out = (float*)d_out;
    p.me = (unsigned char*)d_ws_all[rank];
    for (int i = 0; i < NR - 1; ++i) p.peer[i] = (unsigned char*)d_ws_all[(rank + 1 + i) % NR];
    p.rank = rank;
    p.mode = 0;
    hipLaunchKernelGGL(fused_ag_gemm, dim3(NWG), dim3(NTHREADS), LDS_BYTES, stream, p);
    HIP_CHECK(hipGetLastError());
}
```

```cpp
#include <hip/hip_runtime.h>
#include <rccl/rccl.h>
#include <cstdio>

#define HIP_CHECK(expr) do { hipError_t _e = (expr); if (_e != hipSuccess) { \
    fprintf(stderr, "HIP error %s at %s:%d\n", hipGetErrorString(_e), __FILE__, __LINE__); \
    return; } } while (0)

namespace {

#define LAS __attribute__((address_space(3)))
typedef int v8i __attribute__((ext_vector_type(8)));
typedef int v4i __attribute__((ext_vector_type(4)));
typedef int v2i __attribute__((ext_vector_type(2)));
typedef float f32x2 __attribute__((ext_vector_type(2)));
typedef float f32x4 __attribute__((ext_vector_type(4)));
typedef unsigned u32x2 __attribute__((ext_vector_type(2)));
typedef unsigned u32x4 __attribute__((ext_vector_type(4)));

constexpr int NR = 8;
constexpr int KD = 4096, ND = 16384;
constexpr int MS = 512, NS = 2048;
constexpr int BM = 128, BN = 256, BK = 128;
constexpr int NSTEP = KD / BK;
constexpr int MT = MS / BM;
constexpr int NWG = 256, NTHREADS = 768;
constexpr int NSLOT = 3;
constexpr int A_BYTES = BM * BK, E_BYTES = 16 * BK, B_BYTES = BK * BN;
constexpr int E_OFF = A_BYTES, B_OFF = A_BYTES + E_BYTES, SLOT_BYTES = B_OFF + B_BYTES;
constexpr int SYNC_OFF = NSLOT * SLOT_BYTES;
constexpr int SY_BAD = SYNC_OFF;
constexpr int SY_PROG = SYNC_OFF, SY_META = SYNC_OFF + 64;
constexpr int JUNK_OFF = SYNC_OFF + 512;
constexpr int LDS_BYTES = JUNK_OFF + 2048;
static_assert(LDS_BYTES <= 163840, "LDS");

constexpr int NPUSH = 16;
constexpr int ITEM = 4096;
constexpr int NITEM = MT * NSTEP * 4;
constexpr int NSTATIC = 4 * NPUSH;
constexpr int NDRAW = 512;
static_assert(NITEM - NSTATIC + 4 * NPUSH == NDRAW && 32 * ITEM <= SYNC_OFF, "draw count; staging ring");
constexpr int EXC_OFF = 3584, EXC_MAX = 256;
constexpr unsigned META_RAW = 0x80000000u;

constexpr size_t SHARD = (size_t)MS * KD;
constexpr size_t XBUF = (size_t)NR * SHARD;
constexpr size_t FLAG_SRC = (size_t)NITEM * 8, FLAG_PAR = NR * FLAG_SRC;
constexpr size_t WS_FLAG = 2 * XBUF;
constexpr size_t WS_CNT = WS_FLAG + 2 * FLAG_PAR;
constexpr size_t WS_TMO = WS_CNT + (size_t)NWG * 8 * 128;
constexpr size_t WS_DRAW = WS_TMO + 1024;
constexpr size_t WS_DBG = WS_TMO + 4096;
constexpr size_t WS_TRACE = WS_DBG + (size_t)NWG * 64 * 8;
constexpr size_t WS_END = WS_TRACE + 8192;
constexpr unsigned long long DEADLINE = 50000000ull;

#define LOOP_PAD 0
#define FASTP 1
#define LOCK 1
#define PF_LATE 0
#define WT 2
#define WB 2
#define RAW_R 4, 4, 9, 9
#define MARKER 0
#define COMPRESS 1
#define BASE0 4
#define HPRIO 3
#define PREFETCH 1
#define PF_DELAY 100
#define PF_AHEAD 7
#define STAGE_AHEAD 4
#define POLL_SLEEP 2
#define ORD_FIN 850, 860, 1000, 1000

struct Params {
    const unsigned char* x;
    unsigned char* me;
    const unsigned char* w;
    float* out;
    unsigned char* peer[NR - 1];
    int rank;
    int mode;
    const float* sx; const float* sw;
};

template <int N> __device__ __forceinline__ void wait_vm() { asm volatile("s_waitcnt vmcnt(%0)" :: "i"(N) : "memory"); }
template <int I> struct IC { static constexpr int v = I; };
template <int I, int E, class F> __device__ __forceinline__ void static_for(F&& f) { if constexpr (I < E) { f(IC<I>{}); static_for<I + 1, E>(f); } }
__device__ __forceinline__ unsigned lds_ld(unsigned addr) {
    unsigned v; asm volatile("ds_read_b32 %0, %1\n\ts_waitcnt lgkmcnt(0)" : "=v"(v) : "v"(addr) : "memory"); return v;
}
__device__ __forceinline__ void lds_st(unsigned addr, unsigned v) { asm volatile("ds_write_b32 %0, %1" :: "v"(addr), "v"(v) : "memory"); }
template <int OFF> __device__ __forceinline__ void glds(const unsigned char* base, unsigned voff, unsigned dst) {
    asm volatile("s_add_u32 m0, %2, %3\n\ts_nop 0\n\tglobal_load_lds_dwordx4 %0, %1" :: "v"(voff), "s"(base), "s"(dst), "i"(OFF) : "memory", "scc");
}
constexpr int WH = 4 - WT - WB;
__device__ __forceinline__ unsigned w_off(int lane, int c, int pair) {
    return (unsigned)(c * 4 + (lane >> 4)) * ND + (unsigned)(((lane & 15) ^ ((c * 4 + (lane >> 4)) | (((pair >> 1) & 1) << 3))) << 4) - c * 1024u;
}
__device__ __forceinline__ void w_pair(const unsigned char* wp, unsigned o0, unsigned o1, unsigned dst) {
    asm volatile("s_mov_b32 m0, %3\n\ts_nop 0\n\tglobal_load_lds_dwordx4 %0, %2\n\tglobal_load_lds_dwordx4 %1, %2 offset:1024"
                 :: "v"(o0), "v"(o1), "s"(wp), "s"(dst) : "memory");
}
__device__ __forceinline__ void w_pairs2(const unsigned char* wp, unsigned o0, unsigned o1, unsigned o2, unsigned o3, unsigned dst) {
    asm volatile("s_mov_b32 m0, %6\n\ts_nop 0\n\tglobal_load_lds_dwordx4 %0, %4\n\tglobal_load_lds_dwordx4 %1, %4 offset:1024\n\t"
                 "global_load_lds_dwordx4 %2, %5 offset:2048\n\tglobal_load_lds_dwordx4 %3, %5 offset:3072"
                 :: "v"(o0), "v"(o1), "v"(o2), "v"(o3), "s"(wp), "s"(wp + 8 * ND - 2048), "s"(dst) : "memory");
}
__device__ __forceinline__ unsigned ticket_issue(unsigned char* me, int role) {
    unsigned t = 0;
    unsigned char* a = me + WS_CNT + ((size_t)blockIdx.x * 8 + role) * 128;
    if ((threadIdx.x & 63) == 0) asm volatile("global_atomic_add %0, %1, %2, off sc0" : "+v"(t) : "v"(a), "v"(1u) : "memory");
    return t;
}
__device__ __forceinline__ unsigned bfi(unsigned mask, unsigned a, unsigned b) { return (a & mask) | (b & ~mask); }

__device__ __forceinline__ unsigned rebase(unsigned d, unsigned b4) { return bfi(0x7f7f7f7fu, (d | 0x80808080u) - b4, d); }
__device__ __forceinline__ void pack8(const unsigned* d, unsigned* q) {
#pragma unroll
    for (int i = 0; i < 7; ++i) q[i] = bfi(0x40404040u, i < 6 ? d[7] << (1 + i) : d[7] >> 1, d[i]);
}
#define XREGS "v120", "v121", "v122", "v123", "v124", "v125", "v126", "v127", "v128", "v129", "v130", "v131", "v132", "v133", "v134", "v135", "v136", \
              "v140", "v141", "v142", "v143", "v144", "v145", "v146", "v147", "v148", "v149", "v150", "v151", "v152", "v153", "v154", "v155", "v156"
template <int R> __device__ __forceinline__ void item_loads(const unsigned char* a, unsigned v16, unsigned tail, unsigned ve) {
    asm volatile("global_load_dwordx4 v[%c4:%c5], %0, %3 sc0 sc1\n\tglobal_load_dwordx4 v[%c6:%c7], %0, %3 offset:1024 sc0 sc1\n\t"
                 "global_load_dwordx4 v[%c8:%c9], %0, %3 offset:2048 sc0 sc1\n\tglobal_load_dwordx4 v[%c10:%c11], %1, %3 offset:3072 sc0 sc1\n\t"
                 "global_load_ushort v%c12, %2, %3 offset:3584 sc0 sc1"
                 :: "v"(v16), "v"(tail), "v"(ve), "s"(a), "i"(R), "i"(R + 3), "i"(R + 4), "i"(R + 7), "i"(R + 8), "i"(R + 11),
                    "i"(R + 12), "i"(R + 15), "i"(R + 16) : "memory", XREGS);
}
template <int R, int OFF> __device__ __forceinline__ void load16(const unsigned char* a, unsigned v) {
    asm volatile("global_load_dwordx4 v[%c2:%c3], %0, %1 offset:%4 sc0 sc1" :: "v"(v), "s"(a), "i"(R), "i"(R + 3), "i"(OFF) : "memory", XREGS);
}
template <int R> __device__ __forceinline__ void load2(const unsigned char* a, unsigned v) {
    asm volatile("global_load_ushort v%c2, %0, %1 offset:3584 sc0 sc1" :: "v"(v), "s"(a), "i"(R) : "memory", XREGS);
}
__device__ __forceinline__ void flags_peek(unsigned long long& set, unsigned& meta, unsigned call, int k) {
    asm volatile("v_cmp_le_u32 %0, %2, v159\n\tv_readlane_b32 %1, v158, %3" : "=&s"(set), "=s"(meta) : "s"(call), "s"(k & 31));
}
__device__ __forceinline__ void raw_loads(const unsigned char* a, unsigned v16, unsigned ve, unsigned dst) {
    asm volatile("s_mov_b32 m0, %3\n\ts_nop 0\n\tglobal_load_lds_dwordx4 %0, %2 sc0 sc1\n\tglobal_load_lds_dwordx4 %0, %2 offset:1024 sc0 sc1\n\t"
                 "global_load_lds_dwordx4 %0, %2 offset:2048 sc0 sc1\n\tglobal_load_lds_dwordx4 %0, %2 offset:3072 sc0 sc1\n\t"
                 "global_load_ushort v137, %1, %2 sc0 sc1" :: "v"(v16), "v"(ve), "s"(a), "s"(dst) : "memory", "v137");
}
template <int R> __device__ __forceinline__ unsigned reg_get() { unsigned v; asm volatile("v_mov_b32 %0, v%c1" : "=v"(v) : "i"(R)); return v; }
template <int R> __device__ __forceinline__ unsigned carrier() { unsigned c; asm volatile("v_and_b32 %0, 0x40404040, v%c1" : "=v"(c) : "i"(R)); return c; }
template <int R> __device__ __forceinline__ unsigned restore(unsigned c, unsigned b4) {
    unsigned d; asm volatile("v_xad_u32 %0, v%c1, %2, %3" : "=v"(d) : "i"(R), "v"(c), "s"(b4)); return d;
}
template <int R, class H> __device__ __forceinline__ void unpack8(unsigned* d, unsigned b4, H&& hook) {
    const unsigned c[7] = {carrier<R>(), carrier<R + 1>(), carrier<R + 2>(), carrier<R + 3>(), carrier<R + 4>(), carrier<R + 5>(), carrier<R + 6>()};
    hook(0);
    d[0] = restore<R>(c[0], b4); d[1] = restore<R + 1>(c[1], b4); d[2] = restore<R + 2>(c[2], b4); d[3] = restore<R + 3>(c[3], b4);
    d[4] = restore<R + 4>(c[4], b4); d[5] = restore<R + 5>(c[5], b4); d[6] = restore<R + 6>(c[6], b4);
    hook(1);
    const unsigned t0 = (c[0] << 1) | c[1], t1 = (c[2] << 1) | c[3], t2 = (c[4] << 1) | c[5];
    d[7] = ((t0 >> 2) + (t1 >> 4) + (t2 >> 6)) + ((c[6] << 1) + b4);
}
__device__ __forceinline__ unsigned tops8(const unsigned* d) {
    unsigned m = 0;
#pragma unroll
    for (int i = 0; i < 8; ++i) m |= (i < 7 ? d[i] >> (6 - i) : d[i] << 1) & (0x01010101u << i);
    return m;
}

#define HT(j) do {} while (0)
#define HSAVE() do {} while (0)
#define STAMP(i) do {} while (0)
#define STAMPC(i) do {} while (0)
#define DIAG_MODE(bit) false
#define LTR(v, i) do {} while (0)
#define TRACE_WG 255
#define TRACE_LA 17

#define TILE_INDEX(b) ((((b) - NPUSH) & 7) * 30 + (((b) - NPUSH) >> 3))
#define DIAG_GROUP(b) ((b) < NPUSH ? -1 : (TILE_INDEX(b) >> 3) >= 28 ? 4 : ((TILE_INDEX(b) >> 3) & 3))
#define DIAG_LOCAL(b) ((b) < NPUSH || (TILE_INDEX(b) >> 3) >= 28)

struct Order { unsigned q[NITEM]; };
constexpr Order make_order() {
    constexpr int fin[4] = {ORD_FIN};
    Order o{}; int key[NITEM] = {}; int n = 0;
    for (int k = 0; k < NSTEP; ++k)
        for (int t = 0; t < MT; ++t)
            for (int j = 0; j < 4; ++j) {
                const int due = k < 4 ? 0 : fin[t] * (k + 1);
                int i = n++;
                while (i > 0 && key[i - 1] > due) { key[i] = key[i - 1]; o.q[i] = o.q[i - 1]; --i; }
                key[i] = due; o.q[i] = (unsigned)((t * NSTEP + k) * 4 + j);
            }
    return o;
}
__constant__ const Order k_order = make_order();
__device__ __forceinline__ int static_item(int id) { return (((id >> 2) & 3) * NSTEP + (id >> 4)) * 4 + (id & 3); }
__device__ __forceinline__ size_t item_src(int q) { return (size_t)((q >> 7) * BM + (q & 3) * 32) * KD + ((q >> 2) & 31) * BK; }
__device__ __forceinline__ size_t item_flag(int q) { return (size_t)((((q >> 7) * 4 + (q & 3)) * NSTEP) + ((q >> 2) & 31)) * 8; }

__global__ __launch_bounds__(NTHREADS) void fused_ag_gemm(Params p) {
    extern __shared__ __attribute__((aligned(16))) unsigned char smem[];
    const int tid = threadIdx.x, lane = tid & 63, wid = __builtin_amdgcn_readfirstlane(tid >> 6);
    const unsigned lds0 = (unsigned)(size_t)(LAS unsigned char*)smem;
    unsigned char* me = p.me;
    const int b = blockIdx.x;
    const unsigned aoff0 = (unsigned)(lane >> 3) * KD + (unsigned)(((lane & 7) ^ (lane >> 4)) << 4);
    const unsigned aoff1 = (unsigned)(lane >> 3) * KD + (unsigned)(((lane & 7) ^ (4 + (lane >> 4))) << 4);
    unsigned v16 = (unsigned)lane * 16u;

    if (DIAG_MODE(32)) return;
    if (wid == 0) { STAMP(0); STAMPC(4); }

    if (__builtin_expect(b < NPUSH, 1)) {
        if (tid < 128) *(LAS unsigned*)(smem + SYNC_OFF + tid * 4) = 0u;
        auto load_item = [&](int q, u32x4& r0, u32x4& r1, u32x4& r2, u32x4& r3) {
            asm volatile("global_load_dwordx4 %0, %4, %8\n\tglobal_load_dwordx4 %1, %5, %8\n\t"
                         "global_load_dwordx4 %2, %6, %8\n\tglobal_load_dwordx4 %3, %7, %8"
                         : "=&v"(r0), "=&v"(r1), "=&v"(r2), "=&v"(r3)
                         : "v"(aoff0), "v"(aoff1 + 8 * KD), "v"(aoff0 + 16 * KD), "v"(aoff1 + 24 * KD), "s"(p.x + item_src(q)) : "memory");
        };
        if (wid >= 8) {
            const int L = wid - 8;
            unsigned base = BASE0;
            __syncthreads();
            if (DIAG_MODE(2)) return;
#pragma unroll 1
            for (int i = L ? L : 4;; i += 4) {
                if (i > STAGE_AHEAD) {
                    for (;;) {
                        const unsigned pr = lds_ld(lds0 + SY_PROG + (lane % 7) * 4);
                        if (__any(pr >= (unsigned)(i - STAGE_AHEAD)) && __all(pr + 32u > (unsigned)i)) break;
                        __builtin_amdgcn_s_sleep(2);
                    }
                }
                volatile LAS u32x2* mp = (volatile LAS u32x2*)(smem + SY_META + (i & 31) * 8);
                int q;
                if (i < 4) q = static_item(i * NPUSH + b);
                else {
                    unsigned d = 0;
                    if (lane == 0) d = __hip_atomic_fetch_add((unsigned*)(me + WS_DRAW), 1u, __ATOMIC_RELAXED, __HIP_MEMORY_SCOPE_AGENT);
                    const unsigned id = (__builtin_amdgcn_readfirstlane(d) & (NDRAW - 1)) + NSTATIC;
                    if (id >= (unsigned)NITEM) { *mp = (u32x2){(unsigned)(i + 1) << 16 | 0xffffu, 0u}; return; }
                    q = (int)k_order.q[id];
                }
                u32x4 r0, r1, r2, r3;
                load_item(q, r0, r1, r2, r3);
                asm volatile("s_waitcnt vmcnt(0)" : "+v"(r0), "+v"(r1), "+v"(r2), "+v"(r3) :: "memory");
                const unsigned d[16] = {r0.x, r0.y, r0.z, r0.w, r1.x, r1.y, r1.z, r1.w, r2.x, r2.y, r2.z, r2.w, r3.x, r3.y, r3.z, r3.w};
                LAS unsigned char* slot = (LAS unsigned char*)smem + (i & 31) * ITEM;
                unsigned meta = META_RAW, g[16];
                constexpr int rawr[4] = {RAW_R};
                const int rr = (q >> 7) == 0 ? rawr[0] : (q >> 7) == 1 ? rawr[1] : (q >> 7) == 2 ? rawr[2] : rawr[3];
                if (COMPRESS && ((q >> 2) & 31) < NSTEP - rr) {
                    for (int attempt = 0; attempt < 2; ++attempt) {
#pragma unroll
                        for (int e = 0; e < 16; ++e) g[e] = rebase(d[e], base * 0x01010101u);
                        unsigned long long m = tops8(g) | (unsigned long long)tops8(g + 8) << 32;
                        const unsigned limit = attempt ? EXC_MAX : 96;
                        unsigned cnt = 0;
                        for (;;) {
                            const unsigned long long act = __ballot(m != 0);
                            if (!act) { meta = base | cnt << 8; break; }
                            const unsigned n = (unsigned)__popcll(act);
                            if (cnt + n > limit) break;
                            if (m) {
                                const unsigned bit = (unsigned)__builtin_ctzll(m), dw = (bit >> 5) * 8 + (bit & 7);
                                const unsigned at = cnt + __builtin_amdgcn_mbcnt_hi((unsigned)(act >> 32), __builtin_amdgcn_mbcnt_lo((unsigned)act, 0u));
                                *(LAS unsigned short*)(slot + EXC_OFF + at * 2) = (unsigned short)((dw >> 2) * 1024 + v16 + (dw & 3) * 4 + ((bit >> 3) & 3));
                                m &= m - 1;
                            }
                            cnt += n;
                        }
                        if (meta != META_RAW || attempt) break;
                        unsigned mx = 0;
#pragma unroll
                        for (int e = 0; e < 16; ++e) {
                            unsigned v = d[e];
                            asm volatile("" : "+v"(v));
#pragma unroll
                            for (int s = 0; s < 32; s += 8) mx = max(mx, (v >> s) & 0x7fu);
                        }
#pragma unroll
                        for (int o = 32; o; o >>= 1) mx = max(mx, (unsigned)__shfl_xor((int)mx, o));
                        mx = __builtin_amdgcn_readfirstlane(mx);
                        base = mx > 67u ? mx - 67u : 0u;
                    }
                }
                if (meta == META_RAW) {
                    *(LAS u32x4*)(slot + v16) = r0; *(LAS u32x4*)(slot + 1024 + v16) = r1;
                    *(LAS u32x4*)(slot + 2048 + v16) = r2; *(LAS u32x4*)(slot + 3072 + v16) = r3;
                } else {
                    unsigned k[14];
                    pack8(g, k); pack8(g + 8, k + 7);
                    *(LAS u32x4*)(slot + v16) = (u32x4){k[0], k[1], k[2], k[3]};
                    *(LAS u32x4*)(slot + 1024 + v16) = (u32x4){k[4], k[5], k[6], k[7]};
                    *(LAS u32x4*)(slot + 2048 + v16) = (u32x4){k[8], k[9], k[10], k[11]};
                    *(LAS u32x2*)(slot + 3072 + lane * 8) = (u32x2){k[12], k[13]};
                }
                asm volatile("s_waitcnt lgkmcnt(0)" ::: "memory");
                *mp = (u32x2){(unsigned)(i + 1) << 16 | (unsigned)q, meta};
            }
        }
        if (wid == 7 || DIAG_MODE(2)) { __syncthreads(); return; }
        unsigned char* peer = p.peer[(wid + b) % 7];
        const int rank = p.rank;
        int q = static_item(b);
        u32x4 r0, r1, r2, r3;
        load_item(q, r0, r1, r2, r3);
        unsigned tk = ticket_issue(me, wid);
        asm volatile("" :: "s"(peer), "s"(rank));
        asm volatile("s_waitcnt lgkmcnt(0)\n\ts_barrier\n\ts_waitcnt vmcnt(0)" : "+v"(tk), "+v"(r0), "+v"(r1), "+v"(r2), "+v"(r3) :: "memory");
        const unsigned call = __builtin_amdgcn_readfirstlane(tk) + 1u;
        unsigned char* dst = peer + (call & 1u) * XBUF + (size_t)rank * SHARD;
        unsigned char* fl = peer + WS_FLAG + (call & 1u) * FLAG_PAR + (size_t)rank * FLAG_SRC;
        unsigned char* fprev = nullptr;
        u32x2 fval = {META_RAW, call};
        unsigned ended = 0, meta = META_RAW;
        for (int i = 0; ended != 15u; ++i) {
            if (i > 0) {
                if (ended >> (i & 3) & 1u) continue;
                u32x2 m;
                for (;;) {
                    asm volatile("ds_read_b64 %0, %1\n\ts_waitcnt lgkmcnt(0)" : "=v"(m) : "v"(lds0 + SY_META + (i & 31) * 8) : "memory");
                    if (__builtin_amdgcn_readfirstlane(m.x) >> 16 == (unsigned)(i + 1)) break;
                    __builtin_amdgcn_s_sleep(1);
                }
                q = (int)(__builtin_amdgcn_readfirstlane(m.x) & 0xffffu);
                meta = __builtin_amdgcn_readfirstlane(m.y);
                if (q == 0xffff) { ended |= 1u << (i & 3); lds_st(lds0 + SY_PROG + wid * 4, (unsigned)i); continue; }
                asm volatile("ds_read_b128 %0, %4\n\tds_read_b128 %1, %4 offset:1024\n\tds_read_b128 %2, %4 offset:2048\n\t"
                             "ds_read_b128 %3, %4 offset:3072\n\ts_waitcnt lgkmcnt(0)"
                             : "=&v"(r0), "=&v"(r1), "=&v"(r2), "=&v"(r3) : "v"(lds0 + (unsigned)(i & 31) * ITEM + v16) : "memory");
                lds_st(lds0 + SY_PROG + wid * 4, (unsigned)i);
                while (!__all(lds_ld(lds0 + SY_PROG + (lane % 7) * 4) + LOCK >= (unsigned)i)) __builtin_amdgcn_s_sleep(1);
            }
            unsigned char* o = dst + (size_t)q * ITEM;
            const unsigned nl = meta == META_RAW ? 64u : 32u + (((meta >> 8) + 31u) >> 5) * 4u;
            wait_vm<3>();
            if (i == 0 && wid == 0) STAMP(6);
            asm volatile("global_store_dwordx4 %0, %1, %3 sc0 sc1\n\tglobal_store_dwordx4 %0, %2, %3 offset:1024 sc0 sc1\n\ts_nop 1"
                         :: "v"(v16), "v"(r0), "v"(r1), "s"(o) : "memory");
            if (i > 0) {
                wait_vm<2>();
                if (i == 1 && wid == 0) STAMP(7);
                if (lane == 0) asm volatile("global_store_dwordx2 %0, %1, %2 sc0 sc1" :: "v"(0u), "v"(fval), "s"(fprev) : "memory");
            }
            asm volatile("global_store_dwordx4 %0, %1, %2 offset:2048 sc0 sc1\n\ts_nop 1" :: "v"(v16), "v"(r2), "s"(o) : "memory");
            if ((unsigned)lane < nl) asm volatile("global_store_dwordx4 %0, %1, %2 offset:3072 sc0 sc1\n\ts_nop 1" :: "v"(v16), "v"(r3), "s"(o) : "memory");
            fprev = fl + item_flag(q);
            fval.x = meta;
        }
        wait_vm<0>();
        if (lane == 0) asm volatile("global_store_dwordx2 %0, %1, %2 sc0 sc1" :: "v"(0u), "v"(fval), "s"(fprev) : "memory");
        wait_vm<0>();
        STAMP(8 + wid);
        return;
    }

    const int tile = TILE_INDEX(b), ms = tile >> 3, nt = tile & 7;
    const bool local = ms >= 28;
    const int t = local ? ms - 28 : (ms & 3);
    const int src = local ? p.rank : ((p.rank + 1 + (ms >> 2)) & (NR - 1));
    const bool has_extra = t < 2;
    const int xrow = local ? 256 + 16 * t : 288 + 16 * ((ms >> 2) * 2 + t);

    if (wid >= 8) {
        const int h = wid - 8;
        const bool ex = has_extra && h < 2;
        __builtin_amdgcn_s_setprio(HPRIO);
        if (h == 0 && lane == 0) lds_st(lds0 + SY_BAD, 0u);
        const unsigned eoff = h ? aoff1 + 8 * KD : aoff0;
        const unsigned char* xe = p.x + (size_t)xrow * KD;
        unsigned eslot = lds0 + h * 1024, aslot = lds0 + h * ITEM;
        auto advance = [&](unsigned& s, unsigned first) { s = s == first + (NSLOT - 1) * SLOT_BYTES ? first : s + SLOT_BYTES; };
        auto extra = [&](int k) {
            if (ex && k < NSTEP && !DIAG_MODE(4)) glds<E_OFF>(xe + k * BK, eoff, eslot);
            advance(eslot, lds0 + h * 1024);
        };
        auto barrier = [&]() { asm volatile("s_waitcnt lgkmcnt(0)\n\ts_barrier" ::: "memory"); };
        const int hp = 4 * (WT + WB) + h * WH;
        unsigned wo[2 * WH + 4];
#pragma unroll
        for (int c = 0; c < 2 * WH; ++c) wo[c] = w_off(lane, c & 1, hp + (c >> 1));
        const unsigned char* wp = p.w + nt * BN + (size_t)hp * 8 * ND;
        unsigned wslot = lds0 + B_OFF + hp * 2048;
        auto issue_w = [&](int k) {
            extra(k);
            if (!DIAG_MODE(4)) {
                if constexpr (WH >= 2) w_pairs2(wp, wo[0], wo[1], wo[2], wo[3], wslot);
                if constexpr (WH == 3) w_pair(wp + 16 * ND, wo[4], wo[5], wslot + 4096);
                if constexpr (WH == 4) w_pairs2(wp + 16 * ND, wo[4], wo[5], wo[6], wo[7], wslot + 4096);
            }
            wp += (size_t)BK * ND;
            advance(wslot, lds0 + B_OFF + hp * 2048);
        };
        if (local || DIAG_MODE(4)) {
            const unsigned char* xl = p.x + (size_t)(t * BM + h * 32) * KD;
            auto issue = [&](int k) {
                issue_w(k);
                if (!DIAG_MODE(4))
                    static_for<0, 4>([&](auto cc) { constexpr int c = decltype(cc)::v; glds<c * 1024>(xl + k * BK, ((c & 1) ? aoff1 : aoff0) + c * 8 * KD, aslot); });
                advance(aslot, lds0 + h * ITEM);
            };
            issue(0); issue(1);
#pragma unroll 1
            for (int s = -1; s < NSTEP - 2; ++s) {
                if (s >= 0) issue(s + 2);
                if (ex) wait_vm<2 * WH + 5>(); else wait_vm<2 * WH + 4>();
                barrier();
            }
            wait_vm<0>();
            barrier();
            return;
        }
        auto remote = [&](auto exc) {
        constexpr int NW = 2 * WH + decltype(exc)::v;
        unsigned tk = ticket_issue(me, h);
        issue_w(0); issue_w(1);
        const int pi = (ms >> 2) * 16 + (t & 1) * 8 + nt;
        int pfs = ((t >> 1) != PF_LATE || !PREFETCH) ? NSTEP : pi % 7;
        const unsigned char* pfp = p.w + (size_t)((pfs + PF_AHEAD) * BK + (pi / 7) * 8 + h * 2) * ND;
        auto prefetch_row = [&](const unsigned char* row) {
            asm volatile("s_add_u32 m0, %2, %3\n\ts_nop 0\n\tglobal_load_lds_dwordx4 %0, %1\n\tglobal_load_lds_dwordx4 %0, %1 offset:1024"
                         :: "v"(v16), "s"(row), "s"(lds0), "i"(JUNK_OFF) : "memory", "scc");
        };
        if (PREFETCH) {
            const unsigned long long t0 = __builtin_amdgcn_s_memrealtime();
            while (__builtin_amdgcn_s_memrealtime() - t0 < (unsigned long long)PF_DELAY) __builtin_amdgcn_s_sleep(4);
            for (int n = ms * 32 + nt * 4 + h; n < (PF_AHEAD - 2) * BK; n += 28 * 32) prefetch_row(p.w + (size_t)(2 * BK + n) * ND);
        }
        asm volatile("s_waitcnt vmcnt(0)" : "+v"(tk) :: "memory");
        const unsigned call = __builtin_amdgcn_readfirstlane(tk) + 1u;
        const unsigned char* xr = me + (call & 1u) * XBUF + (size_t)src * SHARD + (size_t)(t * NSTEP * 4 + h) * ITEM;
        const unsigned char* fl = me + WS_FLAG + (call & 1u) * FLAG_PAR + (size_t)src * FLAG_SRC + (size_t)(t * 4 + h) * NSTEP * 8;
        unsigned* tmo = (unsigned*)(me + WS_TMO);
        const unsigned v8 = (unsigned)(lane & 31) * 8u;
        u32x2 fv = {0u, 0u};
        unsigned meta = 0, seen = 0;
        bool dead = false, pend = false, polled = false;
        int behind = 0;
        int next = 0;
        auto look = [&]() {
            seen = (unsigned)__ballot(fv.y >= call);
            if (DIAG_MODE(1)) { seen = ~0u; fv.x = META_RAW; }
        };
        const unsigned vt = v16 >> 1, ve = v16 >> 3;
        auto request = [&](bool second) {
            LTR(lt0, next);
            if (next >= NSTEP - 3) STAMP(20 + (next - (NSTEP - 3)) * 4 + h);
            meta = __builtin_amdgcn_readlane(fv.x, next);
            const unsigned char* a = xr + (size_t)next * 4 * ITEM;
            unsigned place = aslot;
            if (second) advance(place, lds0 + h * ITEM);
            if (meta == META_RAW) raw_loads(a, v16, ve, place);
            else { if (next & 1) item_loads<140>(a, v16, vt, ve); else item_loads<120>(a, v16, vt, ve); }
            ++next;
        };
        auto poll_async = [&]() {
            asm volatile("global_load_dwordx2 v[158:159], %0, %1 sc0 sc1" :: "v"(v8), "s"(fl) : "memory", "v158", "v159");
            polled = true;
        };
        auto take_poll = [&]() {
            asm volatile("v_mov_b64 %0, v[158:159]" : "=v"(fv) :: "memory");
            look(); polled = false;
        };
        auto ahead = [&]() {
            if (next < NSTEP) {
                if (seen >> next & 1u) { request(true); pend = true; }
                poll_async();
            }
        };
        auto flip = [&](unsigned pos, bool on) {
            if (on) asm volatile("ds_xor_b32 %0, %1" :: "v"(aslot + (pos & 0xffcu)), "v"(0x40u << ((pos & 3u) * 8u)) : "memory");
        };
        auto expand_from = [&](auto rc, unsigned m, int k) {
            constexpr int R = decltype(rc)::v;
            const unsigned oa = aslot + v16;
            if (m != META_RAW) {
                LAS unsigned char* o = (LAS unsigned char*)(size_t)oa;
                const unsigned b4 = (m & 0xffu) * 0x01010101u, cnt = m >> 8;
                unsigned d[16];
                unpack8<R>(d, b4, [](int) {});
                *(LAS u32x4*)o = (u32x4){d[0], d[1], d[2], d[3]}; *(LAS u32x4*)(o + 1024) = (u32x4){d[4], d[5], d[6], d[7]};
                __builtin_amdgcn_sched_barrier(0);
                unpack8<R + 7>(d + 8, b4, [](int) {});
                *(LAS u32x4*)(o + 2048) = (u32x4){d[8], d[9], d[10], d[11]}; *(LAS u32x4*)(o + 3072) = (u32x4){d[12], d[13], d[14], d[15]};
                flip(reg_get<R + 16>(), (unsigned)lane < cnt);
                for (unsigned c0 = 64; c0 < cnt && c0 < (unsigned)EXC_MAX; c0 += 64) {
                    unsigned e;
                    const unsigned eo = (c0 + lane) * 2;
                    const unsigned char* ea = xr + (size_t)k * 4 * ITEM;
                    asm volatile("global_load_ushort %0, %1, %2 offset:3584 sc0 sc1\n\ts_waitcnt vmcnt(0)" : "=&v"(e) : "v"(eo), "s"(ea) : "memory");
                    flip(e, c0 + lane < cnt);
                }
            }
        };
        auto expand = [&](unsigned m, int k) {
            if (k & 1) expand_from(IC<140>{}, m, k); else expand_from(IC<120>{}, m, k);
            advance(aslot, lds0 + h * ITEM);
        };
        auto settle = [&]() {
            if (behind == 1) wait_vm<NW>(); else if (behind == 3) wait_vm<NW + 4>(); else if (behind == 2) wait_vm<4>(); else wait_vm<0>();
        };
        auto finish = [&](int s) {
            if (s == pfs) {
                prefetch_row(pfp); prefetch_row(pfp + ND);
                behind |= 2;
                pfs = pfs + 7 + PF_AHEAD < NSTEP ? pfs + 7 : NSTEP;
                pfp += (size_t)7 * BK * ND;
            }
            LTR(lt1, s + 1);
            barrier();
            LTR(lt2, s + 1);
            if (h == 0 && s >= NSTEP - 4) STAMP(32 + s - (NSTEP - 4));
        };
        auto fast = [&](auto rc, auto rn, int s) {
            constexpr int RC = decltype(rc)::v, RN = decltype(rn)::v;
            if (!(seen >> next & 1u)) {
                settle(); take_poll(); polled = true;
                if (!(seen >> next & 1u)) return false;
            }
            const unsigned mn = __builtin_amdgcn_readlane(fv.x, next);
            if (mn >= 0x4100u) return false;
            const unsigned b4 = (meta & 0xffu) * 0x01010101u, cnt = meta >> 8, oa = aslot + v16;
            LTR(lt0, next);
            if (next >= NSTEP - 3) STAMP(20 + (next - (NSTEP - 3)) * 4 + h);
            const unsigned char* a = xr + (size_t)next * 4 * ITEM;
            const bool pf = behind & 2;
            if (behind & 1) { if (pf) wait_vm<NW + 8>(); else wait_vm<NW + 4>(); } else { if (pf) wait_vm<8>(); else wait_vm<4>(); }
            unsigned d[16];
            LAS unsigned char* o = (LAS unsigned char*)(size_t)oa;
            load16<RN, 0>(a, v16);
            unpack8<RC>(d, b4, [&](int i) { if (i == 0) load16<RN + 4, 1024>(a, v16); else load16<RN + 8, 2048>(a, v16); });
            *(LAS u32x4*)o = (u32x4){d[0], d[1], d[2], d[3]}; *(LAS u32x4*)(o + 1024) = (u32x4){d[4], d[5], d[6], d[7]};
            if (behind & 1) { if (pf) wait_vm<NW + 7>(); else wait_vm<NW + 3>(); } else { if (pf) wait_vm<7>(); else wait_vm<3>(); }
            take_poll();
            load16<RN + 12, 3072>(a, vt);
            unpack8<RC + 7>(d + 8, b4, [&](int i) { if (i == 0) load2<RN + 16>(a, ve); else poll_async(); });
            *(LAS u32x4*)(o + 2048) = (u32x4){d[8], d[9], d[10], d[11]}; *(LAS u32x4*)(o + 3072) = (u32x4){d[12], d[13], d[14], d[15]};
            flip(reg_get<RC + 16>(), (unsigned)lane < cnt);
            advance(aslot, lds0 + h * ITEM);
            meta = mn; ++next;
            if (NW) {
                asm volatile("s_waitcnt lgkmcnt(0)" ::: "memory");
                issue_w(s + 2);
                if (pf) wait_vm<NW + 10>(); else wait_vm<NW + 6>();
            }
            behind = NW ? 1 : 0;
            finish(s);
            return true;
        };
        auto fast_raw = [&](int s) {
            if (!(seen >> next & 1u)) {
                settle(); take_poll(); polled = true;
                if (!(seen >> next & 1u)) return false;
            }
            const bool pf = behind & 2;
            request(true);
            advance(aslot, lds0 + h * ITEM);
            if (behind & 1) { if (pf) wait_vm<NW + 9>(); else wait_vm<NW + 5>(); } else { if (pf) wait_vm<9>(); else wait_vm<5>(); }
            take_poll(); poll_async();
            if (NW) { issue_w(s + 2); if (pf) wait_vm<NW + 10>(); else wait_vm<NW + 6>(); }
            behind = NW ? 1 : 0;
            finish(s);
            return true;
        };
#pragma unroll 1
        for (int s = -1; s < NSTEP - 1; ++s) {
#pragma unroll 1
            while (FASTP && pend && meta == META_RAW && next < NSTEP && fast_raw(s)) ++s;
            if (FASTP && pend && meta < 0x4100u) {
                if (!(s & 1)) {
                    if (next < NSTEP && fast(IC<140>{}, IC<120>{}, s)) ++s; else goto general;
                }
#pragma unroll 1
                while (next < NSTEP && fast(IC<120>{}, IC<140>{}, s)) {
                    ++s;
                    if (next < NSTEP && fast(IC<140>{}, IC<120>{}, s)) ++s; else break;
                }
            }
        general:
            settle();
            bool wdue = s >= 0 && s + 2 < NSTEP;
            behind = 0;
            if (polled) take_poll();
            if (pend) {
                const unsigned m = meta;
                pend = false;
                    ahead();
                expand(m, s + 1);
                if (!wdue) { if (s > 0) wait_vm<0>(); }
                else {
                    asm volatile("s_waitcnt lgkmcnt(0)" ::: "memory");
                    issue_w(s + 2); wdue = false; behind = 1;
                    if (pend) wait_vm<NW + 6>(); else wait_vm<NW + 1>();
                }
            } else if (!dead) {
                    if (!(seen >> next & 1u)) {
                    if (wdue) { issue_w(s + 2); wdue = false; }
                    const unsigned long long t0 = __builtin_amdgcn_s_memrealtime();
                    for (unsigned polls = 1;; ++polls) {
                        asm volatile("global_load_dwordx2 %0, %1, %2 sc0 sc1\n\ts_waitcnt vmcnt(0)" : "=&v"(fv) : "v"(v8), "s"(fl) : "memory");
                        look();
                        if (seen >> next & 1u) break;
                        if ((polls & 63u) == 0u
                            && (__builtin_amdgcn_readfirstlane(__hip_atomic_load(tmo, __ATOMIC_RELAXED, __HIP_MEMORY_SCOPE_AGENT)) == call
                                || __builtin_amdgcn_s_memrealtime() - t0 > DEADLINE)) {
                            if (lane == 0) { __hip_atomic_store(tmo, call, __ATOMIC_RELAXED, __HIP_MEMORY_SCOPE_AGENT); lds_st(lds0 + SY_BAD, 1u); }
                            dead = true;
                            break;
                        }
                        __builtin_amdgcn_s_sleep(POLL_SLEEP);
                    }
                    if (next == NSTEP - 1) STAMP(36 + h);
                }
                if (!dead) {
                    request(false); poll_async();
                    wait_vm<0>();
                    const unsigned m = meta;
                    take_poll();
                    ahead();
                    expand(m, s + 1);
                }
            }
            if (wdue) {
                asm volatile("s_waitcnt lgkmcnt(0)" ::: "memory");
                issue_w(s + 2); behind = 1;
            }
            finish(s);
        }
        wait_vm<0>();
        };
        if (ex) remote(IC<1>{}); else remote(IC<0>{});
        return;
    }

    const int wr = wid >> 2, wc = wid & 3;
    unsigned sx = __builtin_amdgcn_readfirstlane(__float_as_uint(p.sx[0])), sw = __builtin_amdgcn_readfirstlane(__float_as_uint(p.sw[0]));
    int ex = 2 - t, top = 1 - wr;
    f32x4 acc[4][4], accx[2];
#pragma unroll
    for (int m = 0; m < 4; ++m)
#pragma unroll
        for (int n = 0; n < 4; ++n) acc[m][n] = (f32x4){0.f, 0.f, 0.f, 0.f};
    accx[0] = accx[1] = (f32x4){0.f, 0.f, 0.f, 0.f};
    const float* ob = p.out + (size_t)(src * MS + t * BM + wr * 64) * NS + nt * BN + wc * 64;
    const float* ox = p.out + (size_t)(p.rank * MS + xrow) * NS + nt * BN + wc * 64 + wr * 32;
    asm volatile("" : "+s"(ob), "+s"(ox), "+s"(sx), "+s"(sw), "+s"(ex), "+s"(top));
    const int cp = wr ? 4 * WT + wc * WB : wc * WT;
    const unsigned char* wp = p.w + nt * BN + (size_t)cp * 8 * ND;
    unsigned wslot = lds0 + B_OFF + cp * 2048;
    unsigned wo0 = w_off(lane, 0, cp), wo1 = w_off(lane, 1, cp);
    auto issue_w = [&](auto npc) {
        if (!DIAG_MODE(4)) {
            if (decltype(npc)::v == 1) w_pair(wp, wo0, wo1, wslot);
            if (decltype(npc)::v == 2) w_pairs2(wp, wo0, wo1, wo0, wo1, wslot);
        }
        wp += (size_t)BK * ND;
        wslot = wslot >= lds0 + B_OFF + (NSLOT - 1) * SLOT_BYTES ? wslot - (NSLOT - 1) * SLOT_BYTES : wslot + SLOT_BYTES;
    };
    if (wr) { issue_w(IC<WB>{}); issue_w(IC<WB>{}); wait_vm<2 * WB>(); } else { issue_w(IC<WT>{}); issue_w(IC<WT>{}); wait_vm<2 * WT>(); }
    asm volatile("" : "+v"(wo0), "+v"(wo1));
    unsigned aa[2], ea[2], bo[4];
    {
        const int fr = lane & 15, fq = lane >> 4, q = fr >> 1, f = q | ((fq & 1) << 3);
        aa[0] = lds0 + (unsigned)((wr * 64 + fr) * 128 + ((fq ^ ((fr >> 1) & 7)) << 4));
        aa[1] = aa[0] ^ 64u;
        ea[0] = aa[0] + (unsigned)(E_OFF - wr * 8192);
        ea[1] = ea[0] ^ 64u;
#pragma unroll
        for (int n = 0; n < 4; ++n) bo[n] = lds0 + (unsigned)((16 * fq + q) * 256 + (((wc * 4 + (n ^ (2 * wr))) ^ f) << 4) + (fr & 1) * 8);
    }
    asm volatile(".p2align 8\n\t.rept %0\n\ts_nop 0\n\t.endr" :: "i"(LOOP_PAD) : "memory");
    asm volatile("s_barrier" ::: "memory");
    {
        int slot = 0;
#pragma unroll 1
        for (int s = 0; s < NSTEP; ++s) {
            v8i af[4], bf[4], ef;
#pragma unroll
            for (int n = 0; n < 4; ++n) {
                LAS unsigned char* q = (LAS unsigned char*)(size_t)bo[n] + B_OFF;
                const v2i t0 = __builtin_amdgcn_ds_read_tr8_b64_v2i32((LAS v2i*)q);
                const v2i t1 = __builtin_amdgcn_ds_read_tr8_b64_v2i32((LAS v2i*)(q + 8 * 256));
                const v2i t2 = __builtin_amdgcn_ds_read_tr8_b64_v2i32((LAS v2i*)(q + 64 * 256));
                const v2i t3 = __builtin_amdgcn_ds_read_tr8_b64_v2i32((LAS v2i*)(q + 72 * 256));
                bf[n] = (v8i){t0.x, t0.y, t1.x, t1.y, t2.x, t2.y, t3.x, t3.y};
            }
#pragma unroll
            for (int m = 0; m < 4; ++m) {
                const v4i lo = *(LAS v4i*)((LAS unsigned char*)(size_t)aa[0] + m * 2048), hi = *(LAS v4i*)((LAS unsigned char*)(size_t)aa[1] + m * 2048);
                af[m] = (v8i){lo.x, lo.y, lo.z, lo.w, hi.x, hi.y, hi.z, hi.w};
            }
            asm volatile("" : "+s"(ex));
            if (ex > 0) {
                const v4i lo = *(LAS v4i*)(size_t)ea[0], hi = *(LAS v4i*)(size_t)ea[1];
                ef = (v8i){lo.x, lo.y, lo.z, lo.w, hi.x, hi.y, hi.z, hi.w};
            }
            int wdue = s + 2 < NSTEP ? 1 + top : 0;
            asm volatile("" : "+s"(wdue));
            if (WB && wdue == 1) issue_w(IC<WB>{});
            asm volatile("" ::: "memory");
            __builtin_amdgcn_s_waitcnt(0xC07F);
            asm volatile("" : "+s"(wdue));
            if (WT && wdue == 2) issue_w(IC<WT>{});
            slot = (slot == NSLOT - 1) ? 0 : slot + 1;
            const int delta = slot ? SLOT_BYTES : -(NSLOT - 1) * SLOT_BYTES;
            int bar = s < NSTEP - 1 ? 1 + top : 0;
            asm volatile("" : "+s"(bar));
            if (bar == 1) { if (WB) { if (wdue > 0) wait_vm<2 * WB>(); else wait_vm<0>(); } asm volatile("s_barrier" ::: "memory"); }
            __builtin_amdgcn_sched_barrier(0);
            asm volatile("s_setprio 1\n\t.p2align 3");
            __builtin_amdgcn_sched_barrier(0);
#pragma unroll
            for (int m = 0; m < 4; ++m) {
#pragma unroll
                for (int n = 0; n < 4; ++n) {
                    acc[m][n] = __builtin_amdgcn_mfma_scale_f32_16x16x128_f8f6f4(bf[n], af[m], acc[m][n], 0, 0, 0, 0, 0, 0);
                    if (m == 0 && n == 0) { __builtin_amdgcn_sched_barrier(0); __builtin_amdgcn_s_setprio(2); __builtin_amdgcn_sched_barrier(0); }
                    if (m == 3 && n == 1) {
                        __builtin_amdgcn_sched_barrier(0);
                        asm volatile("" : "+s"(bar));
                        if (bar == 2) { if (WT) { if (wdue > 0) wait_vm<2 * WT>(); else wait_vm<0>(); } asm volatile("s_barrier" ::: "memory"); }
                        __builtin_amdgcn_sched_barrier(0);
                    }
                }
                __builtin_amdgcn_sched_barrier(0);
                if (m == 0) asm volatile("v_add_u32 %0, %4, %0\n\tv_add_u32 %1, %4, %1\n\tv_add_u32 %2, %4, %2\n\tv_add_u32 %3, %4, %3"
                                         : "+v"(bo[0]), "+v"(bo[1]), "+v"(bo[2]), "+v"(bo[3]) : "s"(delta));
                if (m == 1) asm volatile("v_add_u32 %0, %4, %0\n\tv_add_u32 %1, %4, %1\n\tv_add_u32 %2, %4, %2\n\tv_add_u32 %3, %4, %3"
                                         : "+v"(aa[0]), "+v"(aa[1]), "+v"(ea[0]), "+v"(ea[1]) : "s"(delta));
                __builtin_amdgcn_sched_barrier(0);
            }
            asm volatile("" : "+s"(ex));
            if (ex > 0) {
                accx[0] = __builtin_amdgcn_mfma_scale_f32_16x16x128_f8f6f4(bf[0], ef, accx[0], 0, 0, 0, 0, 0, 0);
                accx[1] = __builtin_amdgcn_mfma_scale_f32_16x16x128_f8f6f4(bf[1], ef, accx[1], 0, 0, 0, 0, 0, 0);
            }
            __builtin_amdgcn_s_setprio(0);
            __builtin_amdgcn_sched_barrier(0);
        }
    }
    if (wid == 0) { STAMP(1); STAMPC(5); }
    STAMP(48 + wid);
    float scale = __uint_as_float(sx) * __uint_as_float(sw);
    if (__builtin_amdgcn_readfirstlane(lds_ld(lds0 + SY_BAD)) != 0u) scale = __builtin_nanf("");
    int ln;
    asm volatile("v_mbcnt_lo_u32_b32 %0, -1, 0\n\tv_mbcnt_hi_u32_b32 %0, -1, %0" : "=v"(ln));
    const int fr = ln & 15, fq = ln >> 4;
    const float zs = scale * -1.44269504088896340736f, zmax = 60.f * 1.44269504088896340736f;
    LAS unsigned char* tb = (LAS unsigned char*)smem + ((NSTEP - 3 + (wc >> 1)) % NSLOT) * SLOT_BYTES + wr * 8192 + (wc & 1) * 4096;
    f32x2 e[8];
    f32x4 tv[4];
    auto expo = [&](auto nqc, const f32x4* c) {
        constexpr int NQ = decltype(nqc)::v;
#pragma unroll
        for (int h = 0; h < 2 * NQ; ++h) {
            f32x2 z = (f32x2){c[h >> 1][(h & 1) * 2], c[h >> 1][(h & 1) * 2 + 1]} * zs;
            asm("" : "+v"(z));
            e[h].x = __builtin_amdgcn_exp2f(__builtin_amdgcn_fmed3f(z.x, -zmax, zmax));
            e[h].y = __builtin_amdgcn_exp2f(__builtin_amdgcn_fmed3f(z.y, -zmax, zmax));
        }
        __builtin_amdgcn_sched_barrier(0);
    };
    auto stage = [&](auto nqc, const f32x4* c) {
        constexpr int NQ = decltype(nqc)::v;
#pragma unroll
        for (int q = 0; q < NQ; ++q) {
            f32x2 d0 = e[2 * q] + 1.0f, d1 = e[2 * q + 1] + 1.0f;
            asm("" : "+v"(d0)); asm("" : "+v"(d1));
            const f32x4 r = {__builtin_amdgcn_rcpf(d0.x), __builtin_amdgcn_rcpf(d0.y), __builtin_amdgcn_rcpf(d1.x), __builtin_amdgcn_rcpf(d1.y)};
            const int cb = NQ == 4 ? (q ^ (2 * wr)) : q;
            *(LAS f32x4*)(tb + fr * 256 + (((cb * 4 + fq) ^ (fr & 7)) << 4)) = (c[q] * scale) * r;
        }
        constexpr int LPR = NQ * 4, RPI = 64 / LPR;
#pragma unroll
        for (int i = 0; i < NQ; ++i) {
            const int r = i * RPI + ln / LPR;
            tv[i] = *(LAS f32x4*)(tb + r * 256 + (((ln % LPR) ^ (r & 7)) << 4));
        }
        __builtin_amdgcn_sched_barrier(0);
    };
    auto put = [&](auto nqc, const float* rowp) {
        constexpr int NQ = decltype(nqc)::v, LPR = NQ * 4, RPI = 64 / LPR;
        const unsigned ovoff = (unsigned)((ln / LPR) * NS * 4 + (ln % LPR) * 16);
#pragma unroll
        for (int i = 0; i < NQ; ++i)
            asm volatile("global_store_dwordx4 %0, %1, %2 sc1\n\ts_nop 1" :: "v"(ovoff), "v"(tv[i]), "s"(rowp + (size_t)i * RPI * NS) : "memory");
        __builtin_amdgcn_sched_barrier(0);
    };
    expo(IC<4>{}, acc[0]);
#pragma unroll
    for (int m = 0; m < 4; ++m) {
        stage(IC<4>{}, acc[m]);
        if (m < 3) expo(IC<4>{}, acc[m + 1]);
        else if (has_extra) expo(IC<2>{}, accx);
        put(IC<4>{}, ob + (size_t)m * 16 * NS);
    }
    if (has_extra) { stage(IC<2>{}, accx); put(IC<2>{}, ox); }
}

}

extern "C" void kernel_launch(int rank, int nranks, ncclComm_t comm,
                              hipStream_t stream,
                              void* const* d_in, const int* in_sizes, int n_in,
                              void* d_out, int out_size,
                              const int* in_dtypes, int out_dtype,
                              void* const* d_out_all,
                              void* const* d_ws_all, size_t ws_bytes) {
    (void)comm; (void)in_dtypes; (void)out_dtype; (void)d_out_all;
    if (nranks != NR || (unsigned)rank >= (unsigned)NR || n_in != 4 || ws_bytes < WS_END || in_sizes[0] != MS * KD
        || in_sizes[1] != KD * ND || out_size != NR * MS * NS) {
        fprintf(stderr, "kernel_launch: unexpected problem shape\n");
        return;
    }
    HIP_CHECK(hipFuncSetAttribute((const void*)fused_ag_gemm, hipFuncAttributeMaxDynamicSharedMemorySize, LDS_BYTES));
    Params p;
    p.x = (const unsigned char*)d_in[0];
    p.w = (const unsigned char*)d_in[1] + (size_t)rank * NS;
    p.sx = (const float*)d_in[2];
    p.sw = (const float*)d_in[3];
    p.out = (float*)d_out;
    p.me = (unsigned char*)d_ws_all[rank];
    for (int i = 0; i < NR - 1; ++i) p.peer[i] = (unsigned char*)d_ws_all[(rank + 1 + i) % NR];
    p.rank = rank;
    p.mode = 0;
    hipLaunchKernelGGL(fused_ag_gemm, dim3(NWG), dim3(NTHREADS), LDS_BYTES, stream, p);
    HIP_CHECK(hipGetLastError());
}
```

```cpp
#include <hip/hip_runtime.h>
#include <rccl/rccl.h>
#include <cstdio>

#define HIP_CHECK(expr) do { hipError_t _e = (expr); if (_e != hipSuccess) { \
    fprintf(stderr, "HIP error %s at %s:%d\n", hipGetErrorString(_e), __FILE__, __LINE__); \
    return; } } while (0)

namespace {

#define LAS __attribute__((address_space(3)))
typedef int v8i __attribute__((ext_vector_type(8)));
typedef int v4i __attribute__((ext_vector_type(4)));
typedef int v2i __attribute__((ext_vector_type(2)));
typedef float f32x2 __attribute__((ext_vector_type(2)));
typedef float f32x4 __attribute__((ext_vector_type(4)));
typedef unsigned u32x2 __attribute__((ext_vector_type(2)));
typedef unsigned u32x4 __attribute__((ext_vector_type(4)));

constexpr int NR = 8;
constexpr int KD = 4096, ND = 16384;
constexpr int MS = 512, NS = 2048;
constexpr int BM = 128, BN = 256, BK = 128;
constexpr int NSTEP = KD / BK;
constexpr int MT = MS / BM;
constexpr int NWG = 256, NTHREADS = 768;
constexpr int NSLOT = 3;
constexpr int A_BYTES = BM * BK, E_BYTES = 16 * BK, B_BYTES = BK * BN;
constexpr int E_OFF = A_BYTES, B_OFF = A_BYTES + E_BYTES, SLOT_BYTES = B_OFF + B_BYTES;
constexpr int SYNC_OFF = NSLOT * SLOT_BYTES;
constexpr int SY_BAD = SYNC_OFF;
constexpr int SY_PROG = SYNC_OFF, SY_META = SYNC_OFF + 64;
constexpr int JUNK_OFF = SYNC_OFF + 512;
constexpr int LDS_BYTES = JUNK_OFF + 2048;
static_assert(LDS_BYTES <= 163840, "LDS");

constexpr int NPUSH = 16;
constexpr int ITEM = 4096;
constexpr int NITEM = MT * NSTEP * 4;
constexpr int NSTATIC = 4 * NPUSH;
constexpr int NDRAW = 512;
static_assert(NITEM - NSTATIC + 4 * NPUSH == NDRAW && 32 * ITEM <= SYNC_OFF, "draw count; staging ring");
constexpr int EXC_OFF = 3584, EXC_MAX = 256;
constexpr unsigned META_RAW = 0x80000000u, FAST_META = 65u << 8;

constexpr size_t SHARD = (size_t)MS * KD;
constexpr size_t XBUF = (size_t)NR * SHARD;
constexpr size_t FLAG_SRC = (size_t)NITEM * 8, FLAG_PAR = NR * FLAG_SRC;
constexpr size_t WS_FLAG = 2 * XBUF;
constexpr size_t WS_CNT = WS_FLAG + 2 * FLAG_PAR;
constexpr size_t WS_TMO = WS_CNT + (size_t)NWG * 8 * 128;
constexpr size_t WS_DRAW = WS_TMO + 1024;
constexpr size_t WS_END = WS_TMO + 4096;
constexpr unsigned long long DEADLINE = 50000000ull;

constexpr unsigned BASE0 = 4;
constexpr int STAGE_AHEAD = 4;
static_assert(STAGE_AHEAD >= 4, "a packing wave may only wait for its own previous position");
constexpr unsigned LOCK = 1;
constexpr int PF_AHEAD = 7, PF_DELAY = 100;
constexpr int FIN[MT] = {850, 860, 1000, 1000};
constexpr int RAW_LAST[MT] = {4, 4, 9, 9};

struct Params {
    const unsigned char* x;
    unsigned char* me;
    const unsigned char* w;
    float* out;
    unsigned char* peer[NR - 1];
    int rank;
    int pad;
    const float* sx; const float* sw;
};

template <int N> __device__ __forceinline__ void wait_vm() { asm volatile("s_waitcnt vmcnt(%0)" :: "i"(N) : "memory"); }
template <int I> struct IC { static constexpr int v = I; };
template <int I, int E, class F> __device__ __forceinline__ void static_for(F&& f) { if constexpr (I < E) { f(IC<I>{}); static_for<I + 1, E>(f); } }
__device__ __forceinline__ unsigned lds_ld(unsigned addr) {
    unsigned v; asm volatile("ds_read_b32 %0, %1\n\ts_waitcnt lgkmcnt(0)" : "=v"(v) : "v"(addr) : "memory"); return v;
}
__device__ __forceinline__ void lds_st(unsigned addr, unsigned v) { asm volatile("ds_write_b32 %0, %1" :: "v"(addr), "v"(v) : "memory"); }
template <int OFF> __device__ __forceinline__ void glds(const unsigned char* base, unsigned voff, unsigned dst) {
    asm volatile("s_add_u32 m0, %2, %3\n\ts_nop 0\n\tglobal_load_lds_dwordx4 %0, %1" :: "v"(voff), "s"(base), "s"(dst), "i"(OFF) : "memory", "scc");
}
__device__ __forceinline__ unsigned w_off(int lane, int c, int g) {
    return (unsigned)(c * 4 + (lane >> 4)) * ND + (unsigned)(((lane & 15) ^ ((c * 4 + (lane >> 4)) | ((g & 1) << 3))) << 4) - c * 1024u;
}
__device__ __forceinline__ void w_loads(const unsigned char* wp, unsigned o0, unsigned o1, unsigned dst) {
    asm volatile("s_mov_b32 m0, %4\n\ts_nop 0\n\tglobal_load_lds_dwordx4 %0, %2\n\tglobal_load_lds_dwordx4 %1, %2 offset:1024\n\t"
                 "global_load_lds_dwordx4 %0, %3 offset:2048\n\tglobal_load_lds_dwordx4 %1, %3 offset:3072"
                 :: "v"(o0), "v"(o1), "s"(wp), "s"(wp + 8 * ND - 2048), "s"(dst) : "memory");
}
__device__ __forceinline__ unsigned ticket_issue(unsigned char* me, int role) {
    unsigned t = 0;
    unsigned char* a = me + WS_CNT + ((size_t)blockIdx.x * 8 + role) * 128;
    if ((threadIdx.x & 63) == 0) asm volatile("global_atomic_add %0, %1, %2, off sc0" : "+v"(t) : "v"(a), "v"(1u) : "memory");
    return t;
}
__device__ __forceinline__ unsigned bfi(unsigned mask, unsigned a, unsigned b) { return (a & mask) | (b & ~mask); }

__device__ __forceinline__ unsigned rebase(unsigned d, unsigned b4) { return bfi(0x7f7f7f7fu, (d | 0x80808080u) - b4, d); }
__device__ __forceinline__ void pack8(const unsigned* d, unsigned* q) {
#pragma unroll
    for (int i = 0; i < 7; ++i) q[i] = bfi(0x40404040u, i < 6 ? d[7] << (1 + i) : d[7] >> 1, d[i]);
}
#define XREGS "v120", "v121", "v122", "v123", "v124", "v125", "v126", "v127", "v128", "v129", "v130", "v131", "v132", "v133", "v134", "v135", "v136", \
              "v140", "v141", "v142", "v143", "v144", "v145", "v146", "v147", "v148", "v149", "v150", "v151", "v152", "v153", "v154", "v155", "v156"
template <int R> __device__ __forceinline__ void item_loads(const unsigned char* a, unsigned v16, unsigned tail, unsigned ve) {
    asm volatile("global_load_dwordx4 v[%c4:%c5], %0, %3 sc0 sc1\n\tglobal_load_dwordx4 v[%c6:%c7], %0, %3 offset:1024 sc0 sc1\n\t"
                 "global_load_dwordx4 v[%c8:%c9], %0, %3 offset:2048 sc0 sc1\n\tglobal_load_dwordx4 v[%c10:%c11], %1, %3 offset:3072 sc0 sc1\n\t"
                 "global_load_ushort v%c12, %2, %3 offset:3584 sc0 sc1"
                 :: "v"(v16), "v"(tail), "v"(ve), "s"(a), "i"(R), "i"(R + 3), "i"(R + 4), "i"(R + 7), "i"(R + 8), "i"(R + 11),
                    "i"(R + 12), "i"(R + 15), "i"(R + 16) : "memory", XREGS);
}
template <int R, int OFF> __device__ __forceinline__ void load16(const unsigned char* a, unsigned v) {
    asm volatile("global_load_dwordx4 v[%c2:%c3], %0, %1 offset:%4 sc0 sc1" :: "v"(v), "s"(a), "i"(R), "i"(R + 3), "i"(OFF) : "memory", XREGS);
}
template <int R> __device__ __forceinline__ void load2(const unsigned char* a, unsigned v) {
    asm volatile("global_load_ushort v%c2, %0, %1 offset:3584 sc0 sc1" :: "v"(v), "s"(a), "i"(R) : "memory", XREGS);
}
__device__ __forceinline__ void raw_loads(const unsigned char* a, unsigned v16, unsigned ve, unsigned dst) {
    asm volatile("s_mov_b32 m0, %3\n\ts_nop 0\n\tglobal_load_lds_dwordx4 %0, %2 sc0 sc1\n\tglobal_load_lds_dwordx4 %0, %2 offset:1024 sc0 sc1\n\t"
                 "global_load_lds_dwordx4 %0, %2 offset:2048 sc0 sc1\n\tglobal_load_lds_dwordx4 %0, %2 offset:3072 sc0 sc1\n\t"
                 "global_load_ushort v137, %1, %2 sc0 sc1" :: "v"(v16), "v"(ve), "s"(a), "s"(dst) : "memory", "v137");
}
template <int R> __device__ __forceinline__ unsigned reg_get() { unsigned v; asm volatile("v_mov_b32 %0, v%c1" : "=v"(v) : "i"(R)); return v; }
template <int R> __device__ __forceinline__ unsigned carrier() { unsigned c; asm volatile("v_and_b32 %0, 0x40404040, v%c1" : "=v"(c) : "i"(R)); return c; }
template <int R> __device__ __forceinline__ unsigned restore(unsigned c, unsigned b4) {
    unsigned d; asm volatile("v_xad_u32 %0, v%c1, %2, %3" : "=v"(d) : "i"(R), "v"(c), "s"(b4)); return d;
}
template <int R, class H> __device__ __forceinline__ void unpack8(unsigned* d, unsigned b4, H&& hook) {
    const unsigned c[7] = {carrier<R>(), carrier<R + 1>(), carrier<R + 2>(), carrier<R + 3>(), carrier<R + 4>(), carrier<R + 5>(), carrier<R + 6>()};
    hook(0);
    d[0] = restore<R>(c[0], b4); d[1] = restore<R + 1>(c[1], b4); d[2] = restore<R + 2>(c[2], b4); d[3] = restore<R + 3>(c[3], b4);
    d[4] = restore<R + 4>(c[4], b4); d[5] = restore<R + 5>(c[5], b4); d[6] = restore<R + 6>(c[6], b4);
    hook(1);
    const unsigned t0 = (c[0] << 1) | c[1], t1 = (c[2] << 1) | c[3], t2 = (c[4] << 1) | c[5];
    d[7] = ((t0 >> 2) + (t1 >> 4) + (t2 >> 6)) + ((c[6] << 1) + b4);
}
__device__ __forceinline__ unsigned tops8(const unsigned* d) {
    unsigned m = 0;
#pragma unroll
    for (int i = 0; i < 8; ++i) m |= (i < 7 ? d[i] >> (6 - i) : d[i] << 1) & (0x01010101u << i);
    return m;
}

#define TILE_INDEX(b) ((((b) - NPUSH) & 7) * 30 + (((b) - NPUSH) >> 3))

struct Order { unsigned q[NITEM]; };
constexpr Order make_order() {
    Order o{}; int key[NITEM] = {}; int n = 0;
    for (int k = 0; k < NSTEP; ++k)
        for (int t = 0; t < MT; ++t)
            for (int j = 0; j < 4; ++j) {
                const int due = k < 4 ? 0 : FIN[t] * (k + 1);
                int i = n++;
                while (i > 0 && key[i - 1] > due) { key[i] = key[i - 1]; o.q[i] = o.q[i - 1]; --i; }
                key[i] = due; o.q[i] = (unsigned)((t * NSTEP + k) * 4 + j);
            }
    return o;
}
__constant__ const Order k_order = make_order();
__device__ __forceinline__ int static_item(int id) { return (((id >> 2) & 3) * NSTEP + (id >> 4)) * 4 + (id & 3); }
__device__ __forceinline__ size_t item_src(int q) { return (size_t)((q >> 7) * BM + (q & 3) * 32) * KD + ((q >> 2) & 31) * BK; }
__device__ __forceinline__ size_t item_flag(int q) { return (size_t)((((q >> 7) * 4 + (q & 3)) * NSTEP) + ((q >> 2) & 31)) * 8; }

__global__ __launch_bounds__(NTHREADS) void fused_ag_gemm(Params p) {
    extern __shared__ __attribute__((aligned(16))) unsigned char smem[];
    const int tid = threadIdx.x, lane = tid & 63, wid = __builtin_amdgcn_readfirstlane(tid >> 6);
    const unsigned lds0 = (unsigned)(size_t)(LAS unsigned char*)smem;
    unsigned char* me = p.me;
    const int b = blockIdx.x;
    const unsigned aoff0 = (unsigned)(lane >> 3) * KD + (unsigned)(((lane & 7) ^ (lane >> 4)) << 4);
    const unsigned aoff1 = (unsigned)(lane >> 3) * KD + (unsigned)(((lane & 7) ^ (4 + (lane >> 4))) << 4);
    const unsigned v16 = (unsigned)lane * 16u;

    if (__builtin_expect(b < NPUSH, 1)) {
        if (tid < 128) *(LAS unsigned*)(smem + SYNC_OFF + tid * 4) = 0u;
        auto load_item = [&](int q, u32x4& r0, u32x4& r1, u32x4& r2, u32x4& r3) {
            asm volatile("global_load_dwordx4 %0, %4, %8\n\tglobal_load_dwordx4 %1, %5, %8\n\t"
                         "global_load_dwordx4 %2, %6, %8\n\tglobal_load_dwordx4 %3, %7, %8"
                         : "=&v"(r0), "=&v"(r1), "=&v"(r2), "=&v"(r3)
                         : "v"(aoff0), "v"(aoff1 + 8 * KD), "v"(aoff0 + 16 * KD), "v"(aoff1 + 24 * KD), "s"(p.x + item_src(q)) : "memory");
        };
        if (wid >= 8) {
            const int L = wid - 8;
            unsigned base = BASE0;
            __syncthreads();
#pragma unroll 1
            for (int i = L ? L : 4;; i += 4) {
                if (i > STAGE_AHEAD) {
                    for (;;) {
                        const unsigned pr = lds_ld(lds0 + SY_PROG + (lane % 7) * 4);
                        if (__any(pr >= (unsigned)(i - STAGE_AHEAD)) && __all(pr + 32u > (unsigned)i)) break;
                        __builtin_amdgcn_s_sleep(2);
                    }
                }
                volatile LAS u32x2* mp = (volatile LAS u32x2*)(smem + SY_META + (i & 31) * 8);
                int q;
                if (i < 4) q = static_item(i * NPUSH + b);
                else {
                    unsigned d = 0;
                    if (lane == 0) d = __hip_atomic_fetch_add((unsigned*)(me + WS_DRAW), 1u, __ATOMIC_RELAXED, __HIP_MEMORY_SCOPE_AGENT);
                    const unsigned id = (__builtin_amdgcn_readfirstlane(d) & (NDRAW - 1)) + NSTATIC;
                    if (id >= (unsigned)NITEM) { *mp = (u32x2){(unsigned)(i + 1) << 16 | 0xffffu, 0u}; return; }
                    q = (int)k_order.q[id];
                }
                u32x4 r0, r1, r2, r3;
                load_item(q, r0, r1, r2, r3);
                asm volatile("s_waitcnt vmcnt(0)" : "+v"(r0), "+v"(r1), "+v"(r2), "+v"(r3) :: "memory");
                const unsigned d[16] = {r0.x, r0.y, r0.z, r0.w, r1.x, r1.y, r1.z, r1.w, r2.x, r2.y, r2.z, r2.w, r3.x, r3.y, r3.z, r3.w};
                LAS unsigned char* slot = (LAS unsigned char*)smem + (i & 31) * ITEM;
                unsigned meta = META_RAW, g[16];
                const int rr = (q >> 7) == 0 ? RAW_LAST[0] : (q >> 7) == 1 ? RAW_LAST[1] : (q >> 7) == 2 ? RAW_LAST[2] : RAW_LAST[3];
                if (((q >> 2) & 31) < NSTEP - rr) {
                    for (int attempt = 0; attempt < 2; ++attempt) {
#pragma unroll
                        for (int e = 0; e < 16; ++e) g[e] = rebase(d[e], base * 0x01010101u);
                        unsigned long long m = tops8(g) | (unsigned long long)tops8(g + 8) << 32;
                        const unsigned limit = attempt ? EXC_MAX : 96;
                        unsigned cnt = 0;
                        for (;;) {
                            const unsigned long long act = __ballot(m != 0);
                            if (!act) { meta = base | cnt << 8; break; }
                            const unsigned n = (unsigned)__popcll(act);
                            if (cnt + n > limit) break;
                            if (m) {
                                const unsigned bit = (unsigned)__builtin_ctzll(m), dw = (bit >> 5) * 8 + (bit & 7);
                                const unsigned at = cnt + __builtin_amdgcn_mbcnt_hi((unsigned)(act >> 32), __builtin_amdgcn_mbcnt_lo((unsigned)act, 0u));
                                *(LAS unsigned short*)(slot + EXC_OFF + at * 2) = (unsigned short)((dw >> 2) * 1024 + v16 + (dw & 3) * 4 + ((bit >> 3) & 3));
                                m &= m - 1;
                            }
                            cnt += n;
                        }
                        if (meta != META_RAW || attempt) break;
                        unsigned mx = 0;
#pragma unroll
                        for (int e = 0; e < 16; ++e) {
                            unsigned v = d[e];
                            asm volatile("" : "+v"(v));
#pragma unroll
                            for (int s = 0; s < 32; s += 8) mx = max(mx, (v >> s) & 0x7fu);
                        }
#pragma unroll
                        for (int o = 32; o; o >>= 1) mx = max(mx, (unsigned)__shfl_xor((int)mx, o));
                        mx = __builtin_amdgcn_readfirstlane(mx);
                        base = mx > 67u ? mx - 67u : 0u;
                    }
                }
                if (meta == META_RAW) {
                    *(LAS u32x4*)(slot + v16) = r0; *(LAS u32x4*)(slot + 1024 + v16) = r1;
                    *(LAS u32x4*)(slot + 2048 + v16) = r2; *(LAS u32x4*)(slot + 3072 + v16) = r3;
                } else {
                    unsigned k[14];
                    pack8(g, k); pack8(g + 8, k + 7);
                    *(LAS u32x4*)(slot + v16) = (u32x4){k[0], k[1], k[2], k[3]};
                    *(LAS u32x4*)(slot + 1024 + v16) = (u32x4){k[4], k[5], k[6], k[7]};
                    *(LAS u32x4*)(slot + 2048 + v16) = (u32x4){k[8], k[9], k[10], k[11]};
                    *(LAS u32x2*)(slot + 3072 + lane * 8) = (u32x2){k[12], k[13]};
                }
                asm volatile("s_waitcnt lgkmcnt(0)" ::: "memory");
                *mp = (u32x2){(unsigned)(i + 1) << 16 | (unsigned)q, meta};
            }
        }
        if (wid == 7) { __syncthreads(); return; }
        unsigned char* peer = p.peer[(wid + b) % 7];
        const int rank = p.rank;
        int q = static_item(b);
        u32x4 r0, r1, r2, r3;
        load_item(q, r0, r1, r2, r3);
        unsigned tk = ticket_issue(me, wid);
        asm volatile("" :: "s"(peer), "s"(rank));
        asm volatile("s_waitcnt lgkmcnt(0)\n\ts_barrier\n\ts_waitcnt vmcnt(0)" : "+v"(tk), "+v"(r0), "+v"(r1), "+v"(r2), "+v"(r3) :: "memory");
        const unsigned call = __builtin_amdgcn_readfirstlane(tk) + 1u;
        unsigned char* dst = peer + (call & 1u) * XBUF + (size_t)rank * SHARD;
        unsigned char* fl = peer + WS_FLAG + (call & 1u) * FLAG_PAR + (size_t)rank * FLAG_SRC;
        unsigned char* fprev = nullptr;
        u32x2 fval = {META_RAW, call};
        unsigned ended = 0, meta = META_RAW;
        for (int i = 0; ended != 15u; ++i) {
            if (i > 0) {
                if (ended >> (i & 3) & 1u) continue;
                u32x2 m;
                for (;;) {
                    asm volatile("ds_read_b64 %0, %1\n\ts_waitcnt lgkmcnt(0)" : "=v"(m) : "v"(lds0 + SY_META + (i & 31) * 8) : "memory");
                    if (__builtin_amdgcn_readfirstlane(m.x) >> 16 == (unsigned)(i + 1)) break;
                    __builtin_amdgcn_s_sleep(1);
                }
                q = (int)(__builtin_amdgcn_readfirstlane(m.x) & 0xffffu);
                meta = __builtin_amdgcn_readfirstlane(m.y);
                if (q == 0xffff) { ended |= 1u << (i & 3); lds_st(lds0 + SY_PROG + wid * 4, (unsigned)i); continue; }
                asm volatile("ds_read_b128 %0, %4\n\tds_read_b128 %1, %4 offset:1024\n\tds_read_b128 %2, %4 offset:2048\n\t"
                             "ds_read_b128 %3, %4 offset:3072\n\ts_waitcnt lgkmcnt(0)"
                             : "=&v"(r0), "=&v"(r1), "=&v"(r2), "=&v"(r3) : "v"(lds0 + (unsigned)(i & 31) * ITEM + v16) : "memory");
                lds_st(lds0 + SY_PROG + wid * 4, (unsigned)i);
                while (!__all(lds_ld(lds0 + SY_PROG + (lane % 7) * 4) + LOCK >= (unsigned)i)) __builtin_amdgcn_s_sleep(1);
            }
            unsigned char* o = dst + (size_t)q * ITEM;
            const unsigned nl = meta == META_RAW ? 64u : 32u + (((meta >> 8) + 31u) >> 5) * 4u;
            wait_vm<3>();
            asm volatile("global_store_dwordx4 %0, %1, %3 sc0 sc1\n\tglobal_store_dwordx4 %0, %2, %3 offset:1024 sc0 sc1\n\ts_nop 1"
                         :: "v"(v16), "v"(r0), "v"(r1), "s"(o) : "memory");
            if (i > 0) {
                wait_vm<2>();
                if (lane == 0) asm volatile("global_store_dwordx2 %0, %1, %2 sc0 sc1" :: "v"(0u), "v"(fval), "s"(fprev) : "memory");
            }
            asm volatile("global_store_dwordx4 %0, %1, %2 offset:2048 sc0 sc1\n\ts_nop 1" :: "v"(v16), "v"(r2), "s"(o) : "memory");
            if ((unsigned)lane < nl) asm volatile("global_store_dwordx4 %0, %1, %2 offset:3072 sc0 sc1\n\ts_nop 1" :: "v"(v16), "v"(r3), "s"(o) : "memory");
            fprev = fl + item_flag(q);
            fval.x = meta;
        }
        wait_vm<0>();
        if (lane == 0) asm volatile("global_store_dwordx2 %0, %1, %2 sc0 sc1" :: "v"(0u), "v"(fval), "s"(fprev) : "memory");
        wait_vm<0>();
        return;
    }

    const int tile = TILE_INDEX(b), ms = tile >> 3, nt = tile & 7;
    const bool local = ms >= 28;
    const int t = local ? ms - 28 : (ms & 3);
    const int src = local ? p.rank : ((p.rank + 1 + (ms >> 2)) & (NR - 1));
    const bool has_extra = t < 2;
    const int xrow = local ? 256 + 16 * t : 288 + 16 * ((ms >> 2) * 2 + t);

    if (wid >= 8) {
        const int h = wid - 8;
        const bool ex = has_extra && h < 2;
        __builtin_amdgcn_s_setprio(3);
        if (h == 0 && lane == 0) lds_st(lds0 + SY_BAD, 0u);
        const unsigned eoff = h ? aoff1 + 8 * KD : aoff0;
        const unsigned char* xe = p.x + (size_t)xrow * KD;
        unsigned eslot = lds0 + h * 1024, aslot = lds0 + h * ITEM;
        auto advance = [&](unsigned& s, unsigned first) { s = s == first + (NSLOT - 1) * SLOT_BYTES ? first : s + SLOT_BYTES; };
        auto extra = [&](int k) {
            if (ex && k < NSTEP) glds<E_OFF>(xe + k * BK, eoff, eslot);
            advance(eslot, lds0 + h * 1024);
        };
        auto barrier = [&]() { asm volatile("s_waitcnt lgkmcnt(0)\n\ts_barrier" ::: "memory"); };
        if (local) {
            const unsigned char* xl = p.x + (size_t)(t * BM + h * 32) * KD;
            auto issue = [&](int k) {
                extra(k);
                static_for<0, 4>([&](auto cc) { constexpr int c = decltype(cc)::v; glds<c * 1024>(xl + k * BK, ((c & 1) ? aoff1 : aoff0) + c * 8 * KD, aslot); });
                advance(aslot, lds0 + h * ITEM);
            };
            issue(0); issue(1);
#pragma unroll 1
            for (int s = -1; s < NSTEP - 2; ++s) {
                if (s >= 0) issue(s + 2);
                if (ex) wait_vm<5>(); else wait_vm<4>();
                barrier();
            }
            wait_vm<0>();
            barrier();
            return;
        }
        auto remote = [&](auto exc) {
        constexpr int NW = decltype(exc)::v;
        unsigned tk = ticket_issue(me, h);
        extra(0); extra(1);
        const int pi = (ms >> 2) * 16 + (t & 1) * 8 + nt;
        int pfs = t < 2 ? pi % 7 : NSTEP;
        const unsigned char* pfp = p.w + (size_t)((pfs + PF_AHEAD) * BK + (pi / 7) * 8 + h * 2) * ND;
        auto prefetch_row = [&](const unsigned char* row) {
            asm volatile("s_add_u32 m0, %2, %3\n\ts_nop 0\n\tglobal_load_lds_dwordx4 %0, %1\n\tglobal_load_lds_dwordx4 %0, %1 offset:1024"
                         :: "v"(v16), "s"(row), "s"(lds0), "i"(JUNK_OFF) : "memory", "scc");
        };
        {
            const unsigned long long t0 = __builtin_amdgcn_s_memrealtime();
            while (__builtin_amdgcn_s_memrealtime() - t0 < (unsigned long long)PF_DELAY) __builtin_amdgcn_s_sleep(4);
            for (int n = ms * 32 + nt * 4 + h; n < (PF_AHEAD - 2) * BK; n += 28 * 32) prefetch_row(p.w + (size_t)(2 * BK + n) * ND);
        }
        asm volatile("s_waitcnt vmcnt(0)" : "+v"(tk) :: "memory");
        const unsigned call = __builtin_amdgcn_readfirstlane(tk) + 1u;
        const unsigned char* xr = me + (call & 1u) * XBUF + (size_t)src * SHARD + (size_t)(t * NSTEP * 4 + h) * ITEM;
        const unsigned char* fl = me + WS_FLAG + (call & 1u) * FLAG_PAR + (size_t)src * FLAG_SRC + (size_t)(t * 4 + h) * NSTEP * 8;
        unsigned* tmo = (unsigned*)(me + WS_TMO);
        const unsigned v8 = (unsigned)(lane & 31) * 8u;
        u32x2 fv = {0u, 0u};
        unsigned meta = 0, seen = 0;
        bool dead = false, pend = false, polled = false;
        int behind = 0;
        int next = 0;
        auto look = [&]() { seen = (unsigned)__ballot(fv.y >= call); };
        const unsigned vt = v16 >> 1, ve = v16 >> 3;
        auto request = [&](bool second) {
            meta = __builtin_amdgcn_readlane(fv.x, next);
            const unsigned char* a = xr + (size_t)next * 4 * ITEM;
            unsigned place = aslot;
            if (second) advance(place, lds0 + h * ITEM);
            if (meta == META_RAW) raw_loads(a, v16, ve, place);
            else { if (next & 1) item_loads<140>(a, v16, vt, ve); else item_loads<120>(a, v16, vt, ve); }
            ++next;
        };
        auto poll_async = [&]() {
            asm volatile("global_load_dwordx2 v[158:159], %0, %1 sc0 sc1" :: "v"(v8), "s"(fl) : "memory", "v158", "v159");
            polled = true;
        };
        auto take_poll = [&]() {
            asm volatile("v_mov_b64 %0, v[158:159]" : "=v"(fv) :: "memory");
            look(); polled = false;
        };
        auto ahead = [&]() {
            if (next < NSTEP) {
                if (seen >> next & 1u) { request(true); pend = true; }
                poll_async();
            }
        };
        auto flip = [&](unsigned pos, bool on) {
            if (on) asm volatile("ds_xor_b32 %0, %1" :: "v"(aslot + (pos & 0xffcu)), "v"(0x40u << ((pos & 3u) * 8u)) : "memory");
        };
        auto expand_from = [&](auto rc, unsigned m, int k) {
            constexpr int R = decltype(rc)::v;
            const unsigned oa = aslot + v16;
            if (m != META_RAW) {
                LAS unsigned char* o = (LAS unsigned char*)(size_t)oa;
                const unsigned b4 = (m & 0xffu) * 0x01010101u, cnt = m >> 8;
                unsigned d[16];
                unpack8<R>(d, b4, [](int) {});
                *(LAS u32x4*)o = (u32x4){d[0], d[1], d[2], d[3]}; *(LAS u32x4*)(o + 1024) = (u32x4){d[4], d[5], d[6], d[7]};
                __builtin_amdgcn_sched_barrier(0);
                unpack8<R + 7>(d + 8, b4, [](int) {});
                *(LAS u32x4*)(o + 2048) = (u32x4){d[8], d[9], d[10], d[11]}; *(LAS u32x4*)(o + 3072) = (u32x4){d[12], d[13], d[14], d[15]};
                flip(reg_get<R + 16>(), (unsigned)lane < cnt);
                for (unsigned c0 = 64; c0 < cnt && c0 < (unsigned)EXC_MAX; c0 += 64) {
                    unsigned e;
                    const unsigned eo = (c0 + lane) * 2;
                    const unsigned char* ea = xr + (size_t)k * 4 * ITEM;
                    asm volatile("global_load_ushort %0, %1, %2 offset:3584 sc0 sc1\n\ts_waitcnt vmcnt(0)" : "=&v"(e) : "v"(eo), "s"(ea) : "memory");
                    flip(e, c0 + lane < cnt);
                }
            }
        };
        auto expand = [&](unsigned m, int k) {
            if (k & 1) expand_from(IC<140>{}, m, k); else expand_from(IC<120>{}, m, k);
            advance(aslot, lds0 + h * ITEM);
        };
        auto settle = [&]() {
            if (behind == 1) wait_vm<NW>(); else if (behind == 3) wait_vm<NW + 4>(); else if (behind == 2) wait_vm<4>(); else wait_vm<0>();
        };
        auto finish = [&](int s) {
            if (s == pfs) {
                prefetch_row(pfp); prefetch_row(pfp + ND);
                behind |= 2;
                pfs = pfs + 7 + PF_AHEAD < NSTEP ? pfs + 7 : NSTEP;
                pfp += (size_t)7 * BK * ND;
            }
            barrier();
        };
        auto fast = [&](auto rc, auto rn, int s) {
            constexpr int RC = decltype(rc)::v, RN = decltype(rn)::v;
            if (!(seen >> next & 1u)) {
                settle(); take_poll(); polled = true;
                if (!(seen >> next & 1u)) return false;
            }
            const unsigned mn = __builtin_amdgcn_readlane(fv.x, next);
            if (mn >= FAST_META) return false;
            const unsigned b4 = (meta & 0xffu) * 0x01010101u, cnt = meta >> 8, oa = aslot + v16;
            const unsigned char* a = xr + (size_t)next * 4 * ITEM;
            const bool pf = behind & 2;
            if (behind & 1) { if (pf) wait_vm<NW + 8>(); else wait_vm<NW + 4>(); } else { if (pf) wait_vm<8>(); else wait_vm<4>(); }
            unsigned d[16];
            LAS unsigned char* o = (LAS unsigned char*)(size_t)oa;
            load16<RN, 0>(a, v16);
            unpack8<RC>(d, b4, [&](int i) { if (i == 0) load16<RN + 4, 1024>(a, v16); else load16<RN + 8, 2048>(a, v16); });
            *(LAS u32x4*)o = (u32x4){d[0], d[1], d[2], d[3]}; *(LAS u32x4*)(o + 1024) = (u32x4){d[4], d[5], d[6], d[7]};
            if (behind & 1) { if (pf) wait_vm<NW + 7>(); else wait_vm<NW + 3>(); } else { if (pf) wait_vm<7>(); else wait_vm<3>(); }
            take_poll();
            load16<RN + 12, 3072>(a, vt);
            unpack8<RC + 7>(d + 8, b4, [&](int i) { if (i == 0) load2<RN + 16>(a, ve); else poll_async(); });
            *(LAS u32x4*)(o + 2048) = (u32x4){d[8], d[9], d[10], d[11]}; *(LAS u32x4*)(o + 3072) = (u32x4){d[12], d[13], d[14], d[15]};
            flip(reg_get<RC + 16>(), (unsigned)lane < cnt);
            advance(aslot, lds0 + h * ITEM);
            meta = mn; ++next;
            if (NW) {
                asm volatile("s_waitcnt lgkmcnt(0)" ::: "memory");
                extra(s + 2);
                if (pf) wait_vm<NW + 10>(); else wait_vm<NW + 6>();
            }
            behind = NW ? 1 : 0;
            finish(s);
            return true;
        };
        auto fast_raw = [&](int s) {
            if (!(seen >> next & 1u)) {
                settle(); take_poll(); polled = true;
                if (!(seen >> next & 1u)) return false;
            }
            const bool pf = behind & 2;
            request(true);
            advance(aslot, lds0 + h * ITEM);
            if (behind & 1) { if (pf) wait_vm<NW + 9>(); else wait_vm<NW + 5>(); } else { if (pf) wait_vm<9>(); else wait_vm<5>(); }
            take_poll(); poll_async();
            if (NW) { extra(s + 2); if (pf) wait_vm<NW + 10>(); else wait_vm<NW + 6>(); }
            behind = NW ? 1 : 0;
            finish(s);
            return true;
        };
#pragma unroll 1
        for (int s = -1; s < NSTEP - 1; ++s) {
#pragma unroll 1
            while (pend && meta == META_RAW && next < NSTEP && fast_raw(s)) ++s;
            if (pend && meta < FAST_META) {
                if (!(s & 1)) {
                    if (next < NSTEP && fast(IC<140>{}, IC<120>{}, s)) ++s; else goto general;
                }
#pragma unroll 1
                while (next < NSTEP && fast(IC<120>{}, IC<140>{}, s)) {
                    ++s;
                    if (next < NSTEP && fast(IC<140>{}, IC<120>{}, s)) ++s; else break;
                }
            }
        general:
            settle();
            bool edue = s >= 0 && s + 2 < NSTEP;
            behind = 0;
            if (polled) take_poll();
            if (pend) {
                const unsigned m = meta;
                pend = false;
                ahead();
                expand(m, s + 1);
                if (!edue) { if (s > 0) wait_vm<0>(); }
                else {
                    asm volatile("s_waitcnt lgkmcnt(0)" ::: "memory");
                    extra(s + 2); edue = false; behind = 1;
                    if (pend) wait_vm<NW + 6>(); else wait_vm<NW + 1>();
                }
            } else if (!dead) {
                if (!(seen >> next & 1u)) {
                    if (edue) { extra(s + 2); edue = false; }
                    const unsigned long long t0 = __builtin_amdgcn_s_memrealtime();
                    for (unsigned polls = 1;; ++polls) {
                        asm volatile("global_load_dwordx2 %0, %1, %2 sc0 sc1\n\ts_waitcnt vmcnt(0)" : "=&v"(fv) : "v"(v8), "s"(fl) : "memory");
                        look();
                        if (seen >> next & 1u) break;
                        if ((polls & 63u) == 0u
                            && (__builtin_amdgcn_readfirstlane(__hip_atomic_load(tmo, __ATOMIC_RELAXED, __HIP_MEMORY_SCOPE_AGENT)) == call
                                || __builtin_amdgcn_s_memrealtime() - t0 > DEADLINE)) {
                            if (lane == 0) { __hip_atomic_store(tmo, call, __ATOMIC_RELAXED, __HIP_MEMORY_SCOPE_AGENT); lds_st(lds0 + SY_BAD, 1u); }
                            dead = true;
                            break;
                        }
                        __builtin_amdgcn_s_sleep(2);
                    }
                }
                if (!dead) {
                    request(false); poll_async();
                    wait_vm<0>();
                    const unsigned m = meta;
                    take_poll();
                    ahead();
                    expand(m, s + 1);
                }
            }
            if (edue) {
                asm volatile("s_waitcnt lgkmcnt(0)" ::: "memory");
                extra(s + 2); behind = 1;
            }
            finish(s);
        }
        wait_vm<0>();
        };
        if (ex) remote(IC<1>{}); else remote(IC<0>{});
        return;
    }

    const int wr = wid >> 2, wc = wid & 3;
    unsigned sx = __builtin_amdgcn_readfirstlane(__float_as_uint(p.sx[0])), sw = __builtin_amdgcn_readfirstlane(__float_as_uint(p.sw[0]));
    int ex = 2 - t, top = 1 - wr;
    f32x4 acc[4][4], accx[2];
#pragma unroll
    for (int m = 0; m < 4; ++m)
#pragma unroll
        for (int n = 0; n < 4; ++n) acc[m][n] = (f32x4){0.f, 0.f, 0.f, 0.f};
    accx[0] = accx[1] = (f32x4){0.f, 0.f, 0.f, 0.f};
    const float* ob = p.out + (size_t)(src * MS + t * BM + wr * 64) * NS + nt * BN + wc * 64;
    const float* ox = p.out + (size_t)(p.rank * MS + xrow) * NS + nt * BN + wc * 64 + wr * 32;
    asm volatile("" : "+s"(ob), "+s"(ox), "+s"(sx), "+s"(sw), "+s"(ex), "+s"(top));
    const unsigned char* wp = p.w + nt * BN + (size_t)wid * 16 * ND;
    unsigned wslot = lds0 + B_OFF + wid * 4096;
    unsigned wo0 = w_off(lane, 0, wid), wo1 = w_off(lane, 1, wid);
    auto issue_w = [&]() {
        w_loads(wp, wo0, wo1, wslot);
        wp += (size_t)BK * ND;
        wslot = wslot >= lds0 + B_OFF + (NSLOT - 1) * SLOT_BYTES ? wslot - (NSLOT - 1) * SLOT_BYTES : wslot + SLOT_BYTES;
    };
    issue_w(); issue_w(); wait_vm<4>();
    asm volatile("" : "+v"(wo0), "+v"(wo1));
    unsigned aa[2], ea[2], bo[4];
    {
        const int fr = lane & 15, fq = lane >> 4, q = fr >> 1, f = q | ((fq & 1) << 3);
        aa[0] = lds0 + (unsigned)((wr * 64 + fr) * 128 + ((fq ^ ((fr >> 1) & 7)) << 4));
        aa[1] = aa[0] ^ 64u;
        ea[0] = aa[0] + (unsigned)(E_OFF - wr * 8192);
        ea[1] = ea[0] ^ 64u;
#pragma unroll
        for (int n = 0; n < 4; ++n) bo[n] = lds0 + (unsigned)((16 * fq + q) * 256 + (((wc * 4 + (n ^ (2 * wr))) ^ f) << 4) + (fr & 1) * 8);
    }
    asm volatile(".p2align 8\n\ts_barrier" ::: "memory");
    {
        int slot = 0;
#pragma unroll 1
        for (int s = 0; s < NSTEP; ++s) {
            v8i af[4], bf[4], ef;
#pragma unroll
            for (int n = 0; n < 4; ++n) {
                LAS unsigned char* q = (LAS unsigned char*)(size_t)bo[n] + B_OFF;
                const v2i t0 = __builtin_amdgcn_ds_read_tr8_b64_v2i32((LAS v2i*)q);
                const v2i t1 = __builtin_amdgcn_ds_read_tr8_b64_v2i32((LAS v2i*)(q + 8 * 256));
                const v2i t2 = __builtin_amdgcn_ds_read_tr8_b64_v2i32((LAS v2i*)(q + 64 * 256));
                const v2i t3 = __builtin_amdgcn_ds_read_tr8_b64_v2i32((LAS v2i*)(q + 72 * 256));
                bf[n] = (v8i){t0.x, t0.y, t1.x, t1.y, t2.x, t2.y, t3.x, t3.y};
            }
#pragma unroll
            for (int m = 0; m < 4; ++m) {
                const v4i lo = *(LAS v4i*)((LAS unsigned char*)(size_t)aa[0] + m * 2048), hi = *(LAS v4i*)((LAS unsigned char*)(size_t)aa[1] + m * 2048);
                af[m] = (v8i){lo.x, lo.y, lo.z, lo.w, hi.x, hi.y, hi.z, hi.w};
            }
            asm volatile("" : "+s"(ex));
            if (ex > 0) {
                const v4i lo = *(LAS v4i*)(size_t)ea[0], hi = *(LAS v4i*)(size_t)ea[1];
                ef = (v8i){lo.x, lo.y, lo.z, lo.w, hi.x, hi.y, hi.z, hi.w};
            }
            int wdue = s + 2 < NSTEP ? 1 + top : 0;
            asm volatile("" : "+s"(wdue));
            if (wdue == 1) issue_w();
            asm volatile("" ::: "memory");
            __builtin_amdgcn_s_waitcnt(0xC07F);
            asm volatile("" : "+s"(wdue));
            if (wdue == 2) issue_w();
            slot = (slot == NSLOT - 1) ? 0 : slot + 1;
            const int delta = slot ? SLOT_BYTES : -(NSLOT - 1) * SLOT_BYTES;
            int bar = s < NSTEP - 1 ? 1 + top : 0;
            asm volatile("" : "+s"(bar));
            if (bar == 1) { if (wdue > 0) wait_vm<4>(); else wait_vm<0>(); asm volatile("s_barrier" ::: "memory"); }
            __builtin_amdgcn_sched_barrier(0);
            asm volatile("s_setprio 1\n\t.p2align 3");
            __builtin_amdgcn_sched_barrier(0);
#pragma unroll
            for (int m = 0; m < 4; ++m) {
#pragma unroll
                for (int n = 0; n < 4; ++n) {
                    acc[m][n] = __builtin_amdgcn_mfma_scale_f32_16x16x128_f8f6f4(bf[n], af[m], acc[m][n], 0, 0, 0, 0, 0, 0);
                    if (m == 0 && n == 0) { __builtin_amdgcn_sched_barrier(0); __builtin_amdgcn_s_setprio(2); __builtin_amdgcn_sched_barrier(0); }
                    if (m == 3 && n == 1) {
                        __builtin_amdgcn_sched_barrier(0);
                        asm volatile("" : "+s"(bar));
                        if (bar == 2) { if (wdue > 0) wait_vm<4>(); else wait_vm<0>(); asm volatile("s_barrier" ::: "memory"); }
                        __builtin_amdgcn_sched_barrier(0);
                    }
                }
                __builtin_amdgcn_sched_barrier(0);
                if (m == 0) asm volatile("v_add_u32 %0, %4, %0\n\tv_add_u32 %1, %4, %1\n\tv_add_u32 %2, %4, %2\n\tv_add_u32 %3, %4, %3"
                                         : "+v"(bo[0]), "+v"(bo[1]), "+v"(bo[2]), "+v"(bo[3]) : "s"(delta));
                if (m == 1) asm volatile("v_add_u32 %0, %4, %0\n\tv_add_u32 %1, %4, %1\n\tv_add_u32 %2, %4, %2\n\tv_add_u32 %3, %4, %3"
                                         : "+v"(aa[0]), "+v"(aa[1]), "+v"(ea[0]), "+v"(ea[1]) : "s"(delta));
                __builtin_amdgcn_sched_barrier(0);
            }
            asm volatile("" : "+s"(ex));
            if (ex > 0) {
                accx[0] = __builtin_amdgcn_mfma_scale_f32_16x16x128_f8f6f4(bf[0], ef, accx[0], 0, 0, 0, 0, 0, 0);
                accx[1] = __builtin_amdgcn_mfma_scale_f32_16x16x128_f8f6f4(bf[1], ef, accx[1], 0, 0, 0, 0, 0, 0);
            }
            __builtin_amdgcn_s_setprio(0);
            __builtin_amdgcn_sched_barrier(0);
        }
    }
    float scale = __uint_as_float(sx) * __uint_as_float(sw);
    if (__builtin_amdgcn_readfirstlane(lds_ld(lds0 + SY_BAD)) != 0u) scale = __builtin_nanf("");
    int ln;
    asm volatile("v_mbcnt_lo_u32_b32 %0, -1, 0\n\tv_mbcnt_hi_u32_b32 %0, -1, %0" : "=v"(ln));
    const int fr = ln & 15, fq = ln >> 4;
    const float zs = scale * -1.44269504088896340736f, zmax = 60.f * 1.44269504088896340736f;
    LAS unsigned char* tb = (LAS unsigned char*)smem + ((NSTEP - 3 + (wc >> 1)) % NSLOT) * SLOT_BYTES + wr * 8192 + (wc & 1) * 4096;
    f32x2 e[8];
    f32x4 tv[4];
    auto expo = [&](auto nqc, const f32x4* c) {
        constexpr int NQ = decltype(nqc)::v;
#pragma unroll
        for (int h = 0; h < 2 * NQ; ++h) {
            f32x2 z = (f32x2){c[h >> 1][(h & 1) * 2], c[h >> 1][(h & 1) * 2 + 1]} * zs;
            asm("" : "+v"(z));
            e[h].x = __builtin_amdgcn_exp2f(__builtin_amdgcn_fmed3f(z.x, -zmax, zmax));
            e[h].y = __builtin_amdgcn_exp2f(__builtin_amdgcn_fmed3f(z.y, -zmax, zmax));
        }
        __builtin_amdgcn_sched_barrier(0);
    };
    auto stage = [&](auto nqc, const f32x4* c) {
        constexpr int NQ = decltype(nqc)::v;
#pragma unroll
        for (int q = 0; q < NQ; ++q) {
            f32x2 d0 = e[2 * q] + 1.0f, d1 = e[2 * q + 1] + 1.0f;
            asm("" : "+v"(d0)); asm("" : "+v"(d1));
            const f32x4 r = {__builtin_amdgcn_rcpf(d0.x), __builtin_amdgcn_rcpf(d0.y), __builtin_amdgcn_rcpf(d1.x), __builtin_amdgcn_rcpf(d1.y)};
            const int cb = NQ == 4 ? (q ^ (2 * wr)) : q;
            *(LAS f32x4*)(tb + fr * 256 + (((cb * 4 + fq) ^ (fr & 7)) << 4)) = (c[q] * scale) * r;
        }
        constexpr int LPR = NQ * 4, RPI = 64 / LPR;
#pragma unroll
        for (int i = 0; i < NQ; ++i) {
            const int r = i * RPI + ln / LPR;
            tv[i] = *(LAS f32x4*)(tb + r * 256 + (((ln % LPR) ^ (r & 7)) << 4));
        }
        __builtin_amdgcn_sched_barrier(0);
    };
    auto put = [&](auto nqc, const float* rowp) {
        constexpr int NQ = decltype(nqc)::v, LPR = NQ * 4, RPI = 64 / LPR;
        const unsigned ovoff = (unsigned)((ln / LPR) * NS * 4 + (ln % LPR) * 16);
#pragma unroll
        for (int i = 0; i < NQ; ++i)
            asm volatile("global_store_dwordx4 %0, %1, %2 sc1\n\ts_nop 1" :: "v"(ovoff), "v"(tv[i]), "s"(rowp + (size_t)i * RPI * NS) : "memory");
        __builtin_amdgcn_sched_barrier(0);
    };
    expo(IC<4>{}, acc[0]);
#pragma unroll
    for (int m = 0; m < 4; ++m) {
        stage(IC<4>{}, acc[m]);
        if (m < 3) expo(IC<4>{}, acc[m + 1]);
        else if (has_extra) expo(IC<2>{}, accx);
        put(IC<4>{}, ob + (size_t)m * 16 * NS);
    }
    if (has_extra) { stage(IC<2>{}, accx); put(IC<2>{}, ox); }
}

}

extern "C" void kernel_launch(int rank, int nranks, ncclComm_t comm,
                              hipStream_t stream,
                              void* const* d_in, const int* in_sizes, int n_in,
                              void* d_out, int out_size,
                              const int* in_dtypes, int out_dtype,
                              void* const* d_out_all,
                              void* const* d_ws_all, size_t ws_bytes) {
    (void)comm; (void)in_dtypes; (void)out_dtype; (void)d_out_all;
    if (nranks != NR || (unsigned)rank >= (unsigned)NR || n_in != 4 || ws_bytes < WS_END || in_sizes[0] != MS * KD
        || in_sizes[1] != KD * ND || out_size != NR * MS * NS) {
        fprintf(stderr, "kernel_launch: unexpected problem shape\n");
        return;
    }
    HIP_CHECK(hipFuncSetAttribute((const void*)fused_ag_gemm, hipFuncAttributeMaxDynamicSharedMemorySize, LDS_BYTES));
    Params p;
    p.x = (const unsigned char*)d_in[0];
    p.w = (const unsigned char*)d_in[1] + (size_t)rank * NS;
    p.sx = (const float*)d_in[2];
    p.sw = (const float*)d_in[3];
    p.out = (float*)d_out;
    p.me = (unsigned char*)d_ws_all[rank];
    for (int i = 0; i < NR - 1; ++i) p.peer[i] = (unsigned char*)d_ws_all[(rank + 1 + i) % NR];
    p.rank = rank;
    p.pad = 0;
    hipLaunchKernelGGL(fused_ag_gemm, dim3(NWG), dim3(NTHREADS), LDS_BYTES, stream, p);
    HIP_CHECK(hipGetLastError());
}
```

```cpp
#include <hip/hip_runtime.h>
#include <rccl/rccl.h>
#include <cstdio>

#define HIP_CHECK(expr) do { hipError_t _e = (expr); if (_e != hipSuccess) { \
    fprintf(stderr, "HIP error %s at %s:%d\n", hipGetErrorString(_e), __FILE__, __LINE__); \
    return; } } while (0)

namespace {

#define LAS __attribute__((address_space(3)))
typedef int v8i __attribute__((ext_vector_type(8)));
typedef int v4i __attribute__((ext_vector_type(4)));
typedef int v2i __attribute__((ext_vector_type(2)));
typedef float f32x2 __attribute__((ext_vector_type(2)));
typedef float f32x4 __attribute__((ext_vector_type(4)));
typedef unsigned u32x2 __attribute__((ext_vector_type(2)));
typedef unsigned u32x4 __attribute__((ext_vector_type(4)));

constexpr int NR = 8;
constexpr int KD = 4096, ND = 16384;
constexpr int MS = 512, NS = 2048;
constexpr int BM = 128, BN = 256, BK = 128;
constexpr int NSTEP = KD / BK;
constexpr int MT = MS / BM;
constexpr int NWG = 256, NTHREADS = 768;
constexpr int NSLOT = 3;
constexpr int A_BYTES = BM * BK, E_BYTES = 16 * BK, B_BYTES = BK * BN;
constexpr int E_OFF = A_BYTES, B_OFF = A_BYTES + E_BYTES, SLOT_BYTES = B_OFF + B_BYTES;
constexpr int SYNC_OFF = NSLOT * SLOT_BYTES;
constexpr int SY_BAD = SYNC_OFF;
constexpr int SY_PROG = SYNC_OFF, SY_HALF = SYNC_OFF + 32, SY_META = SYNC_OFF + 64;
constexpr int JUNK_OFF = SYNC_OFF + 512;
constexpr int LDS_BYTES = JUNK_OFF + 2048;
static_assert(LDS_BYTES <= 163840, "LDS");

constexpr int NPUSH = 16;
constexpr int ITEM = 4096;
constexpr int NITEM = MT * NSTEP * 4;
constexpr int NSTATIC = 4 * NPUSH;
constexpr int NDRAW = 512;
static_assert(NITEM - NSTATIC + 4 * NPUSH == NDRAW && 32 * ITEM <= SYNC_OFF, "draw count; staging ring");
constexpr int EXC_OFF = 3584, EXC_MAX = 256;
constexpr unsigned META_RAW = 0x80000000u, FAST_META = 65u << 8;

constexpr size_t SHARD = (size_t)MS * KD;
constexpr size_t XBUF = (size_t)NR * SHARD;
constexpr size_t FLAG_SRC = (size_t)NITEM * 8, FLAG_PAR = NR * FLAG_SRC;
constexpr size_t WS_FLAG = 2 * XBUF;
constexpr size_t WS_CNT = WS_FLAG + 2 * FLAG_PAR;
constexpr size_t WS_TMO = WS_CNT + (size_t)NWG * 8 * 128;
constexpr size_t WS_DRAW = WS_TMO + 1024;
constexpr size_t WS_END = WS_TMO + 4096;
constexpr unsigned long long DEADLINE = 50000000ull;

constexpr unsigned BASE0 = 4;
constexpr int STAGE_AHEAD = 4;
static_assert(STAGE_AHEAD >= 4, "a packing wave may only wait for its own previous position");
constexpr int PF_AHEAD = 7, PF_DELAY = 100;
constexpr int FIN[MT] = {870, 880, 1000, 1000};
constexpr int RAW_LAST[MT] = {3, 3, 7, 7};

struct Params {
    const unsigned char* x;
    unsigned char* me;
    const unsigned char* w;
    float* out;
    unsigned char* peer[NR - 1];
    int rank;
    int pad;
    const float* sx; const float* sw;
};

template <int N> __device__ __forceinline__ void wait_vm() { asm volatile("s_waitcnt vmcnt(%0)" :: "i"(N) : "memory"); }
template <int I> struct IC { static constexpr int v = I; };
template <int I, int E, class F> __device__ __forceinline__ void static_for(F&& f) { if constexpr (I < E) { f(IC<I>{}); static_for<I + 1, E>(f); } }
__device__ __forceinline__ unsigned lds_ld(unsigned addr) {
    unsigned v; asm volatile("ds_read_b32 %0, %1\n\ts_waitcnt lgkmcnt(0)" : "=v"(v) : "v"(addr) : "memory"); return v;
}
__device__ __forceinline__ void lds_st(unsigned addr, unsigned v) { asm volatile("ds_write_b32 %0, %1" :: "v"(addr), "v"(v) : "memory"); }
template <int OFF> __device__ __forceinline__ void glds(const unsigned char* base, unsigned voff, unsigned dst) {
    asm volatile("s_add_u32 m0, %2, %3\n\ts_nop 0\n\tglobal_load_lds_dwordx4 %0, %1" :: "v"(voff), "s"(base), "s"(dst), "i"(OFF) : "memory", "scc");
}
__device__ __forceinline__ unsigned w_off(int lane, int c, int g) {
    return (unsigned)(c * 4 + (lane >> 4)) * ND + (unsigned)(((lane & 15) ^ ((c * 4 + (lane >> 4)) | ((g & 1) << 3))) << 4) - c * 1024u;
}
__device__ __forceinline__ void w_loads(const unsigned char* wp, unsigned o0, unsigned o1, unsigned dst) {
    asm volatile("s_mov_b32 m0, %4\n\ts_nop 0\n\tglobal_load_lds_dwordx4 %0, %2\n\tglobal_load_lds_dwordx4 %1, %2 offset:1024\n\t"
                 "global_load_lds_dwordx4 %0, %3 offset:2048\n\tglobal_load_lds_dwordx4 %1, %3 offset:3072"
                 :: "v"(o0), "v"(o1), "s"(wp), "s"(wp + 8 * ND - 2048), "s"(dst) : "memory");
}
__device__ __forceinline__ unsigned ticket_issue(unsigned char* me, int role) {
    unsigned t = 0;
    unsigned char* a = me + WS_CNT + ((size_t)blockIdx.x * 8 + role) * 128;
    if ((threadIdx.x & 63) == 0) asm volatile("global_atomic_add %0, %1, %2, off sc0" : "+v"(t) : "v"(a), "v"(1u) : "memory");
    return t;
}
__device__ __forceinline__ unsigned bfi(unsigned mask, unsigned a, unsigned b) { return (a & mask) | (b & ~mask); }

__device__ __forceinline__ unsigned rebase(unsigned d, unsigned b4) { return bfi(0x7f7f7f7fu, (d | 0x80808080u) - b4, d); }
__device__ __forceinline__ void pack8(const unsigned* d, unsigned* q) {
#pragma unroll
    for (int i = 0; i < 7; ++i) q[i] = bfi(0x40404040u, i < 6 ? d[7] << (1 + i) : d[7] >> 1, d[i]);
}
#define XREGS "v120", "v121", "v122", "v123", "v124", "v125", "v126", "v127", "v128", "v129", "v130", "v131", "v132", "v133", "v134", "v135", "v136", \
              "v140", "v141", "v142", "v143", "v144", "v145", "v146", "v147", "v148", "v149", "v150", "v151", "v152", "v153", "v154", "v155", "v156"
template <int R> __device__ __forceinline__ void item_loads(const unsigned char* a, unsigned v16, unsigned tail, unsigned ve) {
    asm volatile("global_load_dwordx4 v[%c4:%c5], %0, %3 sc0 sc1\n\tglobal_load_dwordx4 v[%c6:%c7], %0, %3 offset:1024 sc0 sc1\n\t"
                 "global_load_dwordx4 v[%c8:%c9], %0, %3 offset:2048 sc0 sc1\n\tglobal_load_dwordx4 v[%c10:%c11], %1, %3 offset:3072 sc0 sc1\n\t"
                 "global_load_ushort v%c12, %2, %3 offset:3584 sc0 sc1"
                 :: "v"(v16), "v"(tail), "v"(ve), "s"(a), "i"(R), "i"(R + 3), "i"(R + 4), "i"(R + 7), "i"(R + 8), "i"(R + 11),
                    "i"(R + 12), "i"(R + 15), "i"(R + 16) : "memory", XREGS);
}
template <int R, int OFF> __device__ __forceinline__ void load16(const unsigned char* a, unsigned v) {
    asm volatile("global_load_dwordx4 v[%c2:%c3], %0, %1 offset:%4 sc0 sc1" :: "v"(v), "s"(a), "i"(R), "i"(R + 3), "i"(OFF) : "memory", XREGS);
}
template <int R> __device__ __forceinline__ void load2(const unsigned char* a, unsigned v) {
    asm volatile("global_load_ushort v%c2, %0, %1 offset:3584 sc0 sc1" :: "v"(v), "s"(a), "i"(R) : "memory", XREGS);
}
__device__ __forceinline__ void raw_loads(const unsigned char* a, unsigned v16, unsigned ve, unsigned dst) {
    asm volatile("s_mov_b32 m0, %3\n\ts_nop 0\n\tglobal_load_lds_dwordx4 %0, %2 sc0 sc1\n\tglobal_load_lds_dwordx4 %0, %2 offset:1024 sc0 sc1\n\t"
                 "global_load_lds_dwordx4 %0, %2 offset:2048 sc0 sc1\n\tglobal_load_lds_dwordx4 %0, %2 offset:3072 sc0 sc1\n\t"
                 "global_load_ushort v137, %1, %2 sc0 sc1" :: "v"(v16), "v"(ve), "s"(a), "s"(dst) : "memory", "v137");
}
template <int R> __device__ __forceinline__ unsigned reg_get() { unsigned v; asm volatile("v_mov_b32 %0, v%c1" : "=v"(v) : "i"(R)); return v; }
template <int R> __device__ __forceinline__ unsigned carrier() { unsigned c; asm volatile("v_and_b32 %0, 0x40404040, v%c1" : "=v"(c) : "i"(R)); return c; }
template <int R> __device__ __forceinline__ unsigned restore(unsigned c, unsigned b4) {
    unsigned d; asm volatile("v_xad_u32 %0, v%c1, %2, %3" : "=v"(d) : "i"(R), "v"(c), "s"(b4)); return d;
}
template <int R, class H> __device__ __forceinline__ void unpack8(unsigned* d, unsigned b4, H&& hook) {
    const unsigned c[7] = {carrier<R>(), carrier<R + 1>(), carrier<R + 2>(), carrier<R + 3>(), carrier<R + 4>(), carrier<R + 5>(), carrier<R + 6>()};
    hook(0);
    d[0] = restore<R>(c[0], b4); d[1] = restore<R + 1>(c[1], b4); d[2] = restore<R + 2>(c[2], b4); d[3] = restore<R + 3>(c[3], b4);
    d[4] = restore<R + 4>(c[4], b4); d[5] = restore<R + 5>(c[5], b4); d[6] = restore<R + 6>(c[6], b4);
    hook(1);
    const unsigned t0 = (c[0] << 1) | c[1], t1 = (c[2] << 1) | c[3], t2 = (c[4] << 1) | c[5];
    d[7] = ((t0 >> 2) + (t1 >> 4) + (t2 >> 6)) + ((c[6] << 1) + b4);
}
__device__ __forceinline__ unsigned tops8(const unsigned* d) {
    unsigned m = 0;
#pragma unroll
    for (int i = 0; i < 8; ++i) m |= (i < 7 ? d[i] >> (6 - i) : d[i] << 1) & (0x01010101u << i);
    return m;
}

#define TILE_INDEX(b) ((((b) - NPUSH) & 7) * 30 + (((b) - NPUSH) >> 3))

struct Order { unsigned q[NITEM]; };
constexpr Order make_order() {
    Order o{}; int key[NITEM] = {}; int n = 0;
    for (int k = 0; k < NSTEP; ++k)
        for (int t = 0; t < MT; ++t)
            for (int j = 0; j < 4; ++j) {
                const int due = k < 4 ? 0 : FIN[t] * (k + 1);
                int i = n++;
                while (i > 0 && key[i - 1] > due) { key[i] = key[i - 1]; o.q[i] = o.q[i - 1]; --i; }
                key[i] = due; o.q[i] = (unsigned)((t * NSTEP + k) * 4 + j);
            }
    return o;
}
__constant__ const Order k_order = make_order();
__device__ __forceinline__ int static_item(int id) { return (((id >> 2) & 3) * NSTEP + (id >> 4)) * 4 + (id & 3); }
__device__ __forceinline__ size_t item_src(int q) { return (size_t)((q >> 7) * BM + (q & 3) * 32) * KD + ((q >> 2) & 31) * BK; }
__device__ __forceinline__ size_t item_flag(int q) { return (size_t)((((q >> 7) * 4 + (q & 3)) * NSTEP) + ((q >> 2) & 31)) * 8; }

__global__ __launch_bounds__(NTHREADS) void fused_ag_gemm(Params p) {
    extern __shared__ __attribute__((aligned(16))) unsigned char smem[];
    const int tid = threadIdx.x, lane = tid & 63, wid = __builtin_amdgcn_readfirstlane(tid >> 6);
    const unsigned lds0 = (unsigned)(size_t)(LAS unsigned char*)smem;
    unsigned char* me = p.me;
    const int b = blockIdx.x;
    const unsigned aoff0 = (unsigned)(lane >> 3) * KD + (unsigned)(((lane & 7) ^ (lane >> 4)) << 4);
    const unsigned aoff1 = (unsigned)(lane >> 3) * KD + (unsigned)(((lane & 7) ^ (4 + (lane >> 4))) << 4);
    const unsigned v16 = (unsigned)lane * 16u;

    if (__builtin_expect(b < NPUSH, 1)) {
        if (tid < 128) *(LAS unsigned*)(smem + SYNC_OFF + tid * 4) = 0u;
        auto load_item = [&](int q, u32x4& r0, u32x4& r1, u32x4& r2, u32x4& r3) {
            asm volatile("global_load_dwordx4 %0, %4, %8\n\tglobal_load_dwordx4 %1, %5, %8\n\t"
                         "global_load_dwordx4 %2, %6, %8\n\tglobal_load_dwordx4 %3, %7, %8"
                         : "=&v"(r0), "=&v"(r1), "=&v"(r2), "=&v"(r3)
                         : "v"(aoff0), "v"(aoff1 + 8 * KD), "v"(aoff0 + 16 * KD), "v"(aoff1 + 24 * KD), "s"(p.x + item_src(q)) : "memory");
        };
        if (wid >= 8) {
            const int L = wid - 8;
            unsigned base = BASE0;
            __syncthreads();
#pragma unroll 1
            for (int i = L ? L : 4;; i += 4) {
                if (i > STAGE_AHEAD) {
                    for (;;) {
                        const unsigned pr = lds_ld(lds0 + SY_PROG + (lane % 7) * 4);
                        if (__any(pr >= (unsigned)(i - STAGE_AHEAD)) && __all(pr + 32u > (unsigned)i)) break;
                        __builtin_amdgcn_s_sleep(2);
                    }
                }
                volatile LAS u32x2* mp = (volatile LAS u32x2*)(smem + SY_META + (i & 31) * 8);
                int q;
                if (i < 4) q = static_item(i * NPUSH + b);
                else {
                    unsigned d = 0;
                    if (lane == 0) d = __hip_atomic_fetch_add((unsigned*)(me + WS_DRAW), 1u, __ATOMIC_RELAXED, __HIP_MEMORY_SCOPE_AGENT);
                    const unsigned id = (__builtin_amdgcn_readfirstlane(d) & (NDRAW - 1)) + NSTATIC;
                    if (id >= (unsigned)NITEM) { *mp = (u32x2){(unsigned)(i + 1) << 16 | 0xffffu, 0u}; return; }
                    q = (int)k_order.q[id];
                }
                u32x4 r0, r1, r2, r3;
                load_item(q, r0, r1, r2, r3);
                asm volatile("s_waitcnt vmcnt(0)" : "+v"(r0), "+v"(r1), "+v"(r2), "+v"(r3) :: "memory");
                const unsigned d[16] = {r0.x, r0.y, r0.z, r0.w, r1.x, r1.y, r1.z, r1.w, r2.x, r2.y, r2.z, r2.w, r3.x, r3.y, r3.z, r3.w};
                LAS unsigned char* slot = (LAS unsigned char*)smem + (i & 31) * ITEM;
                unsigned meta = META_RAW, g[16];
                const int rr = (q >> 7) == 0 ? RAW_LAST[0] : (q >> 7) == 1 ? RAW_LAST[1] : (q >> 7) == 2 ? RAW_LAST[2] : RAW_LAST[3];
                if (((q >> 2) & 31) < NSTEP - rr) {
                    for (int attempt = 0; attempt < 2; ++attempt) {
#pragma unroll
                        for (int e = 0; e < 16; ++e) g[e] = rebase(d[e], base * 0x01010101u);
                        unsigned long long m = tops8(g) | (unsigned long long)tops8(g + 8) << 32;
                        const unsigned limit = attempt ? EXC_MAX : 96;
                        unsigned cnt = 0;
                        for (;;) {
                            const unsigned long long act = __ballot(m != 0);
                            if (!act) { meta = base | cnt << 8; break; }
                            const unsigned n = (unsigned)__popcll(act);
                            if (cnt + n > limit) break;
                            if (m) {
                                const unsigned bit = (unsigned)__builtin_ctzll(m), dw = (bit >> 5) * 8 + (bit & 7);
                                const unsigned at = cnt + __builtin_amdgcn_mbcnt_hi((unsigned)(act >> 32), __builtin_amdgcn_mbcnt_lo((unsigned)act, 0u));
                                *(LAS unsigned short*)(slot + EXC_OFF + at * 2) = (unsigned short)((dw >> 2) * 1024 + v16 + (dw & 3) * 4 + ((bit >> 3) & 3));
                                m &= m - 1;
                            }
                            cnt += n;
                        }
                        if (meta != META_RAW || attempt) break;
                        unsigned mx = 0;
#pragma unroll
                        for (int e = 0; e < 16; ++e) {
                            unsigned v = d[e];
                            asm volatile("" : "+v"(v));
#pragma unroll
                            for (int s = 0; s < 32; s += 8) mx = max(mx, (v >> s) & 0x7fu);
                        }
#pragma unroll
                        for (int o = 32; o; o >>= 1) mx = max(mx, (unsigned)__shfl_xor((int)mx, o));
                        mx = __builtin_amdgcn_readfirstlane(mx);
                        base = mx > 67u ? mx - 67u : 0u;
                    }
                }
                if (meta == META_RAW) {
                    *(LAS u32x4*)(slot + v16) = r0; *(LAS u32x4*)(slot + 1024 + v16) = r1;
                    *(LAS u32x4*)(slot + 2048 + v16) = r2; *(LAS u32x4*)(slot + 3072 + v16) = r3;
                } else {
                    unsigned k[14];
                    pack8(g, k); pack8(g + 8, k + 7);
                    *(LAS u32x4*)(slot + v16) = (u32x4){k[0], k[1], k[2], k[3]};
                    *(LAS u32x4*)(slot + 1024 + v16) = (u32x4){k[4], k[5], k[6], k[7]};
                    *(LAS u32x4*)(slot + 2048 + v16) = (u32x4){k[8], k[9], k[10], k[11]};
                    *(LAS u32x2*)(slot + 3072 + lane * 8) = (u32x2){k[12], k[13]};
                }
                asm volatile("s_waitcnt lgkmcnt(0)" ::: "memory");
                *mp = (u32x2){(unsigned)(i + 1) << 16 | (unsigned)q, meta};
            }
        }
        if (wid == 7) { __syncthreads(); return; }
        unsigned char* peer = p.peer[(wid + b) % 7];
        const int rank = p.rank;
        int q = static_item(b);
        u32x4 r0, r1, r2, r3;
        load_item(q, r0, r1, r2, r3);
        unsigned tk = ticket_issue(me, wid);
        asm volatile("" :: "s"(peer), "s"(rank));
        asm volatile("s_waitcnt lgkmcnt(0)\n\ts_barrier\n\ts_waitcnt vmcnt(0)" : "+v"(tk), "+v"(r0), "+v"(r1), "+v"(r2), "+v"(r3) :: "memory");
        const unsigned call = __builtin_amdgcn_readfirstlane(tk) + 1u;
        unsigned char* dst = peer + (call & 1u) * XBUF + (size_t)rank * SHARD;
        unsigned char* fl = peer + WS_FLAG + (call & 1u) * FLAG_PAR + (size_t)rank * FLAG_SRC;
        unsigned char* fprev = nullptr;
        u32x2 fval = {META_RAW, call};
        unsigned ended = 0, meta = META_RAW, halves = 0;
        bool owed = false;
        auto set_flag = [&]() {
            if (lane == 0) asm volatile("global_store_dwordx2 %0, %1, %2 sc0 sc1" :: "v"(0u), "v"(fval), "s"(fprev) : "memory");
            owed = false;
        };
        auto in_step = [&]() { return __all(lds_ld(lds0 + SY_HALF + (lane % 7) * 4) + 1u >= halves); };
        for (int i = 0; ended != 15u; ++i) {
            if (i > 0) {
                if (ended >> (i & 3) & 1u) continue;
                u32x2 m;
                for (;;) {
                    asm volatile("ds_read_b64 %0, %1\n\ts_waitcnt lgkmcnt(0)" : "=v"(m) : "v"(lds0 + SY_META + (i & 31) * 8) : "memory");
                    if (__builtin_amdgcn_readfirstlane(m.x) >> 16 == (unsigned)(i + 1)) break;
                    __builtin_amdgcn_s_sleep(1);
                }
                q = (int)(__builtin_amdgcn_readfirstlane(m.x) & 0xffffu);
                meta = __builtin_amdgcn_readfirstlane(m.y);
                if (q == 0xffff) { ended |= 1u << (i & 3); lds_st(lds0 + SY_PROG + wid * 4, (unsigned)i); continue; }
                asm volatile("ds_read_b128 %0, %4\n\tds_read_b128 %1, %4 offset:1024\n\tds_read_b128 %2, %4 offset:2048\n\t"
                             "ds_read_b128 %3, %4 offset:3072\n\ts_waitcnt lgkmcnt(0)"
                             : "=&v"(r0), "=&v"(r1), "=&v"(r2), "=&v"(r3) : "v"(lds0 + (unsigned)(i & 31) * ITEM + v16) : "memory");
                lds_st(lds0 + SY_PROG + wid * 4, (unsigned)i);
            }
            unsigned char* o = dst + (size_t)q * ITEM;
            const unsigned nl = meta == META_RAW ? 64u : 32u + (((meta >> 8) + 31u) >> 5) * 4u;
            wait_vm<3>();
            lds_st(lds0 + SY_HALF + wid * 4, halves);
            if (!in_step()) {
                if (owed) { wait_vm<0>(); set_flag(); }
                while (!in_step()) __builtin_amdgcn_s_sleep(1);
            }
            asm volatile("global_store_dwordx4 %0, %1, %3 sc0 sc1\n\tglobal_store_dwordx4 %0, %2, %3 offset:1024 sc0 sc1\n\ts_nop 1"
                         :: "v"(v16), "v"(r0), "v"(r1), "s"(o) : "memory");
            if (owed) {
                wait_vm<2>();
                set_flag();
            }
            lds_st(lds0 + SY_HALF + wid * 4, halves + 1u);
            halves += 2u;
            asm volatile("global_store_dwordx4 %0, %1, %2 offset:2048 sc0 sc1\n\ts_nop 1" :: "v"(v16), "v"(r2), "s"(o) : "memory");
            if ((unsigned)lane < nl) asm volatile("global_store_dwordx4 %0, %1, %2 offset:3072 sc0 sc1\n\ts_nop 1" :: "v"(v16), "v"(r3), "s"(o) : "memory");
            fprev = fl + item_flag(q);
            fval.x = meta;
            owed = true;
        }
        wait_vm<0>();
        set_flag();
        wait_vm<0>();
        return;
    }

    const int tile = TILE_INDEX(b), ms = tile >> 3, nt = tile & 7;
    const bool local = ms >= 28;
    const int t = local ? ms - 28 : (ms & 3);
    const int src = local ? p.rank : ((p.rank + 1 + (ms >> 2)) & (NR - 1));
    const bool has_extra = t < 2;
    const int xrow = local ? 256 + 16 * t : 288 + 16 * ((ms >> 2) * 2 + t);

    if (wid >= 8) {
        const int h = wid - 8;
        const bool ex = has_extra && h < 2;
        __builtin_amdgcn_s_setprio(3);
        if (h == 0 && lane == 0) lds_st(lds0 + SY_BAD, 0u);
        const unsigned eoff = h ? aoff1 + 8 * KD : aoff0;
        const unsigned char* xe = p.x + (size_t)xrow * KD;
        unsigned eslot = lds0 + h * 1024, aslot = lds0 + h * ITEM;
        auto advance = [&](unsigned& s, unsigned first) { s = s == first + (NSLOT - 1) * SLOT_BYTES ? first : s + SLOT_BYTES; };
        auto extra = [&](int k) {
            if (ex && k < NSTEP) glds<E_OFF>(xe + k * BK, eoff, eslot);
            advance(eslot, lds0 + h * 1024);
        };
        auto barrier = [&]() { asm volatile("s_waitcnt lgkmcnt(0)\n\ts_barrier" ::: "memory"); };
        if (local) {
            const unsigned char* xl = p.x + (size_t)(t * BM + h * 32) * KD;
            auto issue = [&](int k) {
                extra(k);
                static_for<0, 4>([&](auto cc) { constexpr int c = decltype(cc)::v; glds<c * 1024>(xl + k * BK, ((c & 1) ? aoff1 : aoff0) + c * 8 * KD, aslot); });
                advance(aslot, lds0 + h * ITEM);
            };
            issue(0); issue(1);
#pragma unroll 1
            for (int s = -1; s < NSTEP - 2; ++s) {
                if (s >= 0) issue(s + 2);
                if (ex) wait_vm<5>(); else wait_vm<4>();
                barrier();
            }
            wait_vm<0>();
            barrier();
            return;
        }
        auto remote = [&](auto exc) {
        constexpr int NW = decltype(exc)::v;
        unsigned tk = ticket_issue(me, h);
        extra(0); extra(1);
        const int pi = (ms >> 2) * 16 + (t & 1) * 8 + nt;
        int pfs = t < 2 ? pi % 7 : NSTEP;
        const unsigned char* pfp = p.w + (size_t)((pfs + PF_AHEAD) * BK + (pi / 7) * 8 + h * 2) * ND;
        auto prefetch_row = [&](const unsigned char* row) {
            asm volatile("s_add_u32 m0, %2, %3\n\ts_nop 0\n\tglobal_load_lds_dwordx4 %0, %1\n\tglobal_load_lds_dwordx4 %0, %1 offset:1024"
                         :: "v"(v16), "s"(row), "s"(lds0), "i"(JUNK_OFF) : "memory", "scc");
        };
        {
            const unsigned long long t0 = __builtin_amdgcn_s_memrealtime();
            while (__builtin_amdgcn_s_memrealtime() - t0 < (unsigned long long)PF_DELAY) __builtin_amdgcn_s_sleep(4);
            for (int n = ms * 32 + nt * 4 + h; n < (PF_AHEAD - 2) * BK; n += 28 * 32) prefetch_row(p.w + (size_t)(2 * BK + n) * ND);
        }
        asm volatile("s_waitcnt vmcnt(0)" : "+v"(tk) :: "memory");
        const unsigned call = __builtin_amdgcn_readfirstlane(tk) + 1u;
        const unsigned char* xr = me + (call & 1u) * XBUF + (size_t)src * SHARD + (size_t)(t * NSTEP * 4 + h) * ITEM;
        const unsigned char* fl = me + WS_FLAG + (call & 1u) * FLAG_PAR + (size_t)src * FLAG_SRC + (size_t)(t * 4 + h) * NSTEP * 8;
        unsigned* tmo = (unsigned*)(me + WS_TMO);
        const unsigned v8 = (unsigned)(lane & 31) * 8u;
        u32x2 fv = {0u, 0u};
        unsigned meta = 0, seen = 0;
        bool dead = false, pend = false, polled = false;
        int behind = 0;
        int next = 0;
        auto look = [&]() { seen = (unsigned)__ballot(fv.y >= call); };
        const unsigned vt = v16 >> 1, ve = v16 >> 3;
        auto request = [&](bool second) {
            meta = __builtin_amdgcn_readlane(fv.x, next);
            const unsigned char* a = xr + (size_t)next * 4 * ITEM;
            unsigned place = aslot;
            if (second) advance(place, lds0 + h * ITEM);
            if (meta == META_RAW) raw_loads(a, v16, ve, place);
            else { if (next & 1) item_loads<140>(a, v16, vt, ve); else item_loads<120>(a, v16, vt, ve); }
            ++next;
        };
        auto poll_async = [&]() {
            asm volatile("global_load_dwordx2 v[158:159], %0, %1 sc0 sc1" :: "v"(v8), "s"(fl) : "memory", "v158", "v159");
            polled = true;
        };
        auto take_poll = [&]() {
            asm volatile("v_mov_b64 %0, v[158:159]" : "=v"(fv) :: "memory");
            look(); polled = false;
        };
        auto ahead = [&]() {
            if (next < NSTEP) {
                if (seen >> next & 1u) { request(true); pend = true; }
                poll_async();
            }
        };
        auto second_look = [&]() {
            if (!pend && polled) {
                wait_vm<0>();
                take_poll();
                if (seen >> next & 1u) { request(false); pend = true; }
                poll_async();
            }
        };
        auto flip = [&](unsigned pos, bool on) {
            if (on) asm volatile("ds_xor_b32 %0, %1" :: "v"(aslot + (pos & 0xffcu)), "v"(0x40u << ((pos & 3u) * 8u)) : "memory");
        };
        auto expand_from = [&](auto rc, unsigned m, int k) {
            constexpr int R = decltype(rc)::v;
            const unsigned oa = aslot + v16;
            if (m != META_RAW) {
                LAS unsigned char* o = (LAS unsigned char*)(size_t)oa;
                const unsigned b4 = (m & 0xffu) * 0x01010101u, cnt = m >> 8;
                unsigned d[16];
                unpack8<R>(d, b4, [](int) {});
                *(LAS u32x4*)o = (u32x4){d[0], d[1], d[2], d[3]}; *(LAS u32x4*)(o + 1024) = (u32x4){d[4], d[5], d[6], d[7]};
                __builtin_amdgcn_sched_barrier(0);
                unpack8<R + 7>(d + 8, b4, [](int) {});
                *(LAS u32x4*)(o + 2048) = (u32x4){d[8], d[9], d[10], d[11]}; *(LAS u32x4*)(o + 3072) = (u32x4){d[12], d[13], d[14], d[15]};
                flip(reg_get<R + 16>(), (unsigned)lane < cnt);
                for (unsigned c0 = 64; c0 < cnt && c0 < (unsigned)EXC_MAX; c0 += 64) {
                    unsigned e;
                    const unsigned eo = (c0 + lane) * 2;
                    const unsigned char* ea = xr + (size_t)k * 4 * ITEM;
                    asm volatile("global_load_ushort %0, %1, %2 offset:3584 sc0 sc1\n\ts_waitcnt vmcnt(0)" : "=&v"(e) : "v"(eo), "s"(ea) : "memory");
                    flip(e, c0 + lane < cnt);
                }
            }
        };
        auto expand = [&](unsigned m, int k) {
            if (k & 1) expand_from(IC<140>{}, m, k); else expand_from(IC<120>{}, m, k);
            advance(aslot, lds0 + h * ITEM);
        };
        auto settle = [&]() {
            if (behind == 1) wait_vm<NW>(); else if (behind == 3) wait_vm<NW + 4>(); else if (behind == 2) wait_vm<4>(); else wait_vm<0>();
        };
        auto finish = [&](int s) {
            if (s == pfs) {
                prefetch_row(pfp); prefetch_row(pfp + ND);
                behind |= 2;
                pfs = pfs + 7 + PF_AHEAD < NSTEP ? pfs + 7 : NSTEP;
                pfp += (size_t)7 * BK * ND;
            }
            barrier();
        };
        auto fast = [&](auto rc, auto rn, int s) {
            constexpr int RC = decltype(rc)::v, RN = decltype(rn)::v;
            if (!(seen >> next & 1u)) {
                settle(); take_poll(); polled = true;
                if (!(seen >> next & 1u)) return false;
            }
            const unsigned mn = __builtin_amdgcn_readlane(fv.x, next);
            if (mn >= FAST_META) return false;
            const unsigned b4 = (meta & 0xffu) * 0x01010101u, cnt = meta >> 8, oa = aslot + v16;
            const unsigned char* a = xr + (size_t)next * 4 * ITEM;
            const bool pf = behind & 2;
            if (behind & 1) { if (pf) wait_vm<NW + 8>(); else wait_vm<NW + 4>(); } else { if (pf) wait_vm<8>(); else wait_vm<4>(); }
            unsigned d[16];
            LAS unsigned char* o = (LAS unsigned char*)(size_t)oa;
            load16<RN, 0>(a, v16);
            unpack8<RC>(d, b4, [&](int i) { if (i == 0) load16<RN + 4, 1024>(a, v16); else load16<RN + 8, 2048>(a, v16); });
            *(LAS u32x4*)o = (u32x4){d[0], d[1], d[2], d[3]}; *(LAS u32x4*)(o + 1024) = (u32x4){d[4], d[5], d[6], d[7]};
            if (behind & 1) { if (pf) wait_vm<NW + 7>(); else wait_vm<NW + 3>(); } else { if (pf) wait_vm<7>(); else wait_vm<3>(); }
            take_poll();
            load16<RN + 12, 3072>(a, vt);
            unpack8<RC + 7>(d + 8, b4, [&](int i) { if (i == 0) load2<RN + 16>(a, ve); else poll_async(); });
            *(LAS u32x4*)(o + 2048) = (u32x4){d[8], d[9], d[10], d[11]}; *(LAS u32x4*)(o + 3072) = (u32x4){d[12], d[13], d[14], d[15]};
            flip(reg_get<RC + 16>(), (unsigned)lane < cnt);
            advance(aslot, lds0 + h * ITEM);
            meta = mn; ++next;
            if (NW) {
                asm volatile("s_waitcnt lgkmcnt(0)" ::: "memory");
                extra(s + 2);
                if (pf) wait_vm<NW + 10>(); else wait_vm<NW + 6>();
            }
            behind = NW ? 1 : 0;
            finish(s);
            return true;
        };
        auto fast_raw = [&](int s) {
            if (!(seen >> next & 1u)) {
                settle(); take_poll(); polled = true;
                if (!(seen >> next & 1u)) return false;
            }
            const bool pf = behind & 2;
            request(true);
            advance(aslot, lds0 + h * ITEM);
            if (behind & 1) { if (pf) wait_vm<NW + 9>(); else wait_vm<NW + 5>(); } else { if (pf) wait_vm<9>(); else wait_vm<5>(); }
            take_poll(); poll_async();
            if (NW) { extra(s + 2); if (pf) wait_vm<NW + 10>(); else wait_vm<NW + 6>(); }
            behind = NW ? 1 : 0;
            finish(s);
            return true;
        };
#pragma unroll 1
        for (int s = -1; s < NSTEP - 1; ++s) {
#pragma unroll 1
            while (pend && meta == META_RAW && next < NSTEP && fast_raw(s)) ++s;
            if (pend && meta < FAST_META) {
                if (!(s & 1)) {
                    if (next < NSTEP && fast(IC<140>{}, IC<120>{}, s)) ++s; else goto general;
                }
#pragma unroll 1
                while (next < NSTEP && fast(IC<120>{}, IC<140>{}, s)) {
                    ++s;
                    if (next < NSTEP && fast(IC<140>{}, IC<120>{}, s)) ++s; else break;
                }
            }
        general:
            settle();
            bool edue = s >= 0 && s + 2 < NSTEP;
            behind = 0;
            if (polled) take_poll();
            if (pend) {
                const unsigned m = meta;
                pend = false;
                ahead();
                expand(m, s + 1);
                second_look();
                if (!edue) { if (s > 0) wait_vm<0>(); }
                else {
                    asm volatile("s_waitcnt lgkmcnt(0)" ::: "memory");
                    extra(s + 2); edue = false; behind = 1;
                    if (pend) wait_vm<NW + 6>(); else wait_vm<NW + 1>();
                }
            } else if (!dead) {
                if (!(seen >> next & 1u)) {
                    if (edue) { extra(s + 2); edue = false; }
                    const unsigned long long t0 = __builtin_amdgcn_s_memrealtime();
                    for (unsigned polls = 1;; ++polls) {
                        asm volatile("global_load_dwordx2 %0, %1, %2 sc0 sc1\n\ts_waitcnt vmcnt(0)" : "=&v"(fv) : "v"(v8), "s"(fl) : "memory");
                        look();
                        if (seen >> next & 1u) break;
                        if ((polls & 63u) == 0u
                            && (__builtin_amdgcn_readfirstlane(__hip_atomic_load(tmo, __ATOMIC_RELAXED, __HIP_MEMORY_SCOPE_AGENT)) == call
                                || __builtin_amdgcn_s_memrealtime() - t0 > DEADLINE)) {
                            if (lane == 0) { __hip_atomic_store(tmo, call, __ATOMIC_RELAXED, __HIP_MEMORY_SCOPE_AGENT); lds_st(lds0 + SY_BAD, 1u); }
                            dead = true;
                            break;
                        }
                        __builtin_amdgcn_s_sleep(2);
                    }
                }
                if (!dead) {
                    request(false); poll_async();
                    wait_vm<0>();
                    const unsigned m = meta;
                    take_poll();
                    ahead();
                    expand(m, s + 1);
                    second_look();
                }
            }
            if (edue) {
                asm volatile("s_waitcnt lgkmcnt(0)" ::: "memory");
                extra(s + 2); behind = 1;
            }
            finish(s);
        }
        wait_vm<0>();
        };
        if (ex) remote(IC<1>{}); else remote(IC<0>{});
        return;
    }

    const int wr = wid >> 2, wc = wid & 3;
    unsigned sx = __builtin_amdgcn_readfirstlane(__float_as_uint(p.sx[0])), sw = __builtin_amdgcn_readfirstlane(__float_as_uint(p.sw[0]));
    int ex = 2 - t, top = 1 - wr;
    f32x4 acc[4][4], accx[2];
#pragma unroll
    for (int m = 0; m < 4; ++m)
#pragma unroll
        for (int n = 0; n < 4; ++n) acc[m][n] = (f32x4){0.f, 0.f, 0.f, 0.f};
    accx[0] = accx[1] = (f32x4){0.f, 0.f, 0.f, 0.f};
    const float* ob = p.out + (size_t)(src * MS + t * BM + wr * 64) * NS + nt * BN + wc * 64;
    const float* ox = p.out + (size_t)(p.rank * MS + xrow) * NS + nt * BN + wc * 64 + wr * 32;
    asm volatile("" : "+s"(ob), "+s"(ox), "+s"(sx), "+s"(sw), "+s"(ex), "+s"(top));
    const unsigned char* wp = p.w + nt * BN + (size_t)wid * 16 * ND;
    unsigned wslot = lds0 + B_OFF + wid * 4096;
    unsigned wo0 = w_off(lane, 0, wid), wo1 = w_off(lane, 1, wid);
    auto issue_w = [&]() {
        w_loads(wp, wo0, wo1, wslot);
        wp += (size_t)BK * ND;
        wslot = wslot >= lds0 + B_OFF + (NSLOT - 1) * SLOT_BYTES ? wslot - (NSLOT - 1) * SLOT_BYTES : wslot + SLOT_BYTES;
    };
    issue_w(); issue_w(); wait_vm<4>();
    asm volatile("" : "+v"(wo0), "+v"(wo1));
    unsigned aa[2], ea[2], bo[4];
    {
        const int fr = lane & 15, fq = lane >> 4, q = fr >> 1, f = q | ((fq & 1) << 3);
        aa[0] = lds0 + (unsigned)((wr * 64 + fr) * 128 + ((fq ^ ((fr >> 1) & 7)) << 4));
        aa[1] = aa[0] ^ 64u;
        ea[0] = aa[0] + (unsigned)(E_OFF - wr * 8192);
        ea[1] = ea[0] ^ 64u;
#pragma unroll
        for (int n = 0; n < 4; ++n) bo[n] = lds0 + (unsigned)((16 * fq + q) * 256 + (((wc * 4 + (n ^ (2 * wr))) ^ f) << 4) + (fr & 1) * 8);
    }
    asm volatile(".p2align 8\n\ts_barrier" ::: "memory");
    {
        int slot = 0;
#pragma unroll 1
        for (int s = 0; s < NSTEP; ++s) {
            v8i af[4], bf[4], ef;
#pragma unroll
            for (int n = 0; n < 4; ++n) {
                LAS unsigned char* q = (LAS unsigned char*)(size_t)bo[n] + B_OFF;
                const v2i t0 = __builtin_amdgcn_ds_read_tr8_b64_v2i32((LAS v2i*)q);
                const v2i t1 = __builtin_amdgcn_ds_read_tr8_b64_v2i32((LAS v2i*)(q + 8 * 256));
                const v2i t2 = __builtin_amdgcn_ds_read_tr8_b64_v2i32((LAS v2i*)(q + 64 * 256));
                const v2i t3 = __builtin_amdgcn_ds_read_tr8_b64_v2i32((LAS v2i*)(q + 72 * 256));
                bf[n] = (v8i){t0.x, t0.y, t1.x, t1.y, t2.x, t2.y, t3.x, t3.y};
            }
#pragma unroll
            for (int m = 0; m < 4; ++m) {
                const v4i lo = *(LAS v4i*)((LAS unsigned char*)(size_t)aa[0] + m * 2048), hi = *(LAS v4i*)((LAS unsigned char*)(size_t)aa[1] + m * 2048);
                af[m] = (v8i){lo.x, lo.y, lo.z, lo.w, hi.x, hi.y, hi.z, hi.w};
            }
            asm volatile("" : "+s"(ex));
            if (ex > 0) {
                const v4i lo = *(LAS v4i*)(size_t)ea[0], hi = *(LAS v4i*)(size_t)ea[1];
                ef = (v8i){lo.x, lo.y, lo.z, lo.w, hi.x, hi.y, hi.z, hi.w};
            }
            int wdue = s + 2 < NSTEP ? 1 + top : 0;
            asm volatile("" : "+s"(wdue));
            if (wdue == 1) issue_w();
            asm volatile("" ::: "memory");
            __builtin_amdgcn_s_waitcnt(0xC07F);
            asm volatile("" : "+s"(wdue));
            if (wdue == 2) issue_w();
            slot = (slot == NSLOT - 1) ? 0 : slot + 1;
            const int delta = slot ? SLOT_BYTES : -(NSLOT - 1) * SLOT_BYTES;
            int bar = s < NSTEP - 1 ? 1 + top : 0;
            asm volatile("" : "+s"(bar));
            if (bar == 1) { if (wdue > 0) wait_vm<4>(); else wait_vm<0>(); asm volatile("s_barrier" ::: "memory"); }
            __builtin_amdgcn_sched_barrier(0);
            asm volatile("s_setprio 1\n\t.p2align 3");
            __builtin_amdgcn_sched_barrier(0);
#pragma unroll
            for (int m = 0; m < 4; ++m) {
#pragma unroll
                for (int n = 0; n < 4; ++n) {
                    acc[m][n] = __builtin_amdgcn_mfma_scale_f32_16x16x128_f8f6f4(bf[n], af[m], acc[m][n], 0, 0, 0, 0, 0, 0);
                    if (m == 0 && n == 0) { __builtin_amdgcn_sched_barrier(0); __builtin_amdgcn_s_setprio(2); __builtin_amdgcn_sched_barrier(0); }
                    if (m == 3 && n == 1) {
                        __builtin_amdgcn_sched_barrier(0);
                        asm volatile("" : "+s"(bar));
                        if (bar == 2) { if (wdue > 0) wait_vm<4>(); else wait_vm<0>(); asm volatile("s_barrier" ::: "memory"); }
                        __builtin_amdgcn_sched_barrier(0);
                    }
                }
                __builtin_amdgcn_sched_barrier(0);
                if (m == 0) asm volatile("v_add_u32 %0, %4, %0\n\tv_add_u32 %1, %4, %1\n\tv_add_u32 %2, %4, %2\n\tv_add_u32 %3, %4, %3"
                                         : "+v"(bo[0]), "+v"(bo[1]), "+v"(bo[2]), "+v"(bo[3]) : "s"(delta));
                if (m == 1) asm volatile("v_add_u32 %0, %4, %0\n\tv_add_u32 %1, %4, %1\n\tv_add_u32 %2, %4, %2\n\tv_add_u32 %3, %4, %3"
                                         : "+v"(aa[0]), "+v"(aa[1]), "+v"(ea[0]), "+v"(ea[1]) : "s"(delta));
                __builtin_amdgcn_sched_barrier(0);
            }
            asm volatile("" : "+s"(ex));
            if (ex > 0) {
                accx[0] = __builtin_amdgcn_mfma_scale_f32_16x16x128_f8f6f4(bf[0], ef, accx[0], 0, 0, 0, 0, 0, 0);
                accx[1] = __builtin_amdgcn_mfma_scale_f32_16x16x128_f8f6f4(bf[1], ef, accx[1], 0, 0, 0, 0, 0, 0);
            }
            __builtin_amdgcn_s_setprio(0);
            __builtin_amdgcn_sched_barrier(0);
        }
    }
    float scale = __uint_as_float(sx) * __uint_as_float(sw);
    if (__builtin_amdgcn_readfirstlane(lds_ld(lds0 + SY_BAD)) != 0u) scale = __builtin_nanf("");
    int ln;
    asm volatile("v_mbcnt_lo_u32_b32 %0, -1, 0\n\tv_mbcnt_hi_u32_b32 %0, -1, %0" : "=v"(ln));
    const int fr = ln & 15, fq = ln >> 4;
    const float zs = scale * -1.44269504088896340736f, zmax = 60.f * 1.44269504088896340736f;
    LAS unsigned char* tb = (LAS unsigned char*)smem + ((NSTEP - 3 + (wc >> 1)) % NSLOT) * SLOT_BYTES + wr * 8192 + (wc & 1) * 4096;
    f32x2 e[8];
    f32x4 tv[4];
    auto expo = [&](auto nqc, const f32x4* c) {
        constexpr int NQ = decltype(nqc)::v;
#pragma unroll
        for (int h = 0; h < 2 * NQ; ++h) {
            f32x2 z = (f32x2){c[h >> 1][(h & 1) * 2], c[h >> 1][(h & 1) * 2 + 1]} * zs;
            asm("" : "+v"(z));
            e[h].x = __builtin_amdgcn_exp2f(__builtin_amdgcn_fmed3f(z.x, -zmax, zmax));
            e[h].y = __builtin_amdgcn_exp2f(__builtin_amdgcn_fmed3f(z.y, -zmax, zmax));
        }
        __builtin_amdgcn_sched_barrier(0);
    };
    auto stage = [&](auto nqc, const f32x4* c) {
        constexpr int NQ = decltype(nqc)::v;
#pragma unroll
        for (int q = 0; q < NQ; ++q) {
            f32x2 d0 = e[2 * q] + 1.0f, d1 = e[2 * q + 1] + 1.0f;
            asm("" : "+v"(d0)); asm("" : "+v"(d1));
            const f32x4 r = {__builtin_amdgcn_rcpf(d0.x), __builtin_amdgcn_rcpf(d0.y), __builtin_amdgcn_rcpf(d1.x), __builtin_amdgcn_rcpf(d1.y)};
            const int cb = NQ == 4 ? (q ^ (2 * wr)) : q;
            *(LAS f32x4*)(tb + fr * 256 + (((cb * 4 + fq) ^ (fr & 7)) << 4)) = (c[q] * scale) * r;
        }
        constexpr int LPR = NQ * 4, RPI = 64 / LPR;
#pragma unroll
        for (int i = 0; i < NQ; ++i) {
            const int r = i * RPI + ln / LPR;
            tv[i] = *(LAS f32x4*)(tb + r * 256 + (((ln % LPR) ^ (r & 7)) << 4));
        }
        __builtin_amdgcn_sched_barrier(0);
    };
    auto put = [&](auto nqc, const float* rowp) {
        constexpr int NQ = decltype(nqc)::v, LPR = NQ * 4, RPI = 64 / LPR;
        const unsigned ovoff = (unsigned)((ln / LPR) * NS * 4 + (ln % LPR) * 16);
#pragma unroll
        for (int i = 0; i < NQ; ++i)
            asm volatile("global_store_dwordx4 %0, %1, %2 sc1\n\ts_nop 1" :: "v"(ovoff), "v"(tv[i]), "s"(rowp + (size_t)i * RPI * NS) : "memory");
        __builtin_amdgcn_sched_barrier(0);
    };
    expo(IC<4>{}, acc[0]);
#pragma unroll
    for (int m = 0; m < 4; ++m) {
        stage(IC<4>{}, acc[m]);
        if (m < 3) expo(IC<4>{}, acc[m + 1]);
        else if (has_extra) expo(IC<2>{}, accx);
        put(IC<4>{}, ob + (size_t)m * 16 * NS);
    }
    if (has_extra) { stage(IC<2>{}, accx); put(IC<2>{}, ox); }
}

}

extern "C" void kernel_launch(int rank, int nranks, ncclComm_t comm,
                              hipStream_t stream,
                              void* const* d_in, const int* in_sizes, int n_in,
                              void* d_out, int out_size,
                              const int* in_dtypes, int out_dtype,
                              void* const* d_out_all,
                              void* const* d_ws_all, size_t ws_bytes) {
    (void)comm; (void)in_dtypes; (void)out_dtype; (void)d_out_all;
    if (nranks != NR || (unsigned)rank >= (unsigned)NR || n_in != 4 || ws_bytes < WS_END || in_sizes[0] != MS * KD
        || in_sizes[1] != KD * ND || out_size != NR * MS * NS) {
        fprintf(stderr, "kernel_launch: unexpected problem shape\n");
        return;
    }
    HIP_CHECK(hipFuncSetAttribute((const void*)fused_ag_gemm, hipFuncAttributeMaxDynamicSharedMemorySize, LDS_BYTES));
    Params p;
    p.x = (const unsigned char*)d_in[0];
    p.w = (const unsigned char*)d_in[1] + (size_t)rank * NS;
    p.sx = (const float*)d_in[2];
    p.sw = (const float*)d_in[3];
    p.out = (float*)d_out;
    p.me = (unsigned char*)d_ws_all[rank];
    for (int i = 0; i < NR - 1; ++i) p.peer[i] = (unsigned char*)d_ws_all[(rank + 1 + i) % NR];
    p.rank = rank;
    p.pad = 0;
    hipLaunchKernelGGL(fused_ag_gemm, dim3(NWG), dim3(NTHREADS), LDS_BYTES, stream, p);
    HIP_CHECK(hipGetLastError());
}
```
